```python
import numpy as np

kernel_name = "gpu_a2a_gemm_m4096_k4096_n16384_e4m3e5m2_dqsilu"


ABSMAX_RTOL = 0.05

COMPUTE_DTYPE = "float32"
EPILOGUE = "dequant_silu"
IN_QUANT = None
OUT_QUANT = None
IN_FP8 = "e4m3e5m2"
IN_DTYPES = {"x": "e4m3", "w_mat": "e5m2",
             "scale_x": "float32", "scale_w": "float32"}
WORLD_SIZE = 8

def _snap_fp8(x, mant_bits, max_abs, min_normal):
    a = np.abs(x.astype(np.float64))
    _m, _e = np.frexp(a)
    q = np.where(a >= min_normal, np.ldexp(1.0, _e - 1 - mant_bits),
                 min_normal * 2.0 ** -mant_bits)
    snapped = np.minimum(np.round(a / q) * q, max_abs)
    return (np.sign(x) * snapped).astype(np.float32)

def setup_inputs():

    x = _snap_fp8(np.random.randn(4096, 4096).astype(np.float32), 3, 448.0, 2.0 ** -6)
    w_mat = _snap_fp8(np.random.randn(4096, 16384).astype(np.float32), 2, 57344.0, 2.0 ** -14)
    scale_x = (0.02 * (1.0 + np.random.rand(1))).astype(np.float32)
    scale_w = (0.02 * (1.0 + np.random.rand(1))).astype(np.float32)
    return {"x": x, "w_mat": w_mat, "scale_x": scale_x, "scale_w": scale_w}

def reference(x, w_mat, scale_x, scale_w):
    acc = (x @ w_mat).astype(np.float32)
    y = (acc * (scale_x * scale_w)).astype(np.float32)
    return (y / (1.0 + np.exp(-np.clip(y, -60.0, 60.0)))).astype(np.float32)
```

```cpp
#include <hip/hip_runtime.h>
#include <rccl/rccl.h>
#include <cstdio>

#define HIP_CHECK(expr) do { hipError_t _e = (expr); if (_e != hipSuccess) { \
    fprintf(stderr, "HIP error %s at %s:%d\n", hipGetErrorString(_e), __FILE__, __LINE__); \
    return; } } while (0)

typedef unsigned v2u __attribute__((ext_vector_type(2)));
typedef unsigned v4u __attribute__((ext_vector_type(4)));
typedef unsigned v16u __attribute__((ext_vector_type(16)));
#define LAS __attribute__((address_space(3)))
#define DI __device__ __forceinline__

#define STPOL "sc1"
#define EPISLEEP ""
#define PREEPI ""
#define STAGEPAD ""
#define NIPAD ""
#define P1PAD ""
#define APOL ""
#define XPOL ""
#define COMPUTE_ASM \
  "s_mov_b64 s[40:41], %[xloc]\n\t" \
  "v_mbcnt_lo_u32_b32 v20, -1, 0\n\t" \
  "v_mbcnt_hi_u32_b32 v20, -1, v20\n\t" \
  "s_lshl_b32 s57, %[wid], 13\n\t" \
  "s_add_u32 s57, s57, 67584\n\t" \
  "v_lshrrev_b32 v21, 3, v20\n\t" \
  "v_lshrrev_b32 v22, 4, v20\n\t" \
  "v_and_b32 v23, 7, v20\n\t" \
  "v_xor_b32 v24, v23, v22\n\t" \
  "v_lshlrev_b32 v24, 4, v24\n\t" \
  "v_lshl_add_u32 v255, v21, 9, v24\n\t" \
  "s_add_u32 s68, s40, 0\n\t" \
  "s_addc_u32 s69, s41, 0\n\t" \
  "s_add_u32 m0, s57, 32768\n\t" \
  "s_nop 0\n\t" \
  "global_load_lds_dwordx4 v255, s[68:69] " XPOL "\n\t" \
  "s_add_u32 s68, s40, 4096\n\t" \
  "s_addc_u32 s69, s41, 0\n\t" \
  "s_add_u32 m0, s57, 33792\n\t" \
  "s_nop 0\n\t" \
  "global_load_lds_dwordx4 v255, s[68:69] " XPOL "\n\t" \
  "s_add_u32 s68, s40, 8192\n\t" \
  "s_addc_u32 s69, s41, 0\n\t" \
  "s_add_u32 m0, s57, 34816\n\t" \
  "s_nop 0\n\t" \
  "global_load_lds_dwordx4 v255, s[68:69] " XPOL "\n\t" \
  "s_add_u32 s68, s40, 12288\n\t" \
  "s_addc_u32 s69, s41, 0\n\t" \
  "s_add_u32 m0, s57, 35840\n\t" \
  "s_nop 0\n\t" \
  "global_load_lds_dwordx4 v255, s[68:69] " XPOL "\n\t" \
  "s_add_u32 s68, s40, 16384\n\t" \
  "s_addc_u32 s69, s41, 0\n\t" \
  "s_add_u32 m0, s57, 36864\n\t" \
  "s_nop 0\n\t" \
  "global_load_lds_dwordx4 v255, s[68:69] " XPOL "\n\t" \
  "s_add_u32 s68, s40, 20480\n\t" \
  "s_addc_u32 s69, s41, 0\n\t" \
  "s_add_u32 m0, s57, 37888\n\t" \
  "s_nop 0\n\t" \
  "global_load_lds_dwordx4 v255, s[68:69] " XPOL "\n\t" \
  "s_add_u32 s68, s40, 24576\n\t" \
  "s_addc_u32 s69, s41, 0\n\t" \
  "s_add_u32 m0, s57, 38912\n\t" \
  "s_nop 0\n\t" \
  "global_load_lds_dwordx4 v255, s[68:69] " XPOL "\n\t" \
  "s_add_u32 s68, s40, 28672\n\t" \
  "s_addc_u32 s69, s41, 0\n\t" \
  "s_add_u32 m0, s57, 39936\n\t" \
  "s_nop 0\n\t" \
  "global_load_lds_dwordx4 v255, s[68:69] " XPOL "\n\t" \
  "s_mov_b64 s[42:43], %[wsa]\n\t" \
  "s_mov_b64 s[44:45], %[out]\n\t" \
  "s_load_dword s48, %[sx], 0x0\n\t" \
  "s_load_dword s49, %[sw], 0x0\n\t" \
  "s_lshl_b32 s50, %[wid], 8\n\t" \
  "s_mov_b32 s56, 0x1000000\n\t" \
  "s_mov_b32 s59, 49152\n\t" \
  "v_lshlrev_b32 v252, 4, v20\n\t" \
  "v_and_b32 v25, 15, v20\n\t" \
  "v_lshrrev_b32 v26, 1, v25\n\t" \
  "v_and_b32 v30, 3, v26\n\t" \
  "v_xor_b32 v27, v22, v30\n\t" \
  "v_lshlrev_b32 v27, 4, v27\n\t" \
  "v_lshl_add_u32 v27, v25, 7, v27\n\t" \
  "v_add_u32 v254, s57, v27\n\t" \
  "v_lshrrev_b32 v28, 2, v26\n\t" \
  "v_lshl_add_u32 v29, v22, 2, v28\n\t" \
  "v_lshlrev_b32 v29, 9, v29\n\t" \
  "v_lshl_add_u32 v29, v30, 6, v29\n\t" \
  "v_and_b32 v30, 1, v22\n\t" \
  "v_lshl_add_u32 v30, v30, 1, v28\n\t" \
  "v_lshl_add_u32 v29, v30, 4, v29\n\t" \
  "v_and_b32 v30, 1, v20\n\t" \
  "v_lshl_add_u32 v248, v30, 3, v29\n\t" \
  "v_xor_b32 v249, 16, v248\n\t" \
  "v_xor_b32 v250, 32, v248\n\t" \
  "v_xor_b32 v251, 48, v248\n\t" \
  "s_mov_b32 s51, 2\n\t" \
  "s_mov_b32 s52, 2\n\t" \
  "s_mov_b32 s58, 0\n\t" \
  "s_mov_b32 s70, 0\n\t" \
  "s_waitcnt vmcnt(0)\n\t" \
  "ds_read_b128 v[128:131], v254 offset:32768\n\t" \
  "ds_read_b128 v[132:135], v254 offset:32832\n\t" \
  "ds_read_b128 v[136:139], v254 offset:34816\n\t" \
  "ds_read_b128 v[140:143], v254 offset:34880\n\t" \
  "ds_read_b128 v[144:147], v254 offset:36864\n\t" \
  "ds_read_b128 v[148:151], v254 offset:36928\n\t" \
  "ds_read_b128 v[152:155], v254 offset:38912\n\t" \
  "ds_read_b128 v[156:159], v254 offset:38976\n\t" \
  "s_add_u32 s68, s40, 128\n\t" \
  "s_addc_u32 s69, s41, 0\n\t" \
  "s_add_u32 m0, s57, 0\n\t" \
  "s_nop 0\n\t" \
  "global_load_lds_dwordx4 v255, s[68:69] " XPOL "\n\t" \
  "s_add_u32 s68, s40, 4224\n\t" \
  "s_addc_u32 s69, s41, 0\n\t" \
  "s_add_u32 m0, s57, 1024\n\t" \
  "s_nop 0\n\t" \
  "global_load_lds_dwordx4 v255, s[68:69] " XPOL "\n\t" \
  "s_add_u32 s68, s40, 8320\n\t" \
  "s_addc_u32 s69, s41, 0\n\t" \
  "s_add_u32 m0, s57, 2048\n\t" \
  "s_nop 0\n\t" \
  "global_load_lds_dwordx4 v255, s[68:69] " XPOL "\n\t" \
  "s_add_u32 s68, s40, 12416\n\t" \
  "s_addc_u32 s69, s41, 0\n\t" \
  "s_add_u32 m0, s57, 3072\n\t" \
  "s_nop 0\n\t" \
  "global_load_lds_dwordx4 v255, s[68:69] " XPOL "\n\t" \
  "s_add_u32 s68, s40, 16512\n\t" \
  "s_addc_u32 s69, s41, 0\n\t" \
  "s_add_u32 m0, s57, 4096\n\t" \
  "s_nop 0\n\t" \
  "global_load_lds_dwordx4 v255, s[68:69] " XPOL "\n\t" \
  "s_add_u32 s68, s40, 20608\n\t" \
  "s_addc_u32 s69, s41, 0\n\t" \
  "s_add_u32 m0, s57, 5120\n\t" \
  "s_nop 0\n\t" \
  "global_load_lds_dwordx4 v255, s[68:69] " XPOL "\n\t" \
  "s_add_u32 s68, s40, 24704\n\t" \
  "s_addc_u32 s69, s41, 0\n\t" \
  "s_add_u32 m0, s57, 6144\n\t" \
  "s_nop 0\n\t" \
  "global_load_lds_dwordx4 v255, s[68:69] " XPOL "\n\t" \
  "s_add_u32 s68, s40, 28800\n\t" \
  "s_addc_u32 s69, s41, 0\n\t" \
  "s_add_u32 m0, s57, 7168\n\t" \
  "s_nop 0\n\t" \
  "global_load_lds_dwordx4 v255, s[68:69] " XPOL "\n\t" \
  "s_waitcnt lgkmcnt(0)\n\t" \
  "s_add_u32 s68, s40, 256\n\t" \
  "s_addc_u32 s69, s41, 0\n\t" \
  "s_add_u32 m0, s57, 32768\n\t" \
  "s_nop 0\n\t" \
  "global_load_lds_dwordx4 v255, s[68:69] " XPOL "\n\t" \
  "s_add_u32 s68, s40, 4352\n\t" \
  "s_addc_u32 s69, s41, 0\n\t" \
  "s_add_u32 m0, s57, 33792\n\t" \
  "s_nop 0\n\t" \
  "global_load_lds_dwordx4 v255, s[68:69] " XPOL "\n\t" \
  "s_add_u32 s68, s40, 8448\n\t" \
  "s_addc_u32 s69, s41, 0\n\t" \
  "s_add_u32 m0, s57, 34816\n\t" \
  "s_nop 0\n\t" \
  "global_load_lds_dwordx4 v255, s[68:69] " XPOL "\n\t" \
  "s_add_u32 s68, s40, 12544\n\t" \
  "s_addc_u32 s69, s41, 0\n\t" \
  "s_add_u32 m0, s57, 35840\n\t" \
  "s_nop 0\n\t" \
  "global_load_lds_dwordx4 v255, s[68:69] " XPOL "\n\t" \
  "s_add_u32 s68, s40, 16640\n\t" \
  "s_addc_u32 s69, s41, 0\n\t" \
  "s_add_u32 m0, s57, 36864\n\t" \
  "s_nop 0\n\t" \
  "global_load_lds_dwordx4 v255, s[68:69] " XPOL "\n\t" \
  "s_add_u32 s68, s40, 20736\n\t" \
  "s_addc_u32 s69, s41, 0\n\t" \
  "s_add_u32 m0, s57, 37888\n\t" \
  "s_nop 0\n\t" \
  "global_load_lds_dwordx4 v255, s[68:69] " XPOL "\n\t" \
  "s_add_u32 s68, s40, 24832\n\t" \
  "s_addc_u32 s69, s41, 0\n\t" \
  "s_add_u32 m0, s57, 38912\n\t" \
  "s_nop 0\n\t" \
  "global_load_lds_dwordx4 v255, s[68:69] " XPOL "\n\t" \
  "s_add_u32 s68, s40, 28928\n\t" \
  "s_addc_u32 s69, s41, 0\n\t" \
  "s_add_u32 m0, s57, 39936\n\t" \
  "s_nop 0\n\t" \
  "global_load_lds_dwordx4 v255, s[68:69] " XPOL "\n\t" \
  "v_mov_b32 v31, s48\n\t" \
  "v_mul_f32 v31, s49, v31\n\t" \
  "v_mul_f32 v30, 0xbfb8aa3b, v31\n\t" \
  "v_readfirstlane_b32 s48, v31\n\t" \
  "v_readfirstlane_b32 s49, v30\n\t" \
  "s_mov_b32 m0, 0\n\t" \
  "s_nop 0\n\t" \
  "ds_read_addtid_b32 v253 offset:0\n\t" \
  "s_waitcnt lgkmcnt(0)\n\t" \
  "Lc1_retry%=:\n\t" \
  "v_readlane_b32 s53, v253, 32\n\t" \
  "s_nop 0\n\t" \
  "s_cmp_lt_u32 s53, s51\n\t" \
  "s_cbranch_scc1 Lc1_spin%=\n\t" \
  "Lc1_ok%=:\n\t" \
  "ds_read_b64_tr_b8 v[224:225], v248 offset:2048\n\t" \
  "ds_read_b64_tr_b8 v[226:227], v248 offset:3072\n\t" \
  "ds_read_b64_tr_b8 v[228:229], v248 offset:10240\n\t" \
  "ds_read_b64_tr_b8 v[230:231], v248 offset:11264\n\t" \
  "ds_read_b64_tr_b8 v[232:233], v249 offset:2048\n\t" \
  "ds_read_b64_tr_b8 v[234:235], v249 offset:3072\n\t" \
  "ds_read_b64_tr_b8 v[236:237], v249 offset:10240\n\t" \
  "ds_read_b64_tr_b8 v[238:239], v249 offset:11264\n\t" \
  "" STAGEPAD "\n\t" \
  "s_waitcnt lgkmcnt(4)\n\t" \
  "v_mfma_f32_16x16x128_f8f6f4 v[0:3], v[128:135], v[224:231], 0 blgp:1\n\t" \
  "ds_read_b64_tr_b8 v[240:241], v250 offset:2048\n\t" \
  "ds_read_b64_tr_b8 v[242:243], v250 offset:3072\n\t" \
  "v_mfma_f32_16x16x128_f8f6f4 v[32:35], v[136:143], v[224:231], 0 blgp:1\n\t" \
  "ds_read_b64_tr_b8 v[244:245], v250 offset:10240\n\t" \
  "v_mfma_f32_16x16x128_f8f6f4 v[64:67], v[144:151], v[224:231], 0 blgp:1\n\t" \
  "ds_read_b64_tr_b8 v[246:247], v250 offset:11264\n\t" \
  "" NIPAD "\n\t" \
  "v_mfma_f32_16x16x128_f8f6f4 v[96:99], v[152:159], v[224:231], 0 blgp:1\n\t" \
  "s_waitcnt lgkmcnt(4)\n\t" \
  "v_mfma_f32_16x16x128_f8f6f4 v[4:7], v[128:135], v[232:239], 0 blgp:1\n\t" \
  "ds_read_b64_tr_b8 v[224:225], v251 offset:2048\n\t" \
  "ds_read_b64_tr_b8 v[226:227], v251 offset:3072\n\t" \
  "v_mfma_f32_16x16x128_f8f6f4 v[36:39], v[136:143], v[232:239], 0 blgp:1\n\t" \
  "ds_read_b64_tr_b8 v[228:229], v251 offset:10240\n\t" \
  "v_mfma_f32_16x16x128_f8f6f4 v[68:71], v[144:151], v[232:239], 0 blgp:1\n\t" \
  "ds_read_b64_tr_b8 v[230:231], v251 offset:11264\n\t" \
  "" NIPAD "\n\t" \
  "v_mfma_f32_16x16x128_f8f6f4 v[100:103], v[152:159], v[232:239], 0 blgp:1\n\t" \
  "s_waitcnt lgkmcnt(4)\n\t" \
  "v_mfma_f32_16x16x128_f8f6f4 v[8:11], v[128:135], v[240:247], 0 blgp:1\n\t" \
  "ds_read_b64_tr_b8 v[232:233], v248 offset:2304\n\t" \
  "ds_read_b64_tr_b8 v[234:235], v248 offset:3328\n\t" \
  "v_mfma_f32_16x16x128_f8f6f4 v[40:43], v[136:143], v[240:247], 0 blgp:1\n\t" \
  "ds_read_b64_tr_b8 v[236:237], v248 offset:10496\n\t" \
  "v_mfma_f32_16x16x128_f8f6f4 v[72:75], v[144:151], v[240:247], 0 blgp:1\n\t" \
  "ds_read_b64_tr_b8 v[238:239], v248 offset:11520\n\t" \
  "" NIPAD "\n\t" \
  "v_mfma_f32_16x16x128_f8f6f4 v[104:107], v[152:159], v[240:247], 0 blgp:1\n\t" \
  "s_waitcnt lgkmcnt(4)\n\t" \
  "s_waitcnt vmcnt(8)\n\t" \
  "v_mfma_f32_16x16x128_f8f6f4 v[12:15], v[128:135], v[224:231], 0 blgp:1\n\t" \
  "ds_read_b64_tr_b8 v[240:241], v249 offset:2304\n\t" \
  "ds_read_b64_tr_b8 v[242:243], v249 offset:3328\n\t" \
  "v_mfma_f32_16x16x128_f8f6f4 v[44:47], v[136:143], v[224:231], 0 blgp:1\n\t" \
  "ds_read_b64_tr_b8 v[244:245], v249 offset:10496\n\t" \
  "ds_read_b128 v[160:163], v254 offset:0\n\t" \
  "v_mfma_f32_16x16x128_f8f6f4 v[76:79], v[144:151], v[224:231], 0 blgp:1\n\t" \
  "ds_read_b64_tr_b8 v[246:247], v249 offset:11520\n\t" \
  "" NIPAD "\n\t" \
  "v_mfma_f32_16x16x128_f8f6f4 v[108:111], v[152:159], v[224:231], 0 blgp:1\n\t" \
  "ds_read_b128 v[164:167], v254 offset:64\n\t" \
  "s_waitcnt lgkmcnt(6)\n\t" \
  "v_mfma_f32_16x16x128_f8f6f4 v[16:19], v[128:135], v[232:239], 0 blgp:1\n\t" \
  "ds_read_b64_tr_b8 v[224:225], v250 offset:2304\n\t" \
  "ds_read_b64_tr_b8 v[226:227], v250 offset:3328\n\t" \
  "v_mfma_f32_16x16x128_f8f6f4 v[48:51], v[136:143], v[232:239], 0 blgp:1\n\t" \
  "ds_read_b64_tr_b8 v[228:229], v250 offset:10496\n\t" \
  "ds_read_b128 v[168:171], v254 offset:2048\n\t" \
  "v_mfma_f32_16x16x128_f8f6f4 v[80:83], v[144:151], v[232:239], 0 blgp:1\n\t" \
  "ds_read_b64_tr_b8 v[230:231], v250 offset:11520\n\t" \
  "" NIPAD "\n\t" \
  "v_mfma_f32_16x16x128_f8f6f4 v[112:115], v[152:159], v[232:239], 0 blgp:1\n\t" \
  "ds_read_b128 v[172:175], v254 offset:2112\n\t" \
  "s_mov_b32 m0, 0\n\t" \
  "s_add_u32 s51, s51, 2\n\t" \
  "ds_read_addtid_b32 v253 offset:0\n\t" \
  "s_add_u32 s52, s52, 1\n\t" \
  "s_waitcnt lgkmcnt(8)\n\t" \
  "v_mfma_f32_16x16x128_f8f6f4 v[20:23], v[128:135], v[240:247], 0 blgp:1\n\t" \
  "ds_read_b64_tr_b8 v[232:233], v251 offset:2304\n\t" \
  "ds_read_b64_tr_b8 v[234:235], v251 offset:3328\n\t" \
  "v_mfma_f32_16x16x128_f8f6f4 v[52:55], v[136:143], v[240:247], 0 blgp:1\n\t" \
  "ds_read_b64_tr_b8 v[236:237], v251 offset:10496\n\t" \
  "ds_read_b128 v[176:179], v254 offset:4096\n\t" \
  "v_mfma_f32_16x16x128_f8f6f4 v[84:87], v[144:151], v[240:247], 0 blgp:1\n\t" \
  "ds_read_b64_tr_b8 v[238:239], v251 offset:11520\n\t" \
  "" NIPAD "\n\t" \
  "v_mfma_f32_16x16x128_f8f6f4 v[116:119], v[152:159], v[240:247], 0 blgp:1\n\t" \
  "ds_read_b128 v[180:183], v254 offset:4160\n\t" \
  "s_waitcnt lgkmcnt(6)\n\t" \
  "Lc2_retry%=:\n\t" \
  "v_readlane_b32 s53, v253, 32\n\t" \
  "s_nop 0\n\t" \
  "s_cmp_lt_u32 s53, s51\n\t" \
  "s_cbranch_scc1 Lc2_spin%=\n\t" \
  "Lc2_ok%=:\n\t" \
  "v_mfma_f32_16x16x128_f8f6f4 v[24:27], v[128:135], v[224:231], 0 blgp:1\n\t" \
  "ds_read_b64_tr_b8 v[240:241], v248 offset:18432\n\t" \
  "ds_read_b64_tr_b8 v[242:243], v248 offset:19456\n\t" \
  "v_mfma_f32_16x16x128_f8f6f4 v[56:59], v[136:143], v[224:231], 0 blgp:1\n\t" \
  "ds_read_b64_tr_b8 v[244:245], v248 offset:26624\n\t" \
  "ds_read_b128 v[184:187], v254 offset:6144\n\t" \
  "v_mfma_f32_16x16x128_f8f6f4 v[88:91], v[144:151], v[224:231], 0 blgp:1\n\t" \
  "ds_read_b64_tr_b8 v[246:247], v248 offset:27648\n\t" \
  "" NIPAD "\n\t" \
  "v_mfma_f32_16x16x128_f8f6f4 v[120:123], v[152:159], v[224:231], 0 blgp:1\n\t" \
  "ds_read_b128 v[188:191], v254 offset:6208\n\t" \
  "s_waitcnt lgkmcnt(7)\n\t" \
  "v_mfma_f32_16x16x128_f8f6f4 v[28:31], v[128:135], v[232:239], 0 blgp:1\n\t" \
  "ds_read_b64_tr_b8 v[224:225], v249 offset:18432\n\t" \
  "ds_read_b64_tr_b8 v[226:227], v249 offset:19456\n\t" \
  "v_mfma_f32_16x16x128_f8f6f4 v[60:63], v[136:143], v[232:239], 0 blgp:1\n\t" \
  "ds_read_b64_tr_b8 v[228:229], v249 offset:26624\n\t" \
  "v_mfma_f32_16x16x128_f8f6f4 v[92:95], v[144:151], v[232:239], 0 blgp:1\n\t" \
  "ds_read_b64_tr_b8 v[230:231], v249 offset:27648\n\t" \
  "" NIPAD "\n\t" \
  "v_mfma_f32_16x16x128_f8f6f4 v[124:127], v[152:159], v[232:239], 0 blgp:1\n\t" \
  "s_waitcnt lgkmcnt(4)\n\t" \
  "" STAGEPAD "\n\t" \
  "s_waitcnt vmcnt(0)\n\t" \
  "v_mfma_f32_16x16x128_f8f6f4 v[0:3], v[160:167], v[240:247], v[0:3] blgp:1\n\t" \
  "ds_read_b64_tr_b8 v[232:233], v250 offset:18432\n\t" \
  "ds_read_b64_tr_b8 v[234:235], v250 offset:19456\n\t" \
  "ds_read_b128 v[192:195], v254 offset:32768\n\t" \
  "v_mfma_f32_16x16x128_f8f6f4 v[32:35], v[168:175], v[240:247], v[32:35] blgp:1\n\t" \
  "ds_read_b64_tr_b8 v[236:237], v250 offset:26624\n\t" \
  "ds_read_b128 v[196:199], v254 offset:32832\n\t" \
  "v_mfma_f32_16x16x128_f8f6f4 v[64:67], v[176:183], v[240:247], v[64:67] blgp:1\n\t" \
  "ds_read_b64_tr_b8 v[238:239], v250 offset:27648\n\t" \
  "ds_read_b128 v[200:203], v254 offset:34816\n\t" \
  "" NIPAD "\n\t" \
  "v_mfma_f32_16x16x128_f8f6f4 v[96:99], v[184:191], v[240:247], v[96:99] blgp:1\n\t" \
  "ds_read_b128 v[204:207], v254 offset:34880\n\t" \
  "s_waitcnt lgkmcnt(8)\n\t" \
  "v_mfma_f32_16x16x128_f8f6f4 v[4:7], v[160:167], v[224:231], v[4:7] blgp:1\n\t" \
  "ds_read_b64_tr_b8 v[240:241], v251 offset:18432\n\t" \
  "ds_read_b64_tr_b8 v[242:243], v251 offset:19456\n\t" \
  "ds_read_b128 v[208:211], v254 offset:36864\n\t" \
  "v_mfma_f32_16x16x128_f8f6f4 v[36:39], v[168:175], v[224:231], v[36:39] blgp:1\n\t" \
  "ds_read_b64_tr_b8 v[244:245], v251 offset:26624\n\t" \
  "ds_read_b128 v[212:215], v254 offset:36928\n\t" \
  "v_mfma_f32_16x16x128_f8f6f4 v[68:71], v[176:183], v[224:231], v[68:71] blgp:1\n\t" \
  "ds_read_b64_tr_b8 v[246:247], v251 offset:27648\n\t" \
  "ds_read_b128 v[216:219], v254 offset:38912\n\t" \
  "" NIPAD "\n\t" \
  "v_mfma_f32_16x16x128_f8f6f4 v[100:103], v[184:191], v[224:231], v[100:103] blgp:1\n\t" \
  "ds_read_b128 v[220:223], v254 offset:38976\n\t" \
  "s_waitcnt lgkmcnt(10)\n\t" \
  "v_mfma_f32_16x16x128_f8f6f4 v[8:11], v[160:167], v[232:239], v[8:11] blgp:1\n\t" \
  "ds_read_b64_tr_b8 v[224:225], v248 offset:18688\n\t" \
  "ds_read_b64_tr_b8 v[226:227], v248 offset:19712\n\t" \
  "v_mfma_f32_16x16x128_f8f6f4 v[40:43], v[168:175], v[232:239], v[40:43] blgp:1\n\t" \
  "ds_read_b64_tr_b8 v[228:229], v248 offset:26880\n\t" \
  "v_mfma_f32_16x16x128_f8f6f4 v[72:75], v[176:183], v[232:239], v[72:75] blgp:1\n\t" \
  "ds_read_b64_tr_b8 v[230:231], v248 offset:27904\n\t" \
  "" NIPAD "\n\t" \
  "v_mfma_f32_16x16x128_f8f6f4 v[104:107], v[184:191], v[232:239], v[104:107] blgp:1\n\t" \
  "s_waitcnt lgkmcnt(6)\n\t" \
  "s_waitcnt lgkmcnt(4)\n\t" \
  "v_mfma_f32_16x16x128_f8f6f4 v[12:15], v[160:167], v[240:247], v[12:15] blgp:1\n\t" \
  "ds_read_b64_tr_b8 v[232:233], v249 offset:18688\n\t" \
  "ds_read_b64_tr_b8 v[234:235], v249 offset:19712\n\t" \
  "v_mfma_f32_16x16x128_f8f6f4 v[44:47], v[168:175], v[240:247], v[44:47] blgp:1\n\t" \
  "ds_read_b64_tr_b8 v[236:237], v249 offset:26880\n\t" \
  "s_add_u32 s68, s40, 384\n\t" \
  "s_addc_u32 s69, s41, 0\n\t" \
  "s_add_u32 m0, s57, 32768\n\t" \
  "s_nop 0\n\t" \
  "global_load_lds_dwordx4 v255, s[68:69] " XPOL "\n\t" \
  "v_mfma_f32_16x16x128_f8f6f4 v[76:79], v[176:183], v[240:247], v[76:79] blgp:1\n\t" \
  "ds_read_b64_tr_b8 v[238:239], v249 offset:27904\n\t" \
  "" NIPAD "\n\t" \
  "v_mfma_f32_16x16x128_f8f6f4 v[108:111], v[184:191], v[240:247], v[108:111] blgp:1\n\t" \
  "s_add_u32 s68, s40, 4480\n\t" \
  "s_addc_u32 s69, s41, 0\n\t" \
  "s_add_u32 m0, s57, 33792\n\t" \
  "s_nop 0\n\t" \
  "global_load_lds_dwordx4 v255, s[68:69] " XPOL "\n\t" \
  "s_add_u32 s58, s58, 1\n\t" \
  "s_mov_b32 m0, s50\n\t" \
  "v_mov_b32 v253, s58\n\t" \
  "ds_write_addtid_b32 v253 offset:256\n\t" \
  "s_waitcnt lgkmcnt(5)\n\t" \
  "v_mfma_f32_16x16x128_f8f6f4 v[16:19], v[160:167], v[224:231], v[16:19] blgp:1\n\t" \
  "ds_read_b64_tr_b8 v[240:241], v250 offset:18688\n\t" \
  "ds_read_b64_tr_b8 v[242:243], v250 offset:19712\n\t" \
  "v_mfma_f32_16x16x128_f8f6f4 v[48:51], v[168:175], v[224:231], v[48:51] blgp:1\n\t" \
  "ds_read_b64_tr_b8 v[244:245], v250 offset:26880\n\t" \
  "s_add_u32 s68, s40, 8576\n\t" \
  "s_addc_u32 s69, s41, 0\n\t" \
  "s_add_u32 m0, s57, 34816\n\t" \
  "s_nop 0\n\t" \
  "global_load_lds_dwordx4 v255, s[68:69] " XPOL "\n\t" \
  "v_mfma_f32_16x16x128_f8f6f4 v[80:83], v[176:183], v[224:231], v[80:83] blgp:1\n\t" \
  "ds_read_b64_tr_b8 v[246:247], v250 offset:27904\n\t" \
  "" NIPAD "\n\t" \
  "v_mfma_f32_16x16x128_f8f6f4 v[112:115], v[184:191], v[224:231], v[112:115] blgp:1\n\t" \
  "s_add_u32 s68, s40, 12672\n\t" \
  "s_addc_u32 s69, s41, 0\n\t" \
  "s_add_u32 m0, s57, 35840\n\t" \
  "s_nop 0\n\t" \
  "global_load_lds_dwordx4 v255, s[68:69] " XPOL "\n\t" \
  "s_mov_b32 m0, 0\n\t" \
  "s_add_u32 s51, s51, 2\n\t" \
  "ds_read_addtid_b32 v253 offset:0\n\t" \
  "s_add_u32 s52, s52, 1\n\t" \
  "s_waitcnt lgkmcnt(6)\n\t" \
  "v_mfma_f32_16x16x128_f8f6f4 v[20:23], v[160:167], v[232:239], v[20:23] blgp:1\n\t" \
  "ds_read_b64_tr_b8 v[224:225], v251 offset:18688\n\t" \
  "ds_read_b64_tr_b8 v[226:227], v251 offset:19712\n\t" \
  "v_mfma_f32_16x16x128_f8f6f4 v[52:55], v[168:175], v[232:239], v[52:55] blgp:1\n\t" \
  "ds_read_b64_tr_b8 v[228:229], v251 offset:26880\n\t" \
  "s_add_u32 s68, s40, 16768\n\t" \
  "s_addc_u32 s69, s41, 0\n\t" \
  "s_add_u32 m0, s57, 36864\n\t" \
  "s_nop 0\n\t" \
  "global_load_lds_dwordx4 v255, s[68:69] " XPOL "\n\t" \
  "v_mfma_f32_16x16x128_f8f6f4 v[84:87], v[176:183], v[232:239], v[84:87] blgp:1\n\t" \
  "ds_read_b64_tr_b8 v[230:231], v251 offset:27904\n\t" \
  "" NIPAD "\n\t" \
  "v_mfma_f32_16x16x128_f8f6f4 v[116:119], v[184:191], v[232:239], v[116:119] blgp:1\n\t" \
  "s_add_u32 s68, s40, 20864\n\t" \
  "s_addc_u32 s69, s41, 0\n\t" \
  "s_add_u32 m0, s57, 37888\n\t" \
  "s_nop 0\n\t" \
  "global_load_lds_dwordx4 v255, s[68:69] " XPOL "\n\t" \
  "s_waitcnt lgkmcnt(4)\n\t" \
  "Lc3_retry%=:\n\t" \
  "v_readlane_b32 s53, v253, 32\n\t" \
  "v_readlane_b32 s54, v253, s52\n\t" \
  "s_nop 0\n\t" \
  "s_cmp_lt_u32 s53, s51\n\t" \
  "s_cbranch_scc1 Lc3_spin%=\n\t" \
  "s_cmp_eq_u32 s54, 0\n\t" \
  "s_cbranch_scc1 Lc3_spin%=\n\t" \
  "Lc3_ok%=:\n\t" \
  "v_mfma_f32_16x16x128_f8f6f4 v[24:27], v[160:167], v[240:247], v[24:27] blgp:1\n\t" \
  "ds_read_b64_tr_b8 v[232:233], v248 offset:34816\n\t" \
  "ds_read_b64_tr_b8 v[234:235], v248 offset:35840\n\t" \
  "v_mfma_f32_16x16x128_f8f6f4 v[56:59], v[168:175], v[240:247], v[56:59] blgp:1\n\t" \
  "ds_read_b64_tr_b8 v[236:237], v248 offset:43008\n\t" \
  "s_add_u32 s68, s40, 24960\n\t" \
  "s_addc_u32 s69, s41, 0\n\t" \
  "s_add_u32 m0, s57, 38912\n\t" \
  "s_nop 0\n\t" \
  "global_load_lds_dwordx4 v255, s[68:69] " XPOL "\n\t" \
  "v_mfma_f32_16x16x128_f8f6f4 v[88:91], v[176:183], v[240:247], v[88:91] blgp:1\n\t" \
  "ds_read_b64_tr_b8 v[238:239], v248 offset:44032\n\t" \
  "" NIPAD "\n\t" \
  "v_mfma_f32_16x16x128_f8f6f4 v[120:123], v[184:191], v[240:247], v[120:123] blgp:1\n\t" \
  "s_add_u32 s68, s40, 29056\n\t" \
  "s_addc_u32 s69, s41, 0\n\t" \
  "s_add_u32 m0, s57, 39936\n\t" \
  "s_nop 0\n\t" \
  "global_load_lds_dwordx4 v255, s[68:69] " XPOL "\n\t" \
  "s_and_b32 s54, s54, 127\n\t" \
  "s_lshl_b32 s54, s54, 16\n\t" \
  "s_add_u32 s46, s42, s54\n\t" \
  "s_addc_u32 s47, s43, 0\n\t" \
  "s_waitcnt lgkmcnt(4)\n\t" \
  "v_mfma_f32_16x16x128_f8f6f4 v[28:31], v[160:167], v[224:231], v[28:31] blgp:1\n\t" \
  "ds_read_b64_tr_b8 v[240:241], v249 offset:34816\n\t" \
  "ds_read_b64_tr_b8 v[242:243], v249 offset:35840\n\t" \
  "v_mfma_f32_16x16x128_f8f6f4 v[60:63], v[168:175], v[224:231], v[60:63] blgp:1\n\t" \
  "ds_read_b64_tr_b8 v[244:245], v249 offset:43008\n\t" \
  "v_mfma_f32_16x16x128_f8f6f4 v[92:95], v[176:183], v[224:231], v[92:95] blgp:1\n\t" \
  "ds_read_b64_tr_b8 v[246:247], v249 offset:44032\n\t" \
  "" NIPAD "\n\t" \
  "v_mfma_f32_16x16x128_f8f6f4 v[124:127], v[184:191], v[224:231], v[124:127] blgp:1\n\t" \
  "" STAGEPAD "\n\t" \
  "s_waitcnt lgkmcnt(4)\n\t" \
  "v_mfma_f32_16x16x128_f8f6f4 v[0:3], v[192:199], v[232:239], v[0:3] blgp:1\n\t" \
  "ds_read_b64_tr_b8 v[224:225], v250 offset:34816\n\t" \
  "ds_read_b64_tr_b8 v[226:227], v250 offset:35840\n\t" \
  "v_mfma_f32_16x16x128_f8f6f4 v[32:35], v[200:207], v[232:239], v[32:35] blgp:1\n\t" \
  "ds_read_b64_tr_b8 v[228:229], v250 offset:43008\n\t" \
  "global_load_dwordx4 v[160:163], v252, s[46:47] offset:-4096 " APOL "\n\t" \
  "v_mfma_f32_16x16x128_f8f6f4 v[64:67], v[208:215], v[232:239], v[64:67] blgp:1\n\t" \
  "ds_read_b64_tr_b8 v[230:231], v250 offset:44032\n\t" \
  "" NIPAD "\n\t" \
  "v_mfma_f32_16x16x128_f8f6f4 v[96:99], v[216:223], v[232:239], v[96:99] blgp:1\n\t" \
  "global_load_dwordx4 v[164:167], v252, s[46:47] offset:-3072 " APOL "\n\t" \
  "s_waitcnt lgkmcnt(4)\n\t" \
  "v_mfma_f32_16x16x128_f8f6f4 v[4:7], v[192:199], v[240:247], v[4:7] blgp:1\n\t" \
  "ds_read_b64_tr_b8 v[232:233], v251 offset:34816\n\t" \
  "ds_read_b64_tr_b8 v[234:235], v251 offset:35840\n\t" \
  "v_mfma_f32_16x16x128_f8f6f4 v[36:39], v[200:207], v[240:247], v[36:39] blgp:1\n\t" \
  "ds_read_b64_tr_b8 v[236:237], v251 offset:43008\n\t" \
  "global_load_dwordx4 v[168:171], v252, s[46:47] offset:-2048 " APOL "\n\t" \
  "v_mfma_f32_16x16x128_f8f6f4 v[68:71], v[208:215], v[240:247], v[68:71] blgp:1\n\t" \
  "ds_read_b64_tr_b8 v[238:239], v251 offset:44032\n\t" \
  "" NIPAD "\n\t" \
  "v_mfma_f32_16x16x128_f8f6f4 v[100:103], v[216:223], v[240:247], v[100:103] blgp:1\n\t" \
  "global_load_dwordx4 v[172:175], v252, s[46:47] offset:-1024 " APOL "\n\t" \
  "s_waitcnt lgkmcnt(4)\n\t" \
  "v_mfma_f32_16x16x128_f8f6f4 v[8:11], v[192:199], v[224:231], v[8:11] blgp:1\n\t" \
  "ds_read_b64_tr_b8 v[240:241], v248 offset:35072\n\t" \
  "ds_read_b64_tr_b8 v[242:243], v248 offset:36096\n\t" \
  "v_mfma_f32_16x16x128_f8f6f4 v[40:43], v[200:207], v[224:231], v[40:43] blgp:1\n\t" \
  "ds_read_b64_tr_b8 v[244:245], v248 offset:43264\n\t" \
  "global_load_dwordx4 v[176:179], v252, s[46:47] offset:0 " APOL "\n\t" \
  "v_mfma_f32_16x16x128_f8f6f4 v[72:75], v[208:215], v[224:231], v[72:75] blgp:1\n\t" \
  "ds_read_b64_tr_b8 v[246:247], v248 offset:44288\n\t" \
  "" NIPAD "\n\t" \
  "v_mfma_f32_16x16x128_f8f6f4 v[104:107], v[216:223], v[224:231], v[104:107] blgp:1\n\t" \
  "global_load_dwordx4 v[180:183], v252, s[46:47] offset:1024 " APOL "\n\t" \
  "s_waitcnt lgkmcnt(4)\n\t" \
  "v_mfma_f32_16x16x128_f8f6f4 v[12:15], v[192:199], v[232:239], v[12:15] blgp:1\n\t" \
  "ds_read_b64_tr_b8 v[224:225], v249 offset:35072\n\t" \
  "ds_read_b64_tr_b8 v[226:227], v249 offset:36096\n\t" \
  "v_mfma_f32_16x16x128_f8f6f4 v[44:47], v[200:207], v[232:239], v[44:47] blgp:1\n\t" \
  "ds_read_b64_tr_b8 v[228:229], v249 offset:43264\n\t" \
  "global_load_dwordx4 v[184:187], v252, s[46:47] offset:2048 " APOL "\n\t" \
  "v_mfma_f32_16x16x128_f8f6f4 v[76:79], v[208:215], v[232:239], v[76:79] blgp:1\n\t" \
  "ds_read_b64_tr_b8 v[230:231], v249 offset:44288\n\t" \
  "" NIPAD "\n\t" \
  "v_mfma_f32_16x16x128_f8f6f4 v[108:111], v[216:223], v[232:239], v[108:111] blgp:1\n\t" \
  "global_load_dwordx4 v[188:191], v252, s[46:47] offset:3072 " APOL "\n\t" \
  "s_add_u32 s58, s58, 1\n\t" \
  "s_mov_b32 m0, s50\n\t" \
  "v_mov_b32 v253, s58\n\t" \
  "ds_write_addtid_b32 v253 offset:256\n\t" \
  "s_waitcnt lgkmcnt(5)\n\t" \
  "s_waitcnt vmcnt(8)\n\t" \
  "v_mfma_f32_16x16x128_f8f6f4 v[16:19], v[192:199], v[240:247], v[16:19] blgp:1\n\t" \
  "ds_read_b64_tr_b8 v[232:233], v250 offset:35072\n\t" \
  "ds_read_b64_tr_b8 v[234:235], v250 offset:36096\n\t" \
  "v_mfma_f32_16x16x128_f8f6f4 v[48:51], v[200:207], v[240:247], v[48:51] blgp:1\n\t" \
  "ds_read_b64_tr_b8 v[236:237], v250 offset:43264\n\t" \
  "ds_read_b128 v[128:131], v254 offset:32768\n\t" \
  "v_mfma_f32_16x16x128_f8f6f4 v[80:83], v[208:215], v[240:247], v[80:83] blgp:1\n\t" \
  "ds_read_b64_tr_b8 v[238:239], v250 offset:44288\n\t" \
  "v_add_u32 v248, s59, v248\n\t" \
  "" NIPAD "\n\t" \
  "v_mfma_f32_16x16x128_f8f6f4 v[112:115], v[216:223], v[240:247], v[112:115] blgp:1\n\t" \
  "ds_read_b128 v[132:135], v254 offset:32832\n\t" \
  "s_mov_b32 m0, 0\n\t" \
  "s_add_u32 s51, s51, 2\n\t" \
  "ds_read_addtid_b32 v253 offset:0\n\t" \
  "s_add_u32 s52, s52, 1\n\t" \
  "s_waitcnt lgkmcnt(8)\n\t" \
  "v_mfma_f32_16x16x128_f8f6f4 v[20:23], v[192:199], v[224:231], v[20:23] blgp:1\n\t" \
  "ds_read_b64_tr_b8 v[240:241], v251 offset:35072\n\t" \
  "ds_read_b64_tr_b8 v[242:243], v251 offset:36096\n\t" \
  "v_mfma_f32_16x16x128_f8f6f4 v[52:55], v[200:207], v[224:231], v[52:55] blgp:1\n\t" \
  "ds_read_b64_tr_b8 v[244:245], v251 offset:43264\n\t" \
  "ds_read_b128 v[136:139], v254 offset:34816\n\t" \
  "v_add_u32 v249, s59, v249\n\t" \
  "v_mfma_f32_16x16x128_f8f6f4 v[84:87], v[208:215], v[224:231], v[84:87] blgp:1\n\t" \
  "ds_read_b64_tr_b8 v[246:247], v251 offset:44288\n\t" \
  "" NIPAD "\n\t" \
  "v_mfma_f32_16x16x128_f8f6f4 v[116:119], v[216:223], v[224:231], v[116:119] blgp:1\n\t" \
  "ds_read_b128 v[140:143], v254 offset:34880\n\t" \
  "s_waitcnt lgkmcnt(6)\n\t" \
  "Lc4_retry%=:\n\t" \
  "v_readlane_b32 s53, v253, 32\n\t" \
  "v_readlane_b32 s54, v253, s52\n\t" \
  "s_nop 0\n\t" \
  "s_cmp_lt_u32 s53, s51\n\t" \
  "s_cbranch_scc1 Lc4_spin%=\n\t" \
  "s_cmp_eq_u32 s54, 0\n\t" \
  "s_cbranch_scc1 Lc4_spin%=\n\t" \
  "Lc4_ok%=:\n\t" \
  "v_mfma_f32_16x16x128_f8f6f4 v[24:27], v[192:199], v[232:239], v[24:27] blgp:1\n\t" \
  "ds_read_b64_tr_b8 v[224:225], v248 offset:2048\n\t" \
  "ds_read_b64_tr_b8 v[226:227], v248 offset:3072\n\t" \
  "v_mfma_f32_16x16x128_f8f6f4 v[56:59], v[200:207], v[232:239], v[56:59] blgp:1\n\t" \
  "ds_read_b64_tr_b8 v[228:229], v248 offset:10240\n\t" \
  "ds_read_b128 v[144:147], v254 offset:36864\n\t" \
  "v_mfma_f32_16x16x128_f8f6f4 v[88:91], v[208:215], v[232:239], v[88:91] blgp:1\n\t" \
  "ds_read_b64_tr_b8 v[230:231], v248 offset:11264\n\t" \
  "v_add_u32 v250, s59, v250\n\t" \
  "" NIPAD "\n\t" \
  "v_mfma_f32_16x16x128_f8f6f4 v[120:123], v[216:223], v[232:239], v[120:123] blgp:1\n\t" \
  "ds_read_b128 v[148:151], v254 offset:36928\n\t" \
  "s_and_b32 s54, s54, 127\n\t" \
  "s_lshl_b32 s54, s54, 16\n\t" \
  "s_add_u32 s46, s42, s54\n\t" \
  "s_addc_u32 s47, s43, 0\n\t" \
  "s_waitcnt lgkmcnt(7)\n\t" \
  "v_mfma_f32_16x16x128_f8f6f4 v[28:31], v[192:199], v[240:247], v[28:31] blgp:1\n\t" \
  "ds_read_b64_tr_b8 v[232:233], v249 offset:2048\n\t" \
  "ds_read_b64_tr_b8 v[234:235], v249 offset:3072\n\t" \
  "v_mfma_f32_16x16x128_f8f6f4 v[60:63], v[200:207], v[240:247], v[60:63] blgp:1\n\t" \
  "ds_read_b64_tr_b8 v[236:237], v249 offset:10240\n\t" \
  "ds_read_b128 v[152:155], v254 offset:38912\n\t" \
  "v_add_u32 v251, s59, v251\n\t" \
  "v_mfma_f32_16x16x128_f8f6f4 v[92:95], v[208:215], v[240:247], v[92:95] blgp:1\n\t" \
  "ds_read_b64_tr_b8 v[238:239], v249 offset:11264\n\t" \
  "s_sub_u32 s59, 0, s59\n\t" \
  "" NIPAD "\n\t" \
  "v_mfma_f32_16x16x128_f8f6f4 v[124:127], v[216:223], v[240:247], v[124:127] blgp:1\n\t" \
  "ds_read_b128 v[156:159], v254 offset:38976\n\t" \
  "s_mov_b32 s55, 8\n\t" \
  "s_waitcnt lgkmcnt(0)\n\t" \
  "Lloop%=:\n\t" \
  "s_waitcnt vmcnt(8)\n\t" \
  "" STAGEPAD "\n\t" \
  "s_waitcnt lgkmcnt(4)\n\t" \
  "v_mfma_f32_16x16x128_f8f6f4 v[0:3], v[128:135], v[224:231], v[0:3] blgp:1\n\t" \
  "ds_read_b64_tr_b8 v[240:241], v250 offset:2048\n\t" \
  "ds_read_b64_tr_b8 v[242:243], v250 offset:3072\n\t" \
  "v_mfma_f32_16x16x128_f8f6f4 v[32:35], v[136:143], v[224:231], v[32:35] blgp:1\n\t" \
  "ds_read_b64_tr_b8 v[244:245], v250 offset:10240\n\t" \
  "global_load_dwordx4 v[192:195], v252, s[46:47] offset:-4096 " APOL "\n\t" \
  "v_mfma_f32_16x16x128_f8f6f4 v[64:67], v[144:151], v[224:231], v[64:67] blgp:1\n\t" \
  "ds_read_b64_tr_b8 v[246:247], v250 offset:11264\n\t" \
  "" NIPAD "\n\t" \
  "v_mfma_f32_16x16x128_f8f6f4 v[96:99], v[152:159], v[224:231], v[96:99] blgp:1\n\t" \
  "global_load_dwordx4 v[196:199], v252, s[46:47] offset:-3072 " APOL "\n\t" \
  "s_waitcnt lgkmcnt(4)\n\t" \
  "v_mfma_f32_16x16x128_f8f6f4 v[4:7], v[128:135], v[232:239], v[4:7] blgp:1\n\t" \
  "ds_read_b64_tr_b8 v[224:225], v251 offset:2048\n\t" \
  "ds_read_b64_tr_b8 v[226:227], v251 offset:3072\n\t" \
  "v_mfma_f32_16x16x128_f8f6f4 v[36:39], v[136:143], v[232:239], v[36:39] blgp:1\n\t" \
  "ds_read_b64_tr_b8 v[228:229], v251 offset:10240\n\t" \
  "global_load_dwordx4 v[200:203], v252, s[46:47] offset:-2048 " APOL "\n\t" \
  "v_mfma_f32_16x16x128_f8f6f4 v[68:71], v[144:151], v[232:239], v[68:71] blgp:1\n\t" \
  "ds_read_b64_tr_b8 v[230:231], v251 offset:11264\n\t" \
  "" NIPAD "\n\t" \
  "v_mfma_f32_16x16x128_f8f6f4 v[100:103], v[152:159], v[232:239], v[100:103] blgp:1\n\t" \
  "global_load_dwordx4 v[204:207], v252, s[46:47] offset:-1024 " APOL "\n\t" \
  "s_waitcnt lgkmcnt(4)\n\t" \
  "v_mfma_f32_16x16x128_f8f6f4 v[8:11], v[128:135], v[240:247], v[8:11] blgp:1\n\t" \
  "ds_read_b64_tr_b8 v[232:233], v248 offset:2304\n\t" \
  "ds_read_b64_tr_b8 v[234:235], v248 offset:3328\n\t" \
  "v_mfma_f32_16x16x128_f8f6f4 v[40:43], v[136:143], v[240:247], v[40:43] blgp:1\n\t" \
  "ds_read_b64_tr_b8 v[236:237], v248 offset:10496\n\t" \
  "global_load_dwordx4 v[208:211], v252, s[46:47] offset:0 " APOL "\n\t" \
  "v_mfma_f32_16x16x128_f8f6f4 v[72:75], v[144:151], v[240:247], v[72:75] blgp:1\n\t" \
  "ds_read_b64_tr_b8 v[238:239], v248 offset:11520\n\t" \
  "" NIPAD "\n\t" \
  "v_mfma_f32_16x16x128_f8f6f4 v[104:107], v[152:159], v[240:247], v[104:107] blgp:1\n\t" \
  "global_load_dwordx4 v[212:215], v252, s[46:47] offset:1024 " APOL "\n\t" \
  "s_waitcnt lgkmcnt(4)\n\t" \
  "v_mfma_f32_16x16x128_f8f6f4 v[12:15], v[128:135], v[224:231], v[12:15] blgp:1\n\t" \
  "ds_read_b64_tr_b8 v[240:241], v249 offset:2304\n\t" \
  "ds_read_b64_tr_b8 v[242:243], v249 offset:3328\n\t" \
  "v_mfma_f32_16x16x128_f8f6f4 v[44:47], v[136:143], v[224:231], v[44:47] blgp:1\n\t" \
  "ds_read_b64_tr_b8 v[244:245], v249 offset:10496\n\t" \
  "global_load_dwordx4 v[216:219], v252, s[46:47] offset:2048 " APOL "\n\t" \
  "v_mfma_f32_16x16x128_f8f6f4 v[76:79], v[144:151], v[224:231], v[76:79] blgp:1\n\t" \
  "ds_read_b64_tr_b8 v[246:247], v249 offset:11520\n\t" \
  "" NIPAD "\n\t" \
  "v_mfma_f32_16x16x128_f8f6f4 v[108:111], v[152:159], v[224:231], v[108:111] blgp:1\n\t" \
  "global_load_dwordx4 v[220:223], v252, s[46:47] offset:3072 " APOL "\n\t" \
  "s_add_u32 s58, s58, 1\n\t" \
  "s_mov_b32 m0, s50\n\t" \
  "v_mov_b32 v253, s58\n\t" \
  "ds_write_addtid_b32 v253 offset:256\n\t" \
  "s_waitcnt lgkmcnt(5)\n\t" \
  "v_mfma_f32_16x16x128_f8f6f4 v[16:19], v[128:135], v[232:239], v[16:19] blgp:1\n\t" \
  "ds_read_b64_tr_b8 v[224:225], v250 offset:2304\n\t" \
  "ds_read_b64_tr_b8 v[226:227], v250 offset:3328\n\t" \
  "v_mfma_f32_16x16x128_f8f6f4 v[48:51], v[136:143], v[232:239], v[48:51] blgp:1\n\t" \
  "ds_read_b64_tr_b8 v[228:229], v250 offset:10496\n\t" \
  "v_mfma_f32_16x16x128_f8f6f4 v[80:83], v[144:151], v[232:239], v[80:83] blgp:1\n\t" \
  "ds_read_b64_tr_b8 v[230:231], v250 offset:11520\n\t" \
  "" NIPAD "\n\t" \
  "v_mfma_f32_16x16x128_f8f6f4 v[112:115], v[152:159], v[232:239], v[112:115] blgp:1\n\t" \
  "s_mov_b32 m0, 0\n\t" \
  "s_add_u32 s51, s51, 2\n\t" \
  "ds_read_addtid_b32 v253 offset:0\n\t" \
  "s_add_u32 s52, s52, 1\n\t" \
  "s_waitcnt lgkmcnt(6)\n\t" \
  "v_mfma_f32_16x16x128_f8f6f4 v[20:23], v[128:135], v[240:247], v[20:23] blgp:1\n\t" \
  "ds_read_b64_tr_b8 v[232:233], v251 offset:2304\n\t" \
  "ds_read_b64_tr_b8 v[234:235], v251 offset:3328\n\t" \
  "v_mfma_f32_16x16x128_f8f6f4 v[52:55], v[136:143], v[240:247], v[52:55] blgp:1\n\t" \
  "ds_read_b64_tr_b8 v[236:237], v251 offset:10496\n\t" \
  "v_mfma_f32_16x16x128_f8f6f4 v[84:87], v[144:151], v[240:247], v[84:87] blgp:1\n\t" \
  "ds_read_b64_tr_b8 v[238:239], v251 offset:11520\n\t" \
  "" NIPAD "\n\t" \
  "v_mfma_f32_16x16x128_f8f6f4 v[116:119], v[152:159], v[240:247], v[116:119] blgp:1\n\t" \
  "s_waitcnt lgkmcnt(4)\n\t" \
  "Lc5_retry%=:\n\t" \
  "v_readlane_b32 s53, v253, 32\n\t" \
  "v_readlane_b32 s54, v253, s52\n\t" \
  "s_nop 0\n\t" \
  "s_cmp_lt_u32 s53, s51\n\t" \
  "s_cbranch_scc1 Lc5_spin%=\n\t" \
  "s_cmp_eq_u32 s54, 0\n\t" \
  "s_cbranch_scc1 Lc5_spin%=\n\t" \
  "Lc5_ok%=:\n\t" \
  "v_mfma_f32_16x16x128_f8f6f4 v[24:27], v[128:135], v[224:231], v[24:27] blgp:1\n\t" \
  "ds_read_b64_tr_b8 v[240:241], v248 offset:18432\n\t" \
  "ds_read_b64_tr_b8 v[242:243], v248 offset:19456\n\t" \
  "v_mfma_f32_16x16x128_f8f6f4 v[56:59], v[136:143], v[224:231], v[56:59] blgp:1\n\t" \
  "ds_read_b64_tr_b8 v[244:245], v248 offset:26624\n\t" \
  "v_mfma_f32_16x16x128_f8f6f4 v[88:91], v[144:151], v[224:231], v[88:91] blgp:1\n\t" \
  "ds_read_b64_tr_b8 v[246:247], v248 offset:27648\n\t" \
  "" NIPAD "\n\t" \
  "v_mfma_f32_16x16x128_f8f6f4 v[120:123], v[152:159], v[224:231], v[120:123] blgp:1\n\t" \
  "s_and_b32 s54, s54, 127\n\t" \
  "s_lshl_b32 s54, s54, 16\n\t" \
  "s_add_u32 s46, s42, s54\n\t" \
  "s_addc_u32 s47, s43, 0\n\t" \
  "s_waitcnt lgkmcnt(4)\n\t" \
  "v_mfma_f32_16x16x128_f8f6f4 v[28:31], v[128:135], v[232:239], v[28:31] blgp:1\n\t" \
  "ds_read_b64_tr_b8 v[224:225], v249 offset:18432\n\t" \
  "ds_read_b64_tr_b8 v[226:227], v249 offset:19456\n\t" \
  "v_mfma_f32_16x16x128_f8f6f4 v[60:63], v[136:143], v[232:239], v[60:63] blgp:1\n\t" \
  "ds_read_b64_tr_b8 v[228:229], v249 offset:26624\n\t" \
  "v_mfma_f32_16x16x128_f8f6f4 v[92:95], v[144:151], v[232:239], v[92:95] blgp:1\n\t" \
  "ds_read_b64_tr_b8 v[230:231], v249 offset:27648\n\t" \
  "" NIPAD "\n\t" \
  "v_mfma_f32_16x16x128_f8f6f4 v[124:127], v[152:159], v[232:239], v[124:127] blgp:1\n\t" \
  "s_waitcnt vmcnt(8)\n\t" \
  "" STAGEPAD "\n\t" \
  "s_waitcnt lgkmcnt(4)\n\t" \
  "v_mfma_f32_16x16x128_f8f6f4 v[0:3], v[160:167], v[240:247], v[0:3] blgp:1\n\t" \
  "ds_read_b64_tr_b8 v[232:233], v250 offset:18432\n\t" \
  "ds_read_b64_tr_b8 v[234:235], v250 offset:19456\n\t" \
  "v_mfma_f32_16x16x128_f8f6f4 v[32:35], v[168:175], v[240:247], v[32:35] blgp:1\n\t" \
  "ds_read_b64_tr_b8 v[236:237], v250 offset:26624\n\t" \
  "global_load_dwordx4 v[128:131], v252, s[46:47] offset:-4096 " APOL "\n\t" \
  "v_mfma_f32_16x16x128_f8f6f4 v[64:67], v[176:183], v[240:247], v[64:67] blgp:1\n\t" \
  "ds_read_b64_tr_b8 v[238:239], v250 offset:27648\n\t" \
  "" NIPAD "\n\t" \
  "v_mfma_f32_16x16x128_f8f6f4 v[96:99], v[184:191], v[240:247], v[96:99] blgp:1\n\t" \
  "global_load_dwordx4 v[132:135], v252, s[46:47] offset:-3072 " APOL "\n\t" \
  "s_waitcnt lgkmcnt(4)\n\t" \
  "v_mfma_f32_16x16x128_f8f6f4 v[4:7], v[160:167], v[224:231], v[4:7] blgp:1\n\t" \
  "ds_read_b64_tr_b8 v[240:241], v251 offset:18432\n\t" \
  "ds_read_b64_tr_b8 v[242:243], v251 offset:19456\n\t" \
  "v_mfma_f32_16x16x128_f8f6f4 v[36:39], v[168:175], v[224:231], v[36:39] blgp:1\n\t" \
  "ds_read_b64_tr_b8 v[244:245], v251 offset:26624\n\t" \
  "global_load_dwordx4 v[136:139], v252, s[46:47] offset:-2048 " APOL "\n\t" \
  "v_mfma_f32_16x16x128_f8f6f4 v[68:71], v[176:183], v[224:231], v[68:71] blgp:1\n\t" \
  "ds_read_b64_tr_b8 v[246:247], v251 offset:27648\n\t" \
  "" NIPAD "\n\t" \
  "v_mfma_f32_16x16x128_f8f6f4 v[100:103], v[184:191], v[224:231], v[100:103] blgp:1\n\t" \
  "global_load_dwordx4 v[140:143], v252, s[46:47] offset:-1024 " APOL "\n\t" \
  "s_waitcnt lgkmcnt(4)\n\t" \
  "v_mfma_f32_16x16x128_f8f6f4 v[8:11], v[160:167], v[232:239], v[8:11] blgp:1\n\t" \
  "ds_read_b64_tr_b8 v[224:225], v248 offset:18688\n\t" \
  "ds_read_b64_tr_b8 v[226:227], v248 offset:19712\n\t" \
  "v_mfma_f32_16x16x128_f8f6f4 v[40:43], v[168:175], v[232:239], v[40:43] blgp:1\n\t" \
  "ds_read_b64_tr_b8 v[228:229], v248 offset:26880\n\t" \
  "global_load_dwordx4 v[144:147], v252, s[46:47] offset:0 " APOL "\n\t" \
  "v_mfma_f32_16x16x128_f8f6f4 v[72:75], v[176:183], v[232:239], v[72:75] blgp:1\n\t" \
  "ds_read_b64_tr_b8 v[230:231], v248 offset:27904\n\t" \
  "" NIPAD "\n\t" \
  "v_mfma_f32_16x16x128_f8f6f4 v[104:107], v[184:191], v[232:239], v[104:107] blgp:1\n\t" \
  "global_load_dwordx4 v[148:151], v252, s[46:47] offset:1024 " APOL "\n\t" \
  "s_waitcnt lgkmcnt(4)\n\t" \
  "v_mfma_f32_16x16x128_f8f6f4 v[12:15], v[160:167], v[240:247], v[12:15] blgp:1\n\t" \
  "ds_read_b64_tr_b8 v[232:233], v249 offset:18688\n\t" \
  "ds_read_b64_tr_b8 v[234:235], v249 offset:19712\n\t" \
  "v_mfma_f32_16x16x128_f8f6f4 v[44:47], v[168:175], v[240:247], v[44:47] blgp:1\n\t" \
  "ds_read_b64_tr_b8 v[236:237], v249 offset:26880\n\t" \
  "global_load_dwordx4 v[152:155], v252, s[46:47] offset:2048 " APOL "\n\t" \
  "v_mfma_f32_16x16x128_f8f6f4 v[76:79], v[176:183], v[240:247], v[76:79] blgp:1\n\t" \
  "ds_read_b64_tr_b8 v[238:239], v249 offset:27904\n\t" \
  "" NIPAD "\n\t" \
  "v_mfma_f32_16x16x128_f8f6f4 v[108:111], v[184:191], v[240:247], v[108:111] blgp:1\n\t" \
  "global_load_dwordx4 v[156:159], v252, s[46:47] offset:3072 " APOL "\n\t" \
  "s_add_u32 s58, s58, 1\n\t" \
  "s_mov_b32 m0, s50\n\t" \
  "v_mov_b32 v253, s58\n\t" \
  "ds_write_addtid_b32 v253 offset:256\n\t" \
  "s_waitcnt lgkmcnt(5)\n\t" \
  "v_mfma_f32_16x16x128_f8f6f4 v[16:19], v[160:167], v[224:231], v[16:19] blgp:1\n\t" \
  "ds_read_b64_tr_b8 v[240:241], v250 offset:18688\n\t" \
  "ds_read_b64_tr_b8 v[242:243], v250 offset:19712\n\t" \
  "v_mfma_f32_16x16x128_f8f6f4 v[48:51], v[168:175], v[224:231], v[48:51] blgp:1\n\t" \
  "ds_read_b64_tr_b8 v[244:245], v250 offset:26880\n\t" \
  "v_mfma_f32_16x16x128_f8f6f4 v[80:83], v[176:183], v[224:231], v[80:83] blgp:1\n\t" \
  "ds_read_b64_tr_b8 v[246:247], v250 offset:27904\n\t" \
  "" NIPAD "\n\t" \
  "v_mfma_f32_16x16x128_f8f6f4 v[112:115], v[184:191], v[224:231], v[112:115] blgp:1\n\t" \
  "s_mov_b32 m0, 0\n\t" \
  "s_add_u32 s51, s51, 2\n\t" \
  "ds_read_addtid_b32 v253 offset:0\n\t" \
  "s_add_u32 s52, s52, 1\n\t" \
  "s_waitcnt lgkmcnt(6)\n\t" \
  "v_mfma_f32_16x16x128_f8f6f4 v[20:23], v[160:167], v[232:239], v[20:23] blgp:1\n\t" \
  "ds_read_b64_tr_b8 v[224:225], v251 offset:18688\n\t" \
  "ds_read_b64_tr_b8 v[226:227], v251 offset:19712\n\t" \
  "v_mfma_f32_16x16x128_f8f6f4 v[52:55], v[168:175], v[232:239], v[52:55] blgp:1\n\t" \
  "ds_read_b64_tr_b8 v[228:229], v251 offset:26880\n\t" \
  "v_mfma_f32_16x16x128_f8f6f4 v[84:87], v[176:183], v[232:239], v[84:87] blgp:1\n\t" \
  "ds_read_b64_tr_b8 v[230:231], v251 offset:27904\n\t" \
  "" NIPAD "\n\t" \
  "v_mfma_f32_16x16x128_f8f6f4 v[116:119], v[184:191], v[232:239], v[116:119] blgp:1\n\t" \
  "s_waitcnt lgkmcnt(4)\n\t" \
  "Lc6_retry%=:\n\t" \
  "v_readlane_b32 s53, v253, 32\n\t" \
  "v_readlane_b32 s54, v253, s52\n\t" \
  "s_nop 0\n\t" \
  "s_cmp_lt_u32 s53, s51\n\t" \
  "s_cbranch_scc1 Lc6_spin%=\n\t" \
  "s_cmp_eq_u32 s54, 0\n\t" \
  "s_cbranch_scc1 Lc6_spin%=\n\t" \
  "Lc6_ok%=:\n\t" \
  "v_mfma_f32_16x16x128_f8f6f4 v[24:27], v[160:167], v[240:247], v[24:27] blgp:1\n\t" \
  "ds_read_b64_tr_b8 v[232:233], v248 offset:34816\n\t" \
  "ds_read_b64_tr_b8 v[234:235], v248 offset:35840\n\t" \
  "v_mfma_f32_16x16x128_f8f6f4 v[56:59], v[168:175], v[240:247], v[56:59] blgp:1\n\t" \
  "ds_read_b64_tr_b8 v[236:237], v248 offset:43008\n\t" \
  "v_mfma_f32_16x16x128_f8f6f4 v[88:91], v[176:183], v[240:247], v[88:91] blgp:1\n\t" \
  "ds_read_b64_tr_b8 v[238:239], v248 offset:44032\n\t" \
  "" NIPAD "\n\t" \
  "v_mfma_f32_16x16x128_f8f6f4 v[120:123], v[184:191], v[240:247], v[120:123] blgp:1\n\t" \
  "s_and_b32 s54, s54, 127\n\t" \
  "s_lshl_b32 s54, s54, 16\n\t" \
  "s_add_u32 s46, s42, s54\n\t" \
  "s_addc_u32 s47, s43, 0\n\t" \
  "s_waitcnt lgkmcnt(4)\n\t" \
  "v_mfma_f32_16x16x128_f8f6f4 v[28:31], v[160:167], v[224:231], v[28:31] blgp:1\n\t" \
  "ds_read_b64_tr_b8 v[240:241], v249 offset:34816\n\t" \
  "ds_read_b64_tr_b8 v[242:243], v249 offset:35840\n\t" \
  "v_mfma_f32_16x16x128_f8f6f4 v[60:63], v[168:175], v[224:231], v[60:63] blgp:1\n\t" \
  "ds_read_b64_tr_b8 v[244:245], v249 offset:43008\n\t" \
  "v_mfma_f32_16x16x128_f8f6f4 v[92:95], v[176:183], v[224:231], v[92:95] blgp:1\n\t" \
  "ds_read_b64_tr_b8 v[246:247], v249 offset:44032\n\t" \
  "" NIPAD "\n\t" \
  "v_mfma_f32_16x16x128_f8f6f4 v[124:127], v[184:191], v[224:231], v[124:127] blgp:1\n\t" \
  "s_waitcnt vmcnt(8)\n\t" \
  "" STAGEPAD "\n\t" \
  "s_waitcnt lgkmcnt(4)\n\t" \
  "v_mfma_f32_16x16x128_f8f6f4 v[0:3], v[192:199], v[232:239], v[0:3] blgp:1\n\t" \
  "ds_read_b64_tr_b8 v[224:225], v250 offset:34816\n\t" \
  "ds_read_b64_tr_b8 v[226:227], v250 offset:35840\n\t" \
  "v_mfma_f32_16x16x128_f8f6f4 v[32:35], v[200:207], v[232:239], v[32:35] blgp:1\n\t" \
  "ds_read_b64_tr_b8 v[228:229], v250 offset:43008\n\t" \
  "global_load_dwordx4 v[160:163], v252, s[46:47] offset:-4096 " APOL "\n\t" \
  "v_mfma_f32_16x16x128_f8f6f4 v[64:67], v[208:215], v[232:239], v[64:67] blgp:1\n\t" \
  "ds_read_b64_tr_b8 v[230:231], v250 offset:44032\n\t" \
  "" NIPAD "\n\t" \
  "v_mfma_f32_16x16x128_f8f6f4 v[96:99], v[216:223], v[232:239], v[96:99] blgp:1\n\t" \
  "global_load_dwordx4 v[164:167], v252, s[46:47] offset:-3072 " APOL "\n\t" \
  "s_waitcnt lgkmcnt(4)\n\t" \
  "v_mfma_f32_16x16x128_f8f6f4 v[4:7], v[192:199], v[240:247], v[4:7] blgp:1\n\t" \
  "ds_read_b64_tr_b8 v[232:233], v251 offset:34816\n\t" \
  "ds_read_b64_tr_b8 v[234:235], v251 offset:35840\n\t" \
  "v_mfma_f32_16x16x128_f8f6f4 v[36:39], v[200:207], v[240:247], v[36:39] blgp:1\n\t" \
  "ds_read_b64_tr_b8 v[236:237], v251 offset:43008\n\t" \
  "global_load_dwordx4 v[168:171], v252, s[46:47] offset:-2048 " APOL "\n\t" \
  "v_mfma_f32_16x16x128_f8f6f4 v[68:71], v[208:215], v[240:247], v[68:71] blgp:1\n\t" \
  "ds_read_b64_tr_b8 v[238:239], v251 offset:44032\n\t" \
  "" NIPAD "\n\t" \
  "v_mfma_f32_16x16x128_f8f6f4 v[100:103], v[216:223], v[240:247], v[100:103] blgp:1\n\t" \
  "global_load_dwordx4 v[172:175], v252, s[46:47] offset:-1024 " APOL "\n\t" \
  "s_waitcnt lgkmcnt(4)\n\t" \
  "v_mfma_f32_16x16x128_f8f6f4 v[8:11], v[192:199], v[224:231], v[8:11] blgp:1\n\t" \
  "ds_read_b64_tr_b8 v[240:241], v248 offset:35072\n\t" \
  "ds_read_b64_tr_b8 v[242:243], v248 offset:36096\n\t" \
  "v_mfma_f32_16x16x128_f8f6f4 v[40:43], v[200:207], v[224:231], v[40:43] blgp:1\n\t" \
  "ds_read_b64_tr_b8 v[244:245], v248 offset:43264\n\t" \
  "global_load_dwordx4 v[176:179], v252, s[46:47] offset:0 " APOL "\n\t" \
  "v_mfma_f32_16x16x128_f8f6f4 v[72:75], v[208:215], v[224:231], v[72:75] blgp:1\n\t" \
  "ds_read_b64_tr_b8 v[246:247], v248 offset:44288\n\t" \
  "" NIPAD "\n\t" \
  "v_mfma_f32_16x16x128_f8f6f4 v[104:107], v[216:223], v[224:231], v[104:107] blgp:1\n\t" \
  "global_load_dwordx4 v[180:183], v252, s[46:47] offset:1024 " APOL "\n\t" \
  "s_waitcnt lgkmcnt(4)\n\t" \
  "v_mfma_f32_16x16x128_f8f6f4 v[12:15], v[192:199], v[232:239], v[12:15] blgp:1\n\t" \
  "ds_read_b64_tr_b8 v[224:225], v249 offset:35072\n\t" \
  "ds_read_b64_tr_b8 v[226:227], v249 offset:36096\n\t" \
  "v_mfma_f32_16x16x128_f8f6f4 v[44:47], v[200:207], v[232:239], v[44:47] blgp:1\n\t" \
  "ds_read_b64_tr_b8 v[228:229], v249 offset:43264\n\t" \
  "global_load_dwordx4 v[184:187], v252, s[46:47] offset:2048 " APOL "\n\t" \
  "v_mfma_f32_16x16x128_f8f6f4 v[76:79], v[208:215], v[232:239], v[76:79] blgp:1\n\t" \
  "ds_read_b64_tr_b8 v[230:231], v249 offset:44288\n\t" \
  "" NIPAD "\n\t" \
  "v_mfma_f32_16x16x128_f8f6f4 v[108:111], v[216:223], v[232:239], v[108:111] blgp:1\n\t" \
  "global_load_dwordx4 v[188:191], v252, s[46:47] offset:3072 " APOL "\n\t" \
  "s_add_u32 s58, s58, 1\n\t" \
  "s_mov_b32 m0, s50\n\t" \
  "v_mov_b32 v253, s58\n\t" \
  "ds_write_addtid_b32 v253 offset:256\n\t" \
  "s_waitcnt lgkmcnt(5)\n\t" \
  "v_mfma_f32_16x16x128_f8f6f4 v[16:19], v[192:199], v[240:247], v[16:19] blgp:1\n\t" \
  "ds_read_b64_tr_b8 v[232:233], v250 offset:35072\n\t" \
  "ds_read_b64_tr_b8 v[234:235], v250 offset:36096\n\t" \
  "v_mfma_f32_16x16x128_f8f6f4 v[48:51], v[200:207], v[240:247], v[48:51] blgp:1\n\t" \
  "ds_read_b64_tr_b8 v[236:237], v250 offset:43264\n\t" \
  "v_mfma_f32_16x16x128_f8f6f4 v[80:83], v[208:215], v[240:247], v[80:83] blgp:1\n\t" \
  "ds_read_b64_tr_b8 v[238:239], v250 offset:44288\n\t" \
  "v_add_u32 v248, s59, v248\n\t" \
  "" NIPAD "\n\t" \
  "v_mfma_f32_16x16x128_f8f6f4 v[112:115], v[216:223], v[240:247], v[112:115] blgp:1\n\t" \
  "s_mov_b32 m0, 0\n\t" \
  "s_add_u32 s51, s51, 2\n\t" \
  "ds_read_addtid_b32 v253 offset:0\n\t" \
  "s_add_u32 s52, s52, 1\n\t" \
  "s_waitcnt lgkmcnt(6)\n\t" \
  "v_mfma_f32_16x16x128_f8f6f4 v[20:23], v[192:199], v[224:231], v[20:23] blgp:1\n\t" \
  "ds_read_b64_tr_b8 v[240:241], v251 offset:35072\n\t" \
  "ds_read_b64_tr_b8 v[242:243], v251 offset:36096\n\t" \
  "v_mfma_f32_16x16x128_f8f6f4 v[52:55], v[200:207], v[224:231], v[52:55] blgp:1\n\t" \
  "ds_read_b64_tr_b8 v[244:245], v251 offset:43264\n\t" \
  "v_add_u32 v249, s59, v249\n\t" \
  "v_mfma_f32_16x16x128_f8f6f4 v[84:87], v[208:215], v[224:231], v[84:87] blgp:1\n\t" \
  "ds_read_b64_tr_b8 v[246:247], v251 offset:44288\n\t" \
  "" NIPAD "\n\t" \
  "v_mfma_f32_16x16x128_f8f6f4 v[116:119], v[216:223], v[224:231], v[116:119] blgp:1\n\t" \
  "s_waitcnt lgkmcnt(4)\n\t" \
  "Lc7_retry%=:\n\t" \
  "v_readlane_b32 s53, v253, 32\n\t" \
  "v_readlane_b32 s54, v253, s52\n\t" \
  "s_nop 0\n\t" \
  "s_cmp_lt_u32 s53, s51\n\t" \
  "s_cbranch_scc1 Lc7_spin%=\n\t" \
  "s_cmp_eq_u32 s54, 0\n\t" \
  "s_cbranch_scc1 Lc7_spin%=\n\t" \
  "Lc7_ok%=:\n\t" \
  "v_mfma_f32_16x16x128_f8f6f4 v[24:27], v[192:199], v[232:239], v[24:27] blgp:1\n\t" \
  "ds_read_b64_tr_b8 v[224:225], v248 offset:2048\n\t" \
  "ds_read_b64_tr_b8 v[226:227], v248 offset:3072\n\t" \
  "v_mfma_f32_16x16x128_f8f6f4 v[56:59], v[200:207], v[232:239], v[56:59] blgp:1\n\t" \
  "ds_read_b64_tr_b8 v[228:229], v248 offset:10240\n\t" \
  "v_mfma_f32_16x16x128_f8f6f4 v[88:91], v[208:215], v[232:239], v[88:91] blgp:1\n\t" \
  "ds_read_b64_tr_b8 v[230:231], v248 offset:11264\n\t" \
  "v_add_u32 v250, s59, v250\n\t" \
  "" NIPAD "\n\t" \
  "v_mfma_f32_16x16x128_f8f6f4 v[120:123], v[216:223], v[232:239], v[120:123] blgp:1\n\t" \
  "s_and_b32 s54, s54, 127\n\t" \
  "s_lshl_b32 s54, s54, 16\n\t" \
  "s_add_u32 s46, s42, s54\n\t" \
  "s_addc_u32 s47, s43, 0\n\t" \
  "s_waitcnt lgkmcnt(4)\n\t" \
  "v_mfma_f32_16x16x128_f8f6f4 v[28:31], v[192:199], v[240:247], v[28:31] blgp:1\n\t" \
  "ds_read_b64_tr_b8 v[232:233], v249 offset:2048\n\t" \
  "ds_read_b64_tr_b8 v[234:235], v249 offset:3072\n\t" \
  "s_sub_u32 s55, s55, 1\n\t" \
  "v_mfma_f32_16x16x128_f8f6f4 v[60:63], v[200:207], v[240:247], v[60:63] blgp:1\n\t" \
  "ds_read_b64_tr_b8 v[236:237], v249 offset:10240\n\t" \
  "v_add_u32 v251, s59, v251\n\t" \
  "v_mfma_f32_16x16x128_f8f6f4 v[92:95], v[208:215], v[240:247], v[92:95] blgp:1\n\t" \
  "ds_read_b64_tr_b8 v[238:239], v249 offset:11264\n\t" \
  "s_sub_u32 s59, 0, s59\n\t" \
  "" NIPAD "\n\t" \
  "v_mfma_f32_16x16x128_f8f6f4 v[124:127], v[216:223], v[240:247], v[124:127] blgp:1\n\t" \
  "s_cmp_lg_u32 s55, 0\n\t" \
  "s_cbranch_scc1 Lloop%=\n\t" \
  "s_waitcnt vmcnt(8)\n\t" \
  "s_waitcnt lgkmcnt(4)\n\t" \
  "v_mfma_f32_16x16x128_f8f6f4 v[0:3], v[128:135], v[224:231], v[0:3] blgp:1\n\t" \
  "ds_read_b64_tr_b8 v[240:241], v250 offset:2048\n\t" \
  "ds_read_b64_tr_b8 v[242:243], v250 offset:3072\n\t" \
  "" P1PAD "\n\t" \
  "v_mfma_f32_16x16x128_f8f6f4 v[32:35], v[136:143], v[224:231], v[32:35] blgp:1\n\t" \
  "ds_read_b64_tr_b8 v[244:245], v250 offset:10240\n\t" \
  "ds_read_b64_tr_b8 v[246:247], v250 offset:11264\n\t" \
  "ds_read_b64_tr_b8 v[208:209], v251 offset:2048\n\t" \
  "ds_read_b64_tr_b8 v[210:211], v251 offset:3072\n\t" \
  "ds_read_b64_tr_b8 v[212:213], v251 offset:10240\n\t" \
  "ds_read_b64_tr_b8 v[214:215], v251 offset:11264\n\t" \
  "global_load_dwordx4 v[192:195], v252, s[46:47] offset:-4096 " APOL "\n\t" \
  "s_waitcnt lgkmcnt(8)\n\t" \
  "v_mfma_f32_16x16x128_f8f6f4 v[4:7], v[128:135], v[232:239], v[4:7] blgp:1\n\t" \
  "ds_read_b64_tr_b8 v[224:225], v248 offset:2304\n\t" \
  "ds_read_b64_tr_b8 v[226:227], v248 offset:3328\n\t" \
  "" P1PAD "\n\t" \
  "v_mfma_f32_16x16x128_f8f6f4 v[36:39], v[136:143], v[232:239], v[36:39] blgp:1\n\t" \
  "ds_read_b64_tr_b8 v[228:229], v248 offset:10496\n\t" \
  "ds_read_b64_tr_b8 v[230:231], v248 offset:11520\n\t" \
  "global_load_dwordx4 v[196:199], v252, s[46:47] offset:-3072 " APOL "\n\t" \
  "s_waitcnt lgkmcnt(8)\n\t" \
  "v_mfma_f32_16x16x128_f8f6f4 v[8:11], v[128:135], v[240:247], v[8:11] blgp:1\n\t" \
  "ds_read_b64_tr_b8 v[232:233], v249 offset:2304\n\t" \
  "ds_read_b64_tr_b8 v[234:235], v249 offset:3328\n\t" \
  "" P1PAD "\n\t" \
  "v_mfma_f32_16x16x128_f8f6f4 v[40:43], v[136:143], v[240:247], v[40:43] blgp:1\n\t" \
  "ds_read_b64_tr_b8 v[236:237], v249 offset:10496\n\t" \
  "ds_read_b64_tr_b8 v[238:239], v249 offset:11520\n\t" \
  "global_load_dwordx4 v[200:203], v252, s[46:47] offset:-2048 " APOL "\n\t" \
  "s_mov_b32 m0, 0\n\t" \
  "s_add_u32 s51, s51, 2\n\t" \
  "ds_read_addtid_b32 v253 offset:0\n\t" \
  "s_mov_b32 s52, 31\n\t" \
  "s_waitcnt lgkmcnt(9)\n\t" \
  "v_mfma_f32_16x16x128_f8f6f4 v[12:15], v[128:135], v[208:215], v[12:15] blgp:1\n\t" \
  "ds_read_b64_tr_b8 v[240:241], v250 offset:2304\n\t" \
  "ds_read_b64_tr_b8 v[242:243], v250 offset:3328\n\t" \
  "" P1PAD "\n\t" \
  "v_mfma_f32_16x16x128_f8f6f4 v[44:47], v[136:143], v[208:215], v[44:47] blgp:1\n\t" \
  "ds_read_b64_tr_b8 v[244:245], v250 offset:10496\n\t" \
  "ds_read_b64_tr_b8 v[246:247], v250 offset:11520\n\t" \
  "global_load_dwordx4 v[204:207], v252, s[46:47] offset:-1024 " APOL "\n\t" \
  "s_waitcnt lgkmcnt(9)\n\t" \
  "v_mfma_f32_16x16x128_f8f6f4 v[16:19], v[128:135], v[224:231], v[16:19] blgp:1\n\t" \
  "ds_read_b64_tr_b8 v[208:209], v251 offset:2304\n\t" \
  "ds_read_b64_tr_b8 v[210:211], v251 offset:3328\n\t" \
  "" P1PAD "\n\t" \
  "v_mfma_f32_16x16x128_f8f6f4 v[48:51], v[136:143], v[224:231], v[48:51] blgp:1\n\t" \
  "ds_read_b64_tr_b8 v[212:213], v251 offset:10496\n\t" \
  "ds_read_b64_tr_b8 v[214:215], v251 offset:11520\n\t" \
  "s_waitcnt lgkmcnt(8)\n\t" \
  "Lc8_retry%=:\n\t" \
  "v_readlane_b32 s53, v253, 32\n\t" \
  "v_readlane_b32 s54, v253, s52\n\t" \
  "s_nop 0\n\t" \
  "s_cmp_lt_u32 s53, s51\n\t" \
  "s_cbranch_scc1 Lc8_spin%=\n\t" \
  "s_cmp_eq_u32 s54, 0\n\t" \
  "s_cbranch_scc1 Lc8_spin%=\n\t" \
  "Lc8_ok%=:\n\t" \
  "v_mfma_f32_16x16x128_f8f6f4 v[20:23], v[128:135], v[232:239], v[20:23] blgp:1\n\t" \
  "ds_read_b64_tr_b8 v[224:225], v248 offset:18432\n\t" \
  "ds_read_b64_tr_b8 v[226:227], v248 offset:19456\n\t" \
  "" P1PAD "\n\t" \
  "v_mfma_f32_16x16x128_f8f6f4 v[52:55], v[136:143], v[232:239], v[52:55] blgp:1\n\t" \
  "ds_read_b64_tr_b8 v[228:229], v248 offset:26624\n\t" \
  "ds_read_b64_tr_b8 v[230:231], v248 offset:27648\n\t" \
  "v_readlane_b32 s53, v253, 30\n\t" \
  "s_and_b32 s54, s54, 127\n\t" \
  "s_lshl_b32 s54, s54, 16\n\t" \
  "s_add_u32 s74, s42, s54\n\t" \
  "s_addc_u32 s75, s43, 0\n\t" \
  "s_waitcnt lgkmcnt(8)\n\t" \
  "v_mfma_f32_16x16x128_f8f6f4 v[24:27], v[128:135], v[240:247], v[24:27] blgp:1\n\t" \
  "ds_read_b64_tr_b8 v[232:233], v249 offset:18432\n\t" \
  "ds_read_b64_tr_b8 v[234:235], v249 offset:19456\n\t" \
  "" P1PAD "\n\t" \
  "v_mfma_f32_16x16x128_f8f6f4 v[56:59], v[136:143], v[240:247], v[56:59] blgp:1\n\t" \
  "ds_read_b64_tr_b8 v[236:237], v249 offset:26624\n\t" \
  "ds_read_b64_tr_b8 v[238:239], v249 offset:27648\n\t" \
  "s_and_b32 s53, s53, 127\n\t" \
  "s_lshl_b32 s53, s53, 16\n\t" \
  "s_add_u32 s72, s42, s53\n\t" \
  "s_addc_u32 s73, s43, 0\n\t" \
  "s_waitcnt lgkmcnt(8)\n\t" \
  "v_mfma_f32_16x16x128_f8f6f4 v[28:31], v[128:135], v[208:215], v[28:31] blgp:1\n\t" \
  "ds_read_b64_tr_b8 v[240:241], v250 offset:18432\n\t" \
  "ds_read_b64_tr_b8 v[242:243], v250 offset:19456\n\t" \
  "" P1PAD "\n\t" \
  "v_mfma_f32_16x16x128_f8f6f4 v[60:63], v[136:143], v[208:215], v[60:63] blgp:1\n\t" \
  "ds_read_b64_tr_b8 v[244:245], v250 offset:26624\n\t" \
  "ds_read_b64_tr_b8 v[246:247], v250 offset:27648\n\t" \
  "s_waitcnt vmcnt(4)\n\t" \
  "s_waitcnt lgkmcnt(8)\n\t" \
  "v_mfma_f32_16x16x128_f8f6f4 v[0:3], v[160:167], v[224:231], v[0:3] blgp:1\n\t" \
  "ds_read_b64_tr_b8 v[208:209], v251 offset:18432\n\t" \
  "ds_read_b64_tr_b8 v[210:211], v251 offset:19456\n\t" \
  "" P1PAD "\n\t" \
  "v_mfma_f32_16x16x128_f8f6f4 v[32:35], v[168:175], v[224:231], v[32:35] blgp:1\n\t" \
  "ds_read_b64_tr_b8 v[212:213], v251 offset:26624\n\t" \
  "ds_read_b64_tr_b8 v[214:215], v251 offset:27648\n\t" \
  "global_load_dwordx4 v[128:131], v252, s[72:73] offset:-4096 " APOL "\n\t" \
  "s_waitcnt lgkmcnt(8)\n\t" \
  "v_mfma_f32_16x16x128_f8f6f4 v[4:7], v[160:167], v[232:239], v[4:7] blgp:1\n\t" \
  "ds_read_b64_tr_b8 v[224:225], v248 offset:18688\n\t" \
  "ds_read_b64_tr_b8 v[226:227], v248 offset:19712\n\t" \
  "" P1PAD "\n\t" \
  "v_mfma_f32_16x16x128_f8f6f4 v[36:39], v[168:175], v[232:239], v[36:39] blgp:1\n\t" \
  "ds_read_b64_tr_b8 v[228:229], v248 offset:26880\n\t" \
  "ds_read_b64_tr_b8 v[230:231], v248 offset:27904\n\t" \
  "global_load_dwordx4 v[132:135], v252, s[72:73] offset:-3072 " APOL "\n\t" \
  "s_waitcnt lgkmcnt(8)\n\t" \
  "v_mfma_f32_16x16x128_f8f6f4 v[8:11], v[160:167], v[240:247], v[8:11] blgp:1\n\t" \
  "ds_read_b64_tr_b8 v[232:233], v249 offset:18688\n\t" \
  "ds_read_b64_tr_b8 v[234:235], v249 offset:19712\n\t" \
  "" P1PAD "\n\t" \
  "v_mfma_f32_16x16x128_f8f6f4 v[40:43], v[168:175], v[240:247], v[40:43] blgp:1\n\t" \
  "ds_read_b64_tr_b8 v[236:237], v249 offset:26880\n\t" \
  "ds_read_b64_tr_b8 v[238:239], v249 offset:27904\n\t" \
  "global_load_dwordx4 v[136:139], v252, s[72:73] offset:-2048 " APOL "\n\t" \
  "s_mov_b32 m0, 0\n\t" \
  "s_add_u32 s51, s51, 2\n\t" \
  "ds_read_addtid_b32 v253 offset:0\n\t" \
  "s_waitcnt lgkmcnt(9)\n\t" \
  "v_mfma_f32_16x16x128_f8f6f4 v[12:15], v[160:167], v[208:215], v[12:15] blgp:1\n\t" \
  "ds_read_b64_tr_b8 v[240:241], v250 offset:18688\n\t" \
  "ds_read_b64_tr_b8 v[242:243], v250 offset:19712\n\t" \
  "" P1PAD "\n\t" \
  "v_mfma_f32_16x16x128_f8f6f4 v[44:47], v[168:175], v[208:215], v[44:47] blgp:1\n\t" \
  "ds_read_b64_tr_b8 v[244:245], v250 offset:26880\n\t" \
  "ds_read_b64_tr_b8 v[246:247], v250 offset:27904\n\t" \
  "global_load_dwordx4 v[140:143], v252, s[72:73] offset:-1024 " APOL "\n\t" \
  "s_waitcnt lgkmcnt(9)\n\t" \
  "v_mfma_f32_16x16x128_f8f6f4 v[16:19], v[160:167], v[224:231], v[16:19] blgp:1\n\t" \
  "ds_read_b64_tr_b8 v[208:209], v251 offset:18688\n\t" \
  "ds_read_b64_tr_b8 v[210:211], v251 offset:19712\n\t" \
  "" P1PAD "\n\t" \
  "v_mfma_f32_16x16x128_f8f6f4 v[48:51], v[168:175], v[224:231], v[48:51] blgp:1\n\t" \
  "ds_read_b64_tr_b8 v[212:213], v251 offset:26880\n\t" \
  "ds_read_b64_tr_b8 v[214:215], v251 offset:27904\n\t" \
  "s_waitcnt lgkmcnt(8)\n\t" \
  "Lc9_retry%=:\n\t" \
  "v_readlane_b32 s53, v253, 32\n\t" \
  "s_nop 0\n\t" \
  "s_cmp_lt_u32 s53, s51\n\t" \
  "s_cbranch_scc1 Lc9_spin%=\n\t" \
  "Lc9_ok%=:\n\t" \
  "v_mfma_f32_16x16x128_f8f6f4 v[20:23], v[160:167], v[232:239], v[20:23] blgp:1\n\t" \
  "ds_read_b64_tr_b8 v[224:225], v248 offset:34816\n\t" \
  "ds_read_b64_tr_b8 v[226:227], v248 offset:35840\n\t" \
  "" P1PAD "\n\t" \
  "v_mfma_f32_16x16x128_f8f6f4 v[52:55], v[168:175], v[232:239], v[52:55] blgp:1\n\t" \
  "ds_read_b64_tr_b8 v[228:229], v248 offset:43008\n\t" \
  "ds_read_b64_tr_b8 v[230:231], v248 offset:44032\n\t" \
  "s_waitcnt lgkmcnt(8)\n\t" \
  "v_mfma_f32_16x16x128_f8f6f4 v[24:27], v[160:167], v[240:247], v[24:27] blgp:1\n\t" \
  "ds_read_b64_tr_b8 v[232:233], v249 offset:34816\n\t" \
  "ds_read_b64_tr_b8 v[234:235], v249 offset:35840\n\t" \
  "" P1PAD "\n\t" \
  "v_mfma_f32_16x16x128_f8f6f4 v[56:59], v[168:175], v[240:247], v[56:59] blgp:1\n\t" \
  "ds_read_b64_tr_b8 v[236:237], v249 offset:43008\n\t" \
  "ds_read_b64_tr_b8 v[238:239], v249 offset:44032\n\t" \
  "s_waitcnt lgkmcnt(8)\n\t" \
  "v_mfma_f32_16x16x128_f8f6f4 v[28:31], v[160:167], v[208:215], v[28:31] blgp:1\n\t" \
  "ds_read_b64_tr_b8 v[240:241], v250 offset:34816\n\t" \
  "ds_read_b64_tr_b8 v[242:243], v250 offset:35840\n\t" \
  "" P1PAD "\n\t" \
  "v_mfma_f32_16x16x128_f8f6f4 v[60:63], v[168:175], v[208:215], v[60:63] blgp:1\n\t" \
  "ds_read_b64_tr_b8 v[244:245], v250 offset:43008\n\t" \
  "ds_read_b64_tr_b8 v[246:247], v250 offset:44032\n\t" \
  "s_waitcnt vmcnt(4)\n\t" \
  "s_waitcnt lgkmcnt(8)\n\t" \
  "v_mfma_f32_16x16x128_f8f6f4 v[0:3], v[192:199], v[224:231], v[0:3] blgp:1\n\t" \
  "ds_read_b64_tr_b8 v[208:209], v251 offset:34816\n\t" \
  "ds_read_b64_tr_b8 v[210:211], v251 offset:35840\n\t" \
  "" P1PAD "\n\t" \
  "v_mfma_f32_16x16x128_f8f6f4 v[32:35], v[200:207], v[224:231], v[32:35] blgp:1\n\t" \
  "ds_read_b64_tr_b8 v[212:213], v251 offset:43008\n\t" \
  "ds_read_b64_tr_b8 v[214:215], v251 offset:44032\n\t" \
  "global_load_dwordx4 v[160:163], v252, s[74:75] offset:-4096 " APOL "\n\t" \
  "s_waitcnt lgkmcnt(8)\n\t" \
  "v_mfma_f32_16x16x128_f8f6f4 v[4:7], v[192:199], v[232:239], v[4:7] blgp:1\n\t" \
  "ds_read_b64_tr_b8 v[224:225], v248 offset:35072\n\t" \
  "ds_read_b64_tr_b8 v[226:227], v248 offset:36096\n\t" \
  "" P1PAD "\n\t" \
  "v_mfma_f32_16x16x128_f8f6f4 v[36:39], v[200:207], v[232:239], v[36:39] blgp:1\n\t" \
  "ds_read_b64_tr_b8 v[228:229], v248 offset:43264\n\t" \
  "ds_read_b64_tr_b8 v[230:231], v248 offset:44288\n\t" \
  "global_load_dwordx4 v[164:167], v252, s[74:75] offset:-3072 " APOL "\n\t" \
  "s_waitcnt lgkmcnt(8)\n\t" \
  "v_mfma_f32_16x16x128_f8f6f4 v[8:11], v[192:199], v[240:247], v[8:11] blgp:1\n\t" \
  "ds_read_b64_tr_b8 v[232:233], v249 offset:35072\n\t" \
  "ds_read_b64_tr_b8 v[234:235], v249 offset:36096\n\t" \
  "" P1PAD "\n\t" \
  "v_mfma_f32_16x16x128_f8f6f4 v[40:43], v[200:207], v[240:247], v[40:43] blgp:1\n\t" \
  "ds_read_b64_tr_b8 v[236:237], v249 offset:43264\n\t" \
  "ds_read_b64_tr_b8 v[238:239], v249 offset:44288\n\t" \
  "global_load_dwordx4 v[168:171], v252, s[74:75] offset:-2048 " APOL "\n\t" \
  "s_mov_b32 m0, 0\n\t" \
  "s_add_u32 s51, s51, 2\n\t" \
  "ds_read_addtid_b32 v253 offset:0\n\t" \
  "s_waitcnt lgkmcnt(9)\n\t" \
  "v_mfma_f32_16x16x128_f8f6f4 v[12:15], v[192:199], v[208:215], v[12:15] blgp:1\n\t" \
  "ds_read_b64_tr_b8 v[240:241], v250 offset:35072\n\t" \
  "ds_read_b64_tr_b8 v[242:243], v250 offset:36096\n\t" \
  "" P1PAD "\n\t" \
  "v_mfma_f32_16x16x128_f8f6f4 v[44:47], v[200:207], v[208:215], v[44:47] blgp:1\n\t" \
  "ds_read_b64_tr_b8 v[244:245], v250 offset:43264\n\t" \
  "ds_read_b64_tr_b8 v[246:247], v250 offset:44288\n\t" \
  "global_load_dwordx4 v[172:175], v252, s[74:75] offset:-1024 " APOL "\n\t" \
  "s_waitcnt lgkmcnt(9)\n\t" \
  "v_mfma_f32_16x16x128_f8f6f4 v[16:19], v[192:199], v[224:231], v[16:19] blgp:1\n\t" \
  "ds_read_b64_tr_b8 v[208:209], v251 offset:35072\n\t" \
  "ds_read_b64_tr_b8 v[210:211], v251 offset:36096\n\t" \
  "" P1PAD "\n\t" \
  "v_mfma_f32_16x16x128_f8f6f4 v[48:51], v[200:207], v[224:231], v[48:51] blgp:1\n\t" \
  "ds_read_b64_tr_b8 v[212:213], v251 offset:43264\n\t" \
  "ds_read_b64_tr_b8 v[214:215], v251 offset:44288\n\t" \
  "s_waitcnt lgkmcnt(8)\n\t" \
  "Lc10_retry%=:\n\t" \
  "v_readlane_b32 s53, v253, 32\n\t" \
  "s_nop 0\n\t" \
  "s_cmp_lt_u32 s53, s51\n\t" \
  "s_cbranch_scc1 Lc10_spin%=\n\t" \
  "Lc10_ok%=:\n\t" \
  "v_mfma_f32_16x16x128_f8f6f4 v[20:23], v[192:199], v[232:239], v[20:23] blgp:1\n\t" \
  "v_add_u32 v248, 0xc000, v248\n\t" \
  "ds_read_b64_tr_b8 v[224:225], v248 offset:2048\n\t" \
  "ds_read_b64_tr_b8 v[226:227], v248 offset:3072\n\t" \
  "" P1PAD "\n\t" \
  "v_mfma_f32_16x16x128_f8f6f4 v[52:55], v[200:207], v[232:239], v[52:55] blgp:1\n\t" \
  "ds_read_b64_tr_b8 v[228:229], v248 offset:10240\n\t" \
  "ds_read_b64_tr_b8 v[230:231], v248 offset:11264\n\t" \
  "s_waitcnt lgkmcnt(8)\n\t" \
  "v_mfma_f32_16x16x128_f8f6f4 v[24:27], v[192:199], v[240:247], v[24:27] blgp:1\n\t" \
  "v_add_u32 v249, 0xc000, v249\n\t" \
  "ds_read_b64_tr_b8 v[232:233], v249 offset:2048\n\t" \
  "ds_read_b64_tr_b8 v[234:235], v249 offset:3072\n\t" \
  "" P1PAD "\n\t" \
  "v_mfma_f32_16x16x128_f8f6f4 v[56:59], v[200:207], v[240:247], v[56:59] blgp:1\n\t" \
  "ds_read_b64_tr_b8 v[236:237], v249 offset:10240\n\t" \
  "ds_read_b64_tr_b8 v[238:239], v249 offset:11264\n\t" \
  "s_waitcnt lgkmcnt(8)\n\t" \
  "v_mfma_f32_16x16x128_f8f6f4 v[28:31], v[192:199], v[208:215], v[28:31] blgp:1\n\t" \
  "v_add_u32 v250, 0xc000, v250\n\t" \
  "ds_read_b64_tr_b8 v[240:241], v250 offset:2048\n\t" \
  "ds_read_b64_tr_b8 v[242:243], v250 offset:3072\n\t" \
  "" P1PAD "\n\t" \
  "v_mfma_f32_16x16x128_f8f6f4 v[60:63], v[200:207], v[208:215], v[60:63] blgp:1\n\t" \
  "ds_read_b64_tr_b8 v[244:245], v250 offset:10240\n\t" \
  "ds_read_b64_tr_b8 v[246:247], v250 offset:11264\n\t" \
  "s_waitcnt vmcnt(4)\n\t" \
  "s_waitcnt lgkmcnt(8)\n\t" \
  "v_mfma_f32_16x16x128_f8f6f4 v[0:3], v[128:135], v[224:231], v[0:3] blgp:1\n\t" \
  "v_add_u32 v251, 0xc000, v251\n\t" \
  "ds_read_b64_tr_b8 v[208:209], v251 offset:2048\n\t" \
  "ds_read_b64_tr_b8 v[210:211], v251 offset:3072\n\t" \
  "" P1PAD "\n\t" \
  "v_mfma_f32_16x16x128_f8f6f4 v[32:35], v[136:143], v[224:231], v[32:35] blgp:1\n\t" \
  "ds_read_b64_tr_b8 v[212:213], v251 offset:10240\n\t" \
  "ds_read_b64_tr_b8 v[214:215], v251 offset:11264\n\t" \
  "global_load_dwordx4 v[192:195], v252, s[46:47] offset:0 " APOL "\n\t" \
  "s_waitcnt lgkmcnt(8)\n\t" \
  "v_mfma_f32_16x16x128_f8f6f4 v[4:7], v[128:135], v[232:239], v[4:7] blgp:1\n\t" \
  "ds_read_b64_tr_b8 v[224:225], v248 offset:2304\n\t" \
  "ds_read_b64_tr_b8 v[226:227], v248 offset:3328\n\t" \
  "" P1PAD "\n\t" \
  "v_mfma_f32_16x16x128_f8f6f4 v[36:39], v[136:143], v[232:239], v[36:39] blgp:1\n\t" \
  "ds_read_b64_tr_b8 v[228:229], v248 offset:10496\n\t" \
  "ds_read_b64_tr_b8 v[230:231], v248 offset:11520\n\t" \
  "global_load_dwordx4 v[196:199], v252, s[46:47] offset:1024 " APOL "\n\t" \
  "s_waitcnt lgkmcnt(8)\n\t" \
  "v_mfma_f32_16x16x128_f8f6f4 v[8:11], v[128:135], v[240:247], v[8:11] blgp:1\n\t" \
  "ds_read_b64_tr_b8 v[232:233], v249 offset:2304\n\t" \
  "ds_read_b64_tr_b8 v[234:235], v249 offset:3328\n\t" \
  "" P1PAD "\n\t" \
  "v_mfma_f32_16x16x128_f8f6f4 v[40:43], v[136:143], v[240:247], v[40:43] blgp:1\n\t" \
  "ds_read_b64_tr_b8 v[236:237], v249 offset:10496\n\t" \
  "ds_read_b64_tr_b8 v[238:239], v249 offset:11520\n\t" \
  "global_load_dwordx4 v[200:203], v252, s[46:47] offset:2048 " APOL "\n\t" \
  "s_mov_b32 m0, 0\n\t" \
  "s_add_u32 s51, s51, 2\n\t" \
  "ds_read_addtid_b32 v253 offset:0\n\t" \
  "s_waitcnt lgkmcnt(9)\n\t" \
  "v_mfma_f32_16x16x128_f8f6f4 v[12:15], v[128:135], v[208:215], v[12:15] blgp:1\n\t" \
  "ds_read_b64_tr_b8 v[240:241], v250 offset:2304\n\t" \
  "ds_read_b64_tr_b8 v[242:243], v250 offset:3328\n\t" \
  "" P1PAD "\n\t" \
  "v_mfma_f32_16x16x128_f8f6f4 v[44:47], v[136:143], v[208:215], v[44:47] blgp:1\n\t" \
  "ds_read_b64_tr_b8 v[244:245], v250 offset:10496\n\t" \
  "ds_read_b64_tr_b8 v[246:247], v250 offset:11520\n\t" \
  "global_load_dwordx4 v[204:207], v252, s[46:47] offset:3072 " APOL "\n\t" \
  "s_waitcnt lgkmcnt(9)\n\t" \
  "v_mfma_f32_16x16x128_f8f6f4 v[16:19], v[128:135], v[224:231], v[16:19] blgp:1\n\t" \
  "ds_read_b64_tr_b8 v[208:209], v251 offset:2304\n\t" \
  "ds_read_b64_tr_b8 v[210:211], v251 offset:3328\n\t" \
  "" P1PAD "\n\t" \
  "v_mfma_f32_16x16x128_f8f6f4 v[48:51], v[136:143], v[224:231], v[48:51] blgp:1\n\t" \
  "ds_read_b64_tr_b8 v[212:213], v251 offset:10496\n\t" \
  "ds_read_b64_tr_b8 v[214:215], v251 offset:11520\n\t" \
  "s_waitcnt lgkmcnt(8)\n\t" \
  "Lc11_retry%=:\n\t" \
  "v_readlane_b32 s53, v253, 32\n\t" \
  "s_nop 0\n\t" \
  "s_cmp_lt_u32 s53, s51\n\t" \
  "s_cbranch_scc1 Lc11_spin%=\n\t" \
  "Lc11_ok%=:\n\t" \
  "v_mfma_f32_16x16x128_f8f6f4 v[20:23], v[128:135], v[232:239], v[20:23] blgp:1\n\t" \
  "ds_read_b64_tr_b8 v[224:225], v248 offset:18432\n\t" \
  "ds_read_b64_tr_b8 v[226:227], v248 offset:19456\n\t" \
  "" P1PAD "\n\t" \
  "v_mfma_f32_16x16x128_f8f6f4 v[52:55], v[136:143], v[232:239], v[52:55] blgp:1\n\t" \
  "ds_read_b64_tr_b8 v[228:229], v248 offset:26624\n\t" \
  "ds_read_b64_tr_b8 v[230:231], v248 offset:27648\n\t" \
  "v_readlane_b32 s53, v253, 33\n\t" \
  "s_waitcnt lgkmcnt(8)\n\t" \
  "v_mfma_f32_16x16x128_f8f6f4 v[24:27], v[128:135], v[240:247], v[24:27] blgp:1\n\t" \
  "ds_read_b64_tr_b8 v[232:233], v249 offset:18432\n\t" \
  "ds_read_b64_tr_b8 v[234:235], v249 offset:19456\n\t" \
  "" P1PAD "\n\t" \
  "v_mfma_f32_16x16x128_f8f6f4 v[56:59], v[136:143], v[240:247], v[56:59] blgp:1\n\t" \
  "ds_read_b64_tr_b8 v[236:237], v249 offset:26624\n\t" \
  "ds_read_b64_tr_b8 v[238:239], v249 offset:27648\n\t" \
  "s_cmp_lg_u32 s53, 0\n\t" \
  "s_cselect_b32 s48, 0x7fc00000, s48\n\t" \
  "s_mov_b32 s94, s48\n\t" \
  "s_mov_b32 s95, s48\n\t" \
  "s_mov_b32 s96, s49\n\t" \
  "s_mov_b32 s97, s49\n\t" \
  "s_waitcnt lgkmcnt(8)\n\t" \
  "v_mfma_f32_16x16x128_f8f6f4 v[28:31], v[128:135], v[208:215], v[28:31] blgp:1\n\t" \
  "ds_read_b64_tr_b8 v[240:241], v250 offset:18432\n\t" \
  "ds_read_b64_tr_b8 v[242:243], v250 offset:19456\n\t" \
  "" P1PAD "\n\t" \
  "v_mfma_f32_16x16x128_f8f6f4 v[60:63], v[136:143], v[208:215], v[60:63] blgp:1\n\t" \
  "ds_read_b64_tr_b8 v[244:245], v250 offset:26624\n\t" \
  "ds_read_b64_tr_b8 v[246:247], v250 offset:27648\n\t" \
  "s_waitcnt vmcnt(4)\n\t" \
  "s_waitcnt lgkmcnt(8)\n\t" \
  "v_mfma_f32_16x16x128_f8f6f4 v[0:3], v[160:167], v[224:231], v[0:3] blgp:1\n\t" \
  "ds_read_b64_tr_b8 v[208:209], v251 offset:18432\n\t" \
  "ds_read_b64_tr_b8 v[210:211], v251 offset:19456\n\t" \
  "" P1PAD "\n\t" \
  "v_mbcnt_lo_u32_b32 v255, -1, 0\n\t" \
  "v_mfma_f32_16x16x128_f8f6f4 v[32:35], v[168:175], v[224:231], v[32:35] blgp:1\n\t" \
  "ds_read_b64_tr_b8 v[212:213], v251 offset:26624\n\t" \
  "ds_read_b64_tr_b8 v[214:215], v251 offset:27648\n\t" \
  "global_load_dwordx4 v[128:131], v252, s[72:73] offset:0 " APOL "\n\t" \
  "v_mbcnt_hi_u32_b32 v255, -1, v255\n\t" \
  "s_waitcnt lgkmcnt(8)\n\t" \
  "v_mfma_f32_16x16x128_f8f6f4 v[4:7], v[160:167], v[232:239], v[4:7] blgp:1\n\t" \
  "ds_read_b64_tr_b8 v[224:225], v248 offset:18688\n\t" \
  "ds_read_b64_tr_b8 v[226:227], v248 offset:19712\n\t" \
  "" P1PAD "\n\t" \
  "v_lshrrev_b32 v254, 5, v255\n\t" \
  "v_mfma_f32_16x16x128_f8f6f4 v[36:39], v[168:175], v[232:239], v[36:39] blgp:1\n\t" \
  "ds_read_b64_tr_b8 v[228:229], v248 offset:26880\n\t" \
  "ds_read_b64_tr_b8 v[230:231], v248 offset:27904\n\t" \
  "global_load_dwordx4 v[132:135], v252, s[72:73] offset:1024 " APOL "\n\t" \
  "v_and_b32 v255, 31, v255\n\t" \
  "s_waitcnt lgkmcnt(8)\n\t" \
  "v_mfma_f32_16x16x128_f8f6f4 v[8:11], v[160:167], v[240:247], v[8:11] blgp:1\n\t" \
  "ds_read_b64_tr_b8 v[232:233], v249 offset:18688\n\t" \
  "ds_read_b64_tr_b8 v[234:235], v249 offset:19712\n\t" \
  "" P1PAD "\n\t" \
  "v_lshlrev_b32 v255, 2, v255\n\t" \
  "v_mfma_f32_16x16x128_f8f6f4 v[40:43], v[168:175], v[240:247], v[40:43] blgp:1\n\t" \
  "ds_read_b64_tr_b8 v[236:237], v249 offset:26880\n\t" \
  "ds_read_b64_tr_b8 v[238:239], v249 offset:27904\n\t" \
  "global_load_dwordx4 v[136:139], v252, s[72:73] offset:2048 " APOL "\n\t" \
  "v_lshl_add_u32 v254, v254, 19, v255\n\t" \
  "s_waitcnt lgkmcnt(8)\n\t" \
  "v_mfma_f32_16x16x128_f8f6f4 v[12:15], v[160:167], v[208:215], v[12:15] blgp:1\n\t" \
  "ds_read_b64_tr_b8 v[240:241], v250 offset:18688\n\t" \
  "ds_read_b64_tr_b8 v[242:243], v250 offset:19712\n\t" \
  "" P1PAD "\n\t" \
  "v_add_u32 v255, 0x40000, v254\n\t" \
  "v_mfma_f32_16x16x128_f8f6f4 v[44:47], v[168:175], v[208:215], v[44:47] blgp:1\n\t" \
  "ds_read_b64_tr_b8 v[244:245], v250 offset:26880\n\t" \
  "ds_read_b64_tr_b8 v[246:247], v250 offset:27904\n\t" \
  "global_load_dwordx4 v[140:143], v252, s[72:73] offset:3072 " APOL "\n\t" \
  "s_mov_b32 s92, 0x42ad1f97\n\t" \
  "s_waitcnt lgkmcnt(8)\n\t" \
  "v_mfma_f32_16x16x128_f8f6f4 v[16:19], v[160:167], v[224:231], v[16:19] blgp:1\n\t" \
  "ds_read_b64_tr_b8 v[208:209], v251 offset:18688\n\t" \
  "ds_read_b64_tr_b8 v[210:211], v251 offset:19712\n\t" \
  "" P1PAD "\n\t" \
  "s_add_u32 s76, s44, 0x0\n\ts_addc_u32 s77, s45, 0\n\t" \
  "v_mfma_f32_16x16x128_f8f6f4 v[48:51], v[168:175], v[224:231], v[48:51] blgp:1\n\t" \
  "ds_read_b64_tr_b8 v[212:213], v251 offset:26880\n\t" \
  "ds_read_b64_tr_b8 v[214:215], v251 offset:27904\n\t" \
  "s_add_u32 s78, s44, 0x10000\n\ts_addc_u32 s79, s45, 0\n\t" \
  "s_waitcnt lgkmcnt(8)\n\t" \
  "v_mfma_f32_16x16x128_f8f6f4 v[20:23], v[160:167], v[232:239], v[20:23] blgp:1\n\t" \
  "v_subrev_u32 v248, 0xc000, v248\n\t" \
  "ds_read_b64_tr_b8 v[224:225], v248 offset:2048\n\t" \
  "ds_read_b64_tr_b8 v[226:227], v248 offset:3072\n\t" \
  "" P1PAD "\n\t" \
  "s_add_u32 s80, s44, 0x20000\n\ts_addc_u32 s81, s45, 0\n\t" \
  "v_mfma_f32_16x16x128_f8f6f4 v[52:55], v[168:175], v[232:239], v[52:55] blgp:1\n\t" \
  "ds_read_b64_tr_b8 v[228:229], v248 offset:10240\n\t" \
  "ds_read_b64_tr_b8 v[230:231], v248 offset:11264\n\t" \
  "s_add_u32 s82, s44, 0x30000\n\ts_addc_u32 s83, s45, 0\n\t" \
  "s_waitcnt lgkmcnt(8)\n\t" \
  "v_mfma_f32_16x16x128_f8f6f4 v[24:27], v[160:167], v[240:247], v[24:27] blgp:1\n\t" \
  "v_subrev_u32 v249, 0xc000, v249\n\t" \
  "ds_read_b64_tr_b8 v[232:233], v249 offset:2048\n\t" \
  "ds_read_b64_tr_b8 v[234:235], v249 offset:3072\n\t" \
  "" P1PAD "\n\t" \
  "s_add_u32 s84, s44, 0x100000\n\ts_addc_u32 s85, s45, 0\n\t" \
  "v_mfma_f32_16x16x128_f8f6f4 v[56:59], v[168:175], v[240:247], v[56:59] blgp:1\n\t" \
  "ds_read_b64_tr_b8 v[236:237], v249 offset:10240\n\t" \
  "ds_read_b64_tr_b8 v[238:239], v249 offset:11264\n\t" \
  "s_add_u32 s86, s44, 0x110000\n\ts_addc_u32 s87, s45, 0\n\t" \
  "s_waitcnt lgkmcnt(8)\n\t" \
  "v_mfma_f32_16x16x128_f8f6f4 v[28:31], v[160:167], v[208:215], v[28:31] blgp:1\n\t" \
  "v_subrev_u32 v250, 0xc000, v250\n\t" \
  "ds_read_b64_tr_b8 v[240:241], v250 offset:2048\n\t" \
  "ds_read_b64_tr_b8 v[242:243], v250 offset:3072\n\t" \
  "" P1PAD "\n\t" \
  "s_add_u32 s88, s44, 0x120000\n\ts_addc_u32 s89, s45, 0\n\t" \
  "v_mfma_f32_16x16x128_f8f6f4 v[60:63], v[168:175], v[208:215], v[60:63] blgp:1\n\t" \
  "ds_read_b64_tr_b8 v[244:245], v250 offset:10240\n\t" \
  "ds_read_b64_tr_b8 v[246:247], v250 offset:11264\n\t" \
  "s_add_u32 s90, s44, 0x130000\n\ts_addc_u32 s91, s45, 0\n\t" \
  "s_waitcnt lgkmcnt(8)\n\t" \
  "v_mfma_f32_16x16x128_f8f6f4 v[64:67], v[144:151], v[224:231], v[64:67] blgp:1\n\t" \
  "v_subrev_u32 v251, 0xc000, v251\n\t" \
  "ds_read_b64_tr_b8 v[208:209], v251 offset:2048\n\t" \
  "ds_read_b64_tr_b8 v[210:211], v251 offset:3072\n\t" \
  "v_mfma_f32_16x16x128_f8f6f4 v[96:99], v[152:159], v[224:231], v[96:99] blgp:1\n\t" \
  "ds_read_b64_tr_b8 v[212:213], v251 offset:10240\n\t" \
  "ds_read_b64_tr_b8 v[214:215], v251 offset:11264\n\t" \
  "global_load_dwordx4 v[160:163], v252, s[74:75] offset:0 " APOL "\n\t" \
  "s_waitcnt lgkmcnt(8)\n\t" \
  "v_mfma_f32_16x16x128_f8f6f4 v[68:71], v[144:151], v[232:239], v[68:71] blgp:1\n\t" \
  "ds_read_b64_tr_b8 v[224:225], v248 offset:2304\n\t" \
  "ds_read_b64_tr_b8 v[226:227], v248 offset:3328\n\t" \
  "v_mfma_f32_16x16x128_f8f6f4 v[100:103], v[152:159], v[232:239], v[100:103] blgp:1\n\t" \
  "ds_read_b64_tr_b8 v[228:229], v248 offset:10496\n\t" \
  "ds_read_b64_tr_b8 v[230:231], v248 offset:11520\n\t" \
  "global_load_dwordx4 v[164:167], v252, s[74:75] offset:1024 " APOL "\n\t" \
  "s_waitcnt lgkmcnt(8)\n\t" \
  "v_mfma_f32_16x16x128_f8f6f4 v[72:75], v[144:151], v[240:247], v[72:75] blgp:1\n\t" \
  "ds_read_b64_tr_b8 v[232:233], v249 offset:2304\n\t" \
  "ds_read_b64_tr_b8 v[234:235], v249 offset:3328\n\t" \
  "v_mfma_f32_16x16x128_f8f6f4 v[104:107], v[152:159], v[240:247], v[104:107] blgp:1\n\t" \
  "ds_read_b64_tr_b8 v[236:237], v249 offset:10496\n\t" \
  "ds_read_b64_tr_b8 v[238:239], v249 offset:11520\n\t" \
  "global_load_dwordx4 v[168:171], v252, s[74:75] offset:2048 " APOL "\n\t" \
  "s_waitcnt lgkmcnt(8)\n\t" \
  "v_mfma_f32_16x16x128_f8f6f4 v[76:79], v[144:151], v[208:215], v[76:79] blgp:1\n\t" \
  "ds_read_b64_tr_b8 v[240:241], v250 offset:2304\n\t" \
  "ds_read_b64_tr_b8 v[242:243], v250 offset:3328\n\t" \
  "v_mfma_f32_16x16x128_f8f6f4 v[108:111], v[152:159], v[208:215], v[108:111] blgp:1\n\t" \
  "ds_read_b64_tr_b8 v[244:245], v250 offset:10496\n\t" \
  "ds_read_b64_tr_b8 v[246:247], v250 offset:11520\n\t" \
  "global_load_dwordx4 v[172:175], v252, s[74:75] offset:3072 " APOL "\n\t" \
  "s_waitcnt lgkmcnt(8)\n\t" \
  "v_mfma_f32_16x16x128_f8f6f4 v[80:83], v[144:151], v[224:231], v[80:83] blgp:1\n\t" \
  "ds_read_b64_tr_b8 v[208:209], v251 offset:2304\n\t" \
  "ds_read_b64_tr_b8 v[210:211], v251 offset:3328\n\t" \
  "v_mul_f32 v216, s49, v0\n\t" \
  "v_mul_f32 v217, s49, v1\n\t" \
  "v_mul_f32 v218, s49, v2\n\t" \
  "v_mul_f32 v219, s49, v3\n\t" \
  "v_mul_f32 v220, s49, v4\n\t" \
  "v_mul_f32 v221, s49, v5\n\t" \
  "v_mul_f32 v222, s49, v6\n\t" \
  "v_mul_f32 v223, s49, v7\n\t" \
  "v_min_f32 v216, s92, v216\n\t" \
  "v_mfma_f32_16x16x128_f8f6f4 v[112:115], v[152:159], v[224:231], v[112:115] blgp:1\n\t" \
  "ds_read_b64_tr_b8 v[212:213], v251 offset:10496\n\t" \
  "ds_read_b64_tr_b8 v[214:215], v251 offset:11520\n\t" \
  "v_min_f32 v217, s92, v217\n\t" \
  "v_min_f32 v218, s92, v218\n\t" \
  "v_min_f32 v219, s92, v219\n\t" \
  "v_min_f32 v220, s92, v220\n\t" \
  "v_min_f32 v221, s92, v221\n\t" \
  "v_min_f32 v222, s92, v222\n\t" \
  "v_min_f32 v223, s92, v223\n\t" \
  "v_exp_f32 v216, v216\n\t" \
  "s_waitcnt lgkmcnt(8)\n\t" \
  "v_mfma_f32_16x16x128_f8f6f4 v[84:87], v[144:151], v[232:239], v[84:87] blgp:1\n\t" \
  "ds_read_b64_tr_b8 v[224:225], v248 offset:18432\n\t" \
  "ds_read_b64_tr_b8 v[226:227], v248 offset:19456\n\t" \
  "v_exp_f32 v217, v217\n\t" \
  "v_exp_f32 v218, v218\n\t" \
  "v_exp_f32 v219, v219\n\t" \
  "v_exp_f32 v220, v220\n\t" \
  "v_exp_f32 v221, v221\n\t" \
  "v_mfma_f32_16x16x128_f8f6f4 v[116:119], v[152:159], v[232:239], v[116:119] blgp:1\n\t" \
  "ds_read_b64_tr_b8 v[228:229], v248 offset:26624\n\t" \
  "ds_read_b64_tr_b8 v[230:231], v248 offset:27648\n\t" \
  "v_exp_f32 v222, v222\n\t" \
  "v_exp_f32 v223, v223\n\t" \
  "v_mul_f32 v0, s48, v0\n\t" \
  "v_mul_f32 v1, s48, v1\n\t" \
  "v_mul_f32 v2, s48, v2\n\t" \
  "v_mul_f32 v3, s48, v3\n\t" \
  "v_mul_f32 v4, s48, v4\n\t" \
  "s_waitcnt lgkmcnt(8)\n\t" \
  "v_mfma_f32_16x16x128_f8f6f4 v[88:91], v[144:151], v[240:247], v[88:91] blgp:1\n\t" \
  "ds_read_b64_tr_b8 v[232:233], v249 offset:18432\n\t" \
  "ds_read_b64_tr_b8 v[234:235], v249 offset:19456\n\t" \
  "v_mul_f32 v5, s48, v5\n\t" \
  "v_mul_f32 v6, s48, v6\n\t" \
  "v_mul_f32 v7, s48, v7\n\t" \
  "v_add_f32 v216, 1.0, v216\n\t" \
  "v_add_f32 v217, 1.0, v217\n\t" \
  "v_add_f32 v218, 1.0, v218\n\t" \
  "v_add_f32 v219, 1.0, v219\n\t" \
  "v_add_f32 v220, 1.0, v220\n\t" \
  "v_add_f32 v221, 1.0, v221\n\t" \
  "v_mfma_f32_16x16x128_f8f6f4 v[120:123], v[152:159], v[240:247], v[120:123] blgp:1\n\t" \
  "ds_read_b64_tr_b8 v[236:237], v249 offset:26624\n\t" \
  "ds_read_b64_tr_b8 v[238:239], v249 offset:27648\n\t" \
  "v_add_f32 v222, 1.0, v222\n\t" \
  "v_add_f32 v223, 1.0, v223\n\t" \
  "v_rcp_f32 v216, v216\n\t" \
  "v_rcp_f32 v217, v217\n\t" \
  "v_rcp_f32 v218, v218\n\t" \
  "v_rcp_f32 v219, v219\n\t" \
  "s_waitcnt lgkmcnt(8)\n\t" \
  "v_mfma_f32_16x16x128_f8f6f4 v[92:95], v[144:151], v[208:215], v[92:95] blgp:1\n\t" \
  "ds_read_b64_tr_b8 v[240:241], v250 offset:18432\n\t" \
  "ds_read_b64_tr_b8 v[242:243], v250 offset:19456\n\t" \
  "v_rcp_f32 v220, v220\n\t" \
  "v_rcp_f32 v221, v221\n\t" \
  "v_rcp_f32 v222, v222\n\t" \
  "v_rcp_f32 v223, v223\n\t" \
  "s_nop 0\n\t" \
  "v_mfma_f32_16x16x128_f8f6f4 v[124:127], v[152:159], v[208:215], v[124:127] blgp:1\n\t" \
  "ds_read_b64_tr_b8 v[244:245], v250 offset:26624\n\t" \
  "ds_read_b64_tr_b8 v[246:247], v250 offset:27648\n\t" \
  "v_mul_f32 v0, v0, v216\n\t" \
  "v_mul_f32 v1, v1, v217\n\t" \
  "v_mul_f32 v2, v2, v218\n\t" \
  "v_mul_f32 v3, v3, v219\n\t" \
  "v_mul_f32 v4, v4, v220\n\t" \
  "v_mul_f32 v5, v5, v221\n\t" \
  "v_mul_f32 v6, v6, v222\n\t" \
  "v_mul_f32 v7, v7, v223\n\t" \
  "s_nop 1\n\t" \
  "s_waitcnt lgkmcnt(8)\n\t" \
  "v_mfma_f32_16x16x128_f8f6f4 v[64:67], v[176:183], v[224:231], v[64:67] blgp:1\n\t" \
  "ds_read_b64_tr_b8 v[208:209], v251 offset:18432\n\t" \
  "ds_read_b64_tr_b8 v[210:211], v251 offset:19456\n\t" \
  "v_permlane16_swap_b32 v0, v4\n\t" \
  "v_permlane16_swap_b32 v1, v5\n\t" \
  "v_permlane16_swap_b32 v2, v6\n\t" \
  "v_permlane16_swap_b32 v3, v7\n\t" \
  "global_store_dword v254, v0, s[76:77] offset:0 " STPOL "\n\t" \
  "global_store_dword v254, v1, s[78:79] offset:0 " STPOL "\n\t" \
  "global_store_dword v254, v2, s[80:81] offset:0 " STPOL "\n\t" \
  "global_store_dword v254, v3, s[82:83] offset:0 " STPOL "\n\t" \
  "global_store_dword v255, v4, s[76:77] offset:0 " STPOL "\n\t" \
  "v_mfma_f32_16x16x128_f8f6f4 v[96:99], v[184:191], v[224:231], v[96:99] blgp:1\n\t" \
  "ds_read_b64_tr_b8 v[212:213], v251 offset:26624\n\t" \
  "ds_read_b64_tr_b8 v[214:215], v251 offset:27648\n\t" \
  "global_store_dword v255, v5, s[78:79] offset:0 " STPOL "\n\t" \
  "global_store_dword v255, v6, s[80:81] offset:0 " STPOL "\n\t" \
  "global_store_dword v255, v7, s[82:83] offset:0 " STPOL "\n\t" \
  "v_mul_f32 v216, s49, v32\n\t" \
  "v_mul_f32 v217, s49, v33\n\t" \
  "v_mul_f32 v218, s49, v34\n\t" \
  "v_mul_f32 v219, s49, v35\n\t" \
  "v_mul_f32 v220, s49, v36\n\t" \
  "v_mul_f32 v221, s49, v37\n\t" \
  "s_waitcnt lgkmcnt(8)\n\t" \
  "v_mfma_f32_16x16x128_f8f6f4 v[68:71], v[176:183], v[232:239], v[68:71] blgp:1\n\t" \
  "ds_read_b64_tr_b8 v[224:225], v248 offset:18688\n\t" \
  "ds_read_b64_tr_b8 v[226:227], v248 offset:19712\n\t" \
  "v_mul_f32 v222, s49, v38\n\t" \
  "v_mul_f32 v223, s49, v39\n\t" \
  "v_min_f32 v216, s92, v216\n\t" \
  "v_min_f32 v217, s92, v217\n\t" \
  "v_min_f32 v218, s92, v218\n\t" \
  "v_min_f32 v219, s92, v219\n\t" \
  "v_min_f32 v220, s92, v220\n\t" \
  "v_min_f32 v221, s92, v221\n\t" \
  "v_min_f32 v222, s92, v222\n\t" \
  "v_mfma_f32_16x16x128_f8f6f4 v[100:103], v[184:191], v[232:239], v[100:103] blgp:1\n\t" \
  "ds_read_b64_tr_b8 v[228:229], v248 offset:26880\n\t" \
  "ds_read_b64_tr_b8 v[230:231], v248 offset:27904\n\t" \
  "v_min_f32 v223, s92, v223\n\t" \
  "v_exp_f32 v216, v216\n\t" \
  "v_exp_f32 v217, v217\n\t" \
  "v_exp_f32 v218, v218\n\t" \
  "v_exp_f32 v219, v219\n\t" \
  "s_waitcnt lgkmcnt(8)\n\t" \
  "v_mfma_f32_16x16x128_f8f6f4 v[72:75], v[176:183], v[240:247], v[72:75] blgp:1\n\t" \
  "ds_read_b64_tr_b8 v[232:233], v249 offset:18688\n\t" \
  "ds_read_b64_tr_b8 v[234:235], v249 offset:19712\n\t" \
  "v_exp_f32 v220, v220\n\t" \
  "v_exp_f32 v221, v221\n\t" \
  "v_exp_f32 v222, v222\n\t" \
  "v_exp_f32 v223, v223\n\t" \
  "v_mul_f32 v32, s48, v32\n\t" \
  "v_mfma_f32_16x16x128_f8f6f4 v[104:107], v[184:191], v[240:247], v[104:107] blgp:1\n\t" \
  "ds_read_b64_tr_b8 v[236:237], v249 offset:26880\n\t" \
  "ds_read_b64_tr_b8 v[238:239], v249 offset:27904\n\t" \
  "v_mul_f32 v33, s48, v33\n\t" \
  "v_mul_f32 v34, s48, v34\n\t" \
  "v_mul_f32 v35, s48, v35\n\t" \
  "v_mul_f32 v36, s48, v36\n\t" \
  "v_mul_f32 v37, s48, v37\n\t" \
  "v_mul_f32 v38, s48, v38\n\t" \
  "v_mul_f32 v39, s48, v39\n\t" \
  "v_add_f32 v216, 1.0, v216\n\t" \
  "v_add_f32 v217, 1.0, v217\n\t" \
  "s_waitcnt lgkmcnt(8)\n\t" \
  "v_mfma_f32_16x16x128_f8f6f4 v[76:79], v[176:183], v[208:215], v[76:79] blgp:1\n\t" \
  "ds_read_b64_tr_b8 v[240:241], v250 offset:18688\n\t" \
  "ds_read_b64_tr_b8 v[242:243], v250 offset:19712\n\t" \
  "v_add_f32 v218, 1.0, v218\n\t" \
  "v_add_f32 v219, 1.0, v219\n\t" \
  "v_add_f32 v220, 1.0, v220\n\t" \
  "v_add_f32 v221, 1.0, v221\n\t" \
  "v_add_f32 v222, 1.0, v222\n\t" \
  "v_add_f32 v223, 1.0, v223\n\t" \
  "v_rcp_f32 v216, v216\n\t" \
  "v_rcp_f32 v217, v217\n\t" \
  "v_mfma_f32_16x16x128_f8f6f4 v[108:111], v[184:191], v[208:215], v[108:111] blgp:1\n\t" \
  "ds_read_b64_tr_b8 v[244:245], v250 offset:26880\n\t" \
  "ds_read_b64_tr_b8 v[246:247], v250 offset:27904\n\t" \
  "v_rcp_f32 v218, v218\n\t" \
  "v_rcp_f32 v219, v219\n\t" \
  "v_rcp_f32 v220, v220\n\t" \
  "v_rcp_f32 v221, v221\n\t" \
  "v_rcp_f32 v222, v222\n\t" \
  "s_waitcnt lgkmcnt(8)\n\t" \
  "v_mfma_f32_16x16x128_f8f6f4 v[80:83], v[176:183], v[224:231], v[80:83] blgp:1\n\t" \
  "ds_read_b64_tr_b8 v[208:209], v251 offset:18688\n\t" \
  "ds_read_b64_tr_b8 v[210:211], v251 offset:19712\n\t" \
  "v_rcp_f32 v223, v223\n\t" \
  "s_nop 0\n\t" \
  "v_mul_f32 v32, v32, v216\n\t" \
  "v_mul_f32 v33, v33, v217\n\t" \
  "v_mul_f32 v34, v34, v218\n\t" \
  "v_mul_f32 v35, v35, v219\n\t" \
  "v_mul_f32 v36, v36, v220\n\t" \
  "v_mul_f32 v37, v37, v221\n\t" \
  "v_mfma_f32_16x16x128_f8f6f4 v[112:115], v[184:191], v[224:231], v[112:115] blgp:1\n\t" \
  "ds_read_b64_tr_b8 v[212:213], v251 offset:26880\n\t" \
  "ds_read_b64_tr_b8 v[214:215], v251 offset:27904\n\t" \
  "v_mul_f32 v38, v38, v222\n\t" \
  "v_mul_f32 v39, v39, v223\n\t" \
  "s_nop 1\n\t" \
  "v_permlane16_swap_b32 v32, v36\n\t" \
  "v_permlane16_swap_b32 v33, v37\n\t" \
  "v_permlane16_swap_b32 v34, v38\n\t" \
  "v_permlane16_swap_b32 v35, v39\n\t" \
  "global_store_dword v254, v32, s[84:85] offset:0 " STPOL "\n\t" \
  "global_store_dword v254, v33, s[86:87] offset:0 " STPOL "\n\t" \
  "s_waitcnt lgkmcnt(8)\n\t" \
  "v_mfma_f32_16x16x128_f8f6f4 v[84:87], v[176:183], v[232:239], v[84:87] blgp:1\n\t" \
  "ds_read_b64_tr_b8 v[224:225], v248 offset:34816\n\t" \
  "ds_read_b64_tr_b8 v[226:227], v248 offset:35840\n\t" \
  "global_store_dword v254, v34, s[88:89] offset:0 " STPOL "\n\t" \
  "global_store_dword v254, v35, s[90:91] offset:0 " STPOL "\n\t" \
  "global_store_dword v255, v36, s[84:85] offset:0 " STPOL "\n\t" \
  "global_store_dword v255, v37, s[86:87] offset:0 " STPOL "\n\t" \
  "global_store_dword v255, v38, s[88:89] offset:0 " STPOL "\n\t" \
  "global_store_dword v255, v39, s[90:91] offset:0 " STPOL "\n\t" \
  "v_mul_f32 v216, s49, v8\n\t" \
  "v_mul_f32 v217, s49, v9\n\t" \
  "v_mul_f32 v218, s49, v10\n\t" \
  "v_mfma_f32_16x16x128_f8f6f4 v[116:119], v[184:191], v[232:239], v[116:119] blgp:1\n\t" \
  "ds_read_b64_tr_b8 v[228:229], v248 offset:43008\n\t" \
  "ds_read_b64_tr_b8 v[230:231], v248 offset:44032\n\t" \
  "v_mul_f32 v219, s49, v11\n\t" \
  "v_mul_f32 v220, s49, v12\n\t" \
  "v_mul_f32 v221, s49, v13\n\t" \
  "v_mul_f32 v222, s49, v14\n\t" \
  "v_mul_f32 v223, s49, v15\n\t" \
  "v_min_f32 v216, s92, v216\n\t" \
  "v_min_f32 v217, s92, v217\n\t" \
  "v_min_f32 v218, s92, v218\n\t" \
  "v_min_f32 v219, s92, v219\n\t" \
  "s_waitcnt lgkmcnt(8)\n\t" \
  "v_mfma_f32_16x16x128_f8f6f4 v[88:91], v[176:183], v[240:247], v[88:91] blgp:1\n\t" \
  "ds_read_b64_tr_b8 v[232:233], v249 offset:34816\n\t" \
  "ds_read_b64_tr_b8 v[234:235], v249 offset:35840\n\t" \
  "v_min_f32 v220, s92, v220\n\t" \
  "v_min_f32 v221, s92, v221\n\t" \
  "v_min_f32 v222, s92, v222\n\t" \
  "v_min_f32 v223, s92, v223\n\t" \
  "v_exp_f32 v216, v216\n\t" \
  "v_exp_f32 v217, v217\n\t" \
  "v_exp_f32 v218, v218\n\t" \
  "v_mfma_f32_16x16x128_f8f6f4 v[120:123], v[184:191], v[240:247], v[120:123] blgp:1\n\t" \
  "ds_read_b64_tr_b8 v[236:237], v249 offset:43008\n\t" \
  "ds_read_b64_tr_b8 v[238:239], v249 offset:44032\n\t" \
  "v_exp_f32 v219, v219\n\t" \
  "v_exp_f32 v220, v220\n\t" \
  "v_exp_f32 v221, v221\n\t" \
  "v_exp_f32 v222, v222\n\t" \
  "v_exp_f32 v223, v223\n\t" \
  "s_waitcnt lgkmcnt(8)\n\t" \
  "v_mfma_f32_16x16x128_f8f6f4 v[92:95], v[176:183], v[208:215], v[92:95] blgp:1\n\t" \
  "ds_read_b64_tr_b8 v[240:241], v250 offset:34816\n\t" \
  "ds_read_b64_tr_b8 v[242:243], v250 offset:35840\n\t" \
  "v_mul_f32 v8, s48, v8\n\t" \
  "v_mul_f32 v9, s48, v9\n\t" \
  "v_mul_f32 v10, s48, v10\n\t" \
  "v_mul_f32 v11, s48, v11\n\t" \
  "v_mul_f32 v12, s48, v12\n\t" \
  "v_mul_f32 v13, s48, v13\n\t" \
  "v_mul_f32 v14, s48, v14\n\t" \
  "v_mul_f32 v15, s48, v15\n\t" \
  "v_add_f32 v216, 1.0, v216\n\t" \
  "v_mfma_f32_16x16x128_f8f6f4 v[124:127], v[184:191], v[208:215], v[124:127] blgp:1\n\t" \
  "ds_read_b64_tr_b8 v[244:245], v250 offset:43008\n\t" \
  "ds_read_b64_tr_b8 v[246:247], v250 offset:44032\n\t" \
  "v_add_f32 v217, 1.0, v217\n\t" \
  "v_add_f32 v218, 1.0, v218\n\t" \
  "v_add_f32 v219, 1.0, v219\n\t" \
  "v_add_f32 v220, 1.0, v220\n\t" \
  "v_add_f32 v221, 1.0, v221\n\t" \
  "v_add_f32 v222, 1.0, v222\n\t" \
  "v_add_f32 v223, 1.0, v223\n\t" \
  "v_rcp_f32 v216, v216\n\t" \
  "s_waitcnt vmcnt(24)\n\t" \
  "s_waitcnt lgkmcnt(8)\n\t" \
  "v_mfma_f32_16x16x128_f8f6f4 v[64:67], v[192:199], v[224:231], v[64:67] blgp:1\n\t" \
  "ds_read_b64_tr_b8 v[208:209], v251 offset:34816\n\t" \
  "ds_read_b64_tr_b8 v[210:211], v251 offset:35840\n\t" \
  "v_rcp_f32 v217, v217\n\t" \
  "v_rcp_f32 v218, v218\n\t" \
  "v_rcp_f32 v219, v219\n\t" \
  "v_rcp_f32 v220, v220\n\t" \
  "v_rcp_f32 v221, v221\n\t" \
  "v_mfma_f32_16x16x128_f8f6f4 v[96:99], v[200:207], v[224:231], v[96:99] blgp:1\n\t" \
  "ds_read_b64_tr_b8 v[212:213], v251 offset:43008\n\t" \
  "ds_read_b64_tr_b8 v[214:215], v251 offset:44032\n\t" \
  "v_rcp_f32 v222, v222\n\t" \
  "v_rcp_f32 v223, v223\n\t" \
  "s_nop 0\n\t" \
  "v_mul_f32 v8, v8, v216\n\t" \
  "v_mul_f32 v9, v9, v217\n\t" \
  "v_mul_f32 v10, v10, v218\n\t" \
  "v_mul_f32 v11, v11, v219\n\t" \
  "s_waitcnt lgkmcnt(8)\n\t" \
  "v_mfma_f32_16x16x128_f8f6f4 v[68:71], v[192:199], v[232:239], v[68:71] blgp:1\n\t" \
  "ds_read_b64_tr_b8 v[224:225], v248 offset:35072\n\t" \
  "ds_read_b64_tr_b8 v[226:227], v248 offset:36096\n\t" \
  "v_mul_f32 v12, v12, v220\n\t" \
  "v_mul_f32 v13, v13, v221\n\t" \
  "v_mul_f32 v14, v14, v222\n\t" \
  "v_mul_f32 v15, v15, v223\n\t" \
  "s_nop 1\n\t" \
  "v_permlane16_swap_b32 v8, v12\n\t" \
  "v_permlane16_swap_b32 v9, v13\n\t" \
  "v_permlane16_swap_b32 v10, v14\n\t" \
  "v_permlane16_swap_b32 v11, v15\n\t" \
  "v_mfma_f32_16x16x128_f8f6f4 v[100:103], v[200:207], v[232:239], v[100:103] blgp:1\n\t" \
  "ds_read_b64_tr_b8 v[228:229], v248 offset:43264\n\t" \
  "ds_read_b64_tr_b8 v[230:231], v248 offset:44288\n\t" \
  "global_store_dword v254, v8, s[76:77] offset:128 " STPOL "\n\t" \
  "global_store_dword v254, v9, s[78:79] offset:128 " STPOL "\n\t" \
  "global_store_dword v254, v10, s[80:81] offset:128 " STPOL "\n\t" \
  "global_store_dword v254, v11, s[82:83] offset:128 " STPOL "\n\t" \
  "global_store_dword v255, v12, s[76:77] offset:128 " STPOL "\n\t" \
  "global_store_dword v255, v13, s[78:79] offset:128 " STPOL "\n\t" \
  "global_store_dword v255, v14, s[80:81] offset:128 " STPOL "\n\t" \
  "global_store_dword v255, v15, s[82:83] offset:128 " STPOL "\n\t" \
  "v_mul_f32 v216, s49, v40\n\t" \
  "s_waitcnt lgkmcnt(8)\n\t" \
  "v_mfma_f32_16x16x128_f8f6f4 v[72:75], v[192:199], v[240:247], v[72:75] blgp:1\n\t" \
  "ds_read_b64_tr_b8 v[232:233], v249 offset:35072\n\t" \
  "ds_read_b64_tr_b8 v[234:235], v249 offset:36096\n\t" \
  "v_mul_f32 v217, s49, v41\n\t" \
  "v_mul_f32 v218, s49, v42\n\t" \
  "v_mul_f32 v219, s49, v43\n\t" \
  "v_mul_f32 v220, s49, v44\n\t" \
  "v_mul_f32 v221, s49, v45\n\t" \
  "v_mul_f32 v222, s49, v46\n\t" \
  "v_mul_f32 v223, s49, v47\n\t" \
  "v_min_f32 v216, s92, v216\n\t" \
  "v_min_f32 v217, s92, v217\n\t" \
  "v_mfma_f32_16x16x128_f8f6f4 v[104:107], v[200:207], v[240:247], v[104:107] blgp:1\n\t" \
  "ds_read_b64_tr_b8 v[236:237], v249 offset:43264\n\t" \
  "ds_read_b64_tr_b8 v[238:239], v249 offset:44288\n\t" \
  "v_min_f32 v218, s92, v218\n\t" \
  "v_min_f32 v219, s92, v219\n\t" \
  "v_min_f32 v220, s92, v220\n\t" \
  "v_min_f32 v221, s92, v221\n\t" \
  "v_min_f32 v222, s92, v222\n\t" \
  "v_min_f32 v223, s92, v223\n\t" \
  "v_exp_f32 v216, v216\n\t" \
  "v_exp_f32 v217, v217\n\t" \
  "s_waitcnt lgkmcnt(8)\n\t" \
  "v_mfma_f32_16x16x128_f8f6f4 v[76:79], v[192:199], v[208:215], v[76:79] blgp:1\n\t" \
  "ds_read_b64_tr_b8 v[240:241], v250 offset:35072\n\t" \
  "ds_read_b64_tr_b8 v[242:243], v250 offset:36096\n\t" \
  "v_exp_f32 v218, v218\n\t" \
  "v_exp_f32 v219, v219\n\t" \
  "v_exp_f32 v220, v220\n\t" \
  "v_exp_f32 v221, v221\n\t" \
  "v_exp_f32 v222, v222\n\t" \
  "v_mfma_f32_16x16x128_f8f6f4 v[108:111], v[200:207], v[208:215], v[108:111] blgp:1\n\t" \
  "ds_read_b64_tr_b8 v[244:245], v250 offset:43264\n\t" \
  "ds_read_b64_tr_b8 v[246:247], v250 offset:44288\n\t" \
  "v_exp_f32 v223, v223\n\t" \
  "v_mul_f32 v40, s48, v40\n\t" \
  "v_mul_f32 v41, s48, v41\n\t" \
  "v_mul_f32 v42, s48, v42\n\t" \
  "v_mul_f32 v43, s48, v43\n\t" \
  "v_mul_f32 v44, s48, v44\n\t" \
  "v_mul_f32 v45, s48, v45\n\t" \
  "v_mul_f32 v46, s48, v46\n\t" \
  "s_waitcnt lgkmcnt(8)\n\t" \
  "v_mfma_f32_16x16x128_f8f6f4 v[80:83], v[192:199], v[224:231], v[80:83] blgp:1\n\t" \
  "ds_read_b64_tr_b8 v[208:209], v251 offset:35072\n\t" \
  "ds_read_b64_tr_b8 v[210:211], v251 offset:36096\n\t" \
  "v_mul_f32 v47, s48, v47\n\t" \
  "v_add_f32 v216, 1.0, v216\n\t" \
  "v_add_f32 v217, 1.0, v217\n\t" \
  "v_add_f32 v218, 1.0, v218\n\t" \
  "v_add_f32 v219, 1.0, v219\n\t" \
  "v_add_f32 v220, 1.0, v220\n\t" \
  "v_add_f32 v221, 1.0, v221\n\t" \
  "v_add_f32 v222, 1.0, v222\n\t" \
  "v_add_f32 v223, 1.0, v223\n\t" \
  "v_mfma_f32_16x16x128_f8f6f4 v[112:115], v[200:207], v[224:231], v[112:115] blgp:1\n\t" \
  "ds_read_b64_tr_b8 v[212:213], v251 offset:43264\n\t" \
  "ds_read_b64_tr_b8 v[214:215], v251 offset:44288\n\t" \
  "v_rcp_f32 v216, v216\n\t" \
  "v_rcp_f32 v217, v217\n\t" \
  "v_rcp_f32 v218, v218\n\t" \
  "v_rcp_f32 v219, v219\n\t" \
  "v_rcp_f32 v220, v220\n\t" \
  "s_waitcnt lgkmcnt(8)\n\t" \
  "v_mfma_f32_16x16x128_f8f6f4 v[84:87], v[192:199], v[232:239], v[84:87] blgp:1\n\t" \
  "v_add_u32 v248, 0xc000, v248\n\t" \
  "ds_read_b64_tr_b8 v[224:225], v248 offset:2048\n\t" \
  "ds_read_b64_tr_b8 v[226:227], v248 offset:3072\n\t" \
  "v_rcp_f32 v221, v221\n\t" \
  "v_rcp_f32 v222, v222\n\t" \
  "v_rcp_f32 v223, v223\n\t" \
  "s_nop 0\n\t" \
  "v_mul_f32 v40, v40, v216\n\t" \
  "v_mul_f32 v41, v41, v217\n\t" \
  "v_mfma_f32_16x16x128_f8f6f4 v[116:119], v[200:207], v[232:239], v[116:119] blgp:1\n\t" \
  "ds_read_b64_tr_b8 v[228:229], v248 offset:10240\n\t" \
  "ds_read_b64_tr_b8 v[230:231], v248 offset:11264\n\t" \
  "v_mul_f32 v42, v42, v218\n\t" \
  "v_mul_f32 v43, v43, v219\n\t" \
  "v_mul_f32 v44, v44, v220\n\t" \
  "v_mul_f32 v45, v45, v221\n\t" \
  "v_mul_f32 v46, v46, v222\n\t" \
  "v_mul_f32 v47, v47, v223\n\t" \
  "s_nop 1\n\t" \
  "v_permlane16_swap_b32 v40, v44\n\t" \
  "v_permlane16_swap_b32 v41, v45\n\t" \
  "s_waitcnt lgkmcnt(8)\n\t" \
  "v_mfma_f32_16x16x128_f8f6f4 v[88:91], v[192:199], v[240:247], v[88:91] blgp:1\n\t" \
  "v_add_u32 v249, 0xc000, v249\n\t" \
  "ds_read_b64_tr_b8 v[232:233], v249 offset:2048\n\t" \
  "ds_read_b64_tr_b8 v[234:235], v249 offset:3072\n\t" \
  "v_permlane16_swap_b32 v42, v46\n\t" \
  "v_permlane16_swap_b32 v43, v47\n\t" \
  "global_store_dword v254, v40, s[84:85] offset:128 " STPOL "\n\t" \
  "global_store_dword v254, v41, s[86:87] offset:128 " STPOL "\n\t" \
  "global_store_dword v254, v42, s[88:89] offset:128 " STPOL "\n\t" \
  "global_store_dword v254, v43, s[90:91] offset:128 " STPOL "\n\t" \
  "global_store_dword v255, v44, s[84:85] offset:128 " STPOL "\n\t" \
  "global_store_dword v255, v45, s[86:87] offset:128 " STPOL "\n\t" \
  "global_store_dword v255, v46, s[88:89] offset:128 " STPOL "\n\t" \
  "v_mfma_f32_16x16x128_f8f6f4 v[120:123], v[200:207], v[240:247], v[120:123] blgp:1\n\t" \
  "ds_read_b64_tr_b8 v[236:237], v249 offset:10240\n\t" \
  "ds_read_b64_tr_b8 v[238:239], v249 offset:11264\n\t" \
  "global_store_dword v255, v47, s[90:91] offset:128 " STPOL "\n\t" \
  "v_mul_f32 v216, s49, v16\n\t" \
  "v_mul_f32 v217, s49, v17\n\t" \
  "v_mul_f32 v218, s49, v18\n\t" \
  "v_mul_f32 v219, s49, v19\n\t" \
  "v_mul_f32 v220, s49, v20\n\t" \
  "v_mul_f32 v221, s49, v21\n\t" \
  "v_mul_f32 v222, s49, v22\n\t" \
  "v_mul_f32 v223, s49, v23\n\t" \
  "s_waitcnt lgkmcnt(8)\n\t" \
  "v_mfma_f32_16x16x128_f8f6f4 v[92:95], v[192:199], v[208:215], v[92:95] blgp:1\n\t" \
  "v_add_u32 v250, 0xc000, v250\n\t" \
  "ds_read_b64_tr_b8 v[240:241], v250 offset:2048\n\t" \
  "ds_read_b64_tr_b8 v[242:243], v250 offset:3072\n\t" \
  "v_min_f32 v216, s92, v216\n\t" \
  "v_min_f32 v217, s92, v217\n\t" \
  "v_min_f32 v218, s92, v218\n\t" \
  "v_min_f32 v219, s92, v219\n\t" \
  "v_min_f32 v220, s92, v220\n\t" \
  "v_min_f32 v221, s92, v221\n\t" \
  "v_min_f32 v222, s92, v222\n\t" \
  "v_min_f32 v223, s92, v223\n\t" \
  "v_exp_f32 v216, v216\n\t" \
  "v_mfma_f32_16x16x128_f8f6f4 v[124:127], v[200:207], v[208:215], v[124:127] blgp:1\n\t" \
  "ds_read_b64_tr_b8 v[244:245], v250 offset:10240\n\t" \
  "ds_read_b64_tr_b8 v[246:247], v250 offset:11264\n\t" \
  "v_exp_f32 v217, v217\n\t" \
  "v_exp_f32 v218, v218\n\t" \
  "v_exp_f32 v219, v219\n\t" \
  "v_exp_f32 v220, v220\n\t" \
  "v_exp_f32 v221, v221\n\t" \
  "s_waitcnt vmcnt(36)\n\t" \
  "s_waitcnt lgkmcnt(8)\n\t" \
  "v_mfma_f32_16x16x128_f8f6f4 v[64:67], v[128:135], v[224:231], v[64:67] blgp:1\n\t" \
  "v_add_u32 v251, 0xc000, v251\n\t" \
  "ds_read_b64_tr_b8 v[208:209], v251 offset:2048\n\t" \
  "ds_read_b64_tr_b8 v[210:211], v251 offset:3072\n\t" \
  "v_exp_f32 v222, v222\n\t" \
  "v_exp_f32 v223, v223\n\t" \
  "v_mul_f32 v16, s48, v16\n\t" \
  "v_mul_f32 v17, s48, v17\n\t" \
  "v_mul_f32 v18, s48, v18\n\t" \
  "v_mul_f32 v19, s48, v19\n\t" \
  "v_mul_f32 v20, s48, v20\n\t" \
  "v_mfma_f32_16x16x128_f8f6f4 v[96:99], v[136:143], v[224:231], v[96:99] blgp:1\n\t" \
  "ds_read_b64_tr_b8 v[212:213], v251 offset:10240\n\t" \
  "ds_read_b64_tr_b8 v[214:215], v251 offset:11264\n\t" \
  "v_mul_f32 v21, s48, v21\n\t" \
  "v_mul_f32 v22, s48, v22\n\t" \
  "v_mul_f32 v23, s48, v23\n\t" \
  "v_add_f32 v216, 1.0, v216\n\t" \
  "v_add_f32 v217, 1.0, v217\n\t" \
  "v_add_f32 v218, 1.0, v218\n\t" \
  "v_add_f32 v219, 1.0, v219\n\t" \
  "v_add_f32 v220, 1.0, v220\n\t" \
  "v_add_f32 v221, 1.0, v221\n\t" \
  "s_waitcnt lgkmcnt(8)\n\t" \
  "v_mfma_f32_16x16x128_f8f6f4 v[68:71], v[128:135], v[232:239], v[68:71] blgp:1\n\t" \
  "ds_read_b64_tr_b8 v[224:225], v248 offset:2304\n\t" \
  "ds_read_b64_tr_b8 v[226:227], v248 offset:3328\n\t" \
  "v_add_f32 v222, 1.0, v222\n\t" \
  "v_add_f32 v223, 1.0, v223\n\t" \
  "v_rcp_f32 v216, v216\n\t" \
  "v_rcp_f32 v217, v217\n\t" \
  "v_rcp_f32 v218, v218\n\t" \
  "v_rcp_f32 v219, v219\n\t" \
  "v_mfma_f32_16x16x128_f8f6f4 v[100:103], v[136:143], v[232:239], v[100:103] blgp:1\n\t" \
  "ds_read_b64_tr_b8 v[228:229], v248 offset:10496\n\t" \
  "ds_read_b64_tr_b8 v[230:231], v248 offset:11520\n\t" \
  "v_rcp_f32 v220, v220\n\t" \
  "v_rcp_f32 v221, v221\n\t" \
  "v_rcp_f32 v222, v222\n\t" \
  "v_rcp_f32 v223, v223\n\t" \
  "s_nop 0\n\t" \
  "s_waitcnt lgkmcnt(8)\n\t" \
  "v_mfma_f32_16x16x128_f8f6f4 v[72:75], v[128:135], v[240:247], v[72:75] blgp:1\n\t" \
  "ds_read_b64_tr_b8 v[232:233], v249 offset:2304\n\t" \
  "ds_read_b64_tr_b8 v[234:235], v249 offset:3328\n\t" \
  "v_mul_f32 v16, v16, v216\n\t" \
  "v_mul_f32 v17, v17, v217\n\t" \
  "v_mul_f32 v18, v18, v218\n\t" \
  "v_mul_f32 v19, v19, v219\n\t" \
  "v_mul_f32 v20, v20, v220\n\t" \
  "v_mul_f32 v21, v21, v221\n\t" \
  "v_mul_f32 v22, v22, v222\n\t" \
  "v_mul_f32 v23, v23, v223\n\t" \
  "s_nop 1\n\t" \
  "v_mfma_f32_16x16x128_f8f6f4 v[104:107], v[136:143], v[240:247], v[104:107] blgp:1\n\t" \
  "ds_read_b64_tr_b8 v[236:237], v249 offset:10496\n\t" \
  "ds_read_b64_tr_b8 v[238:239], v249 offset:11520\n\t" \
  "v_permlane16_swap_b32 v16, v20\n\t" \
  "v_permlane16_swap_b32 v17, v21\n\t" \
  "v_permlane16_swap_b32 v18, v22\n\t" \
  "v_permlane16_swap_b32 v19, v23\n\t" \
  "global_store_dword v254, v16, s[76:77] offset:256 " STPOL "\n\t" \
  "global_store_dword v254, v17, s[78:79] offset:256 " STPOL "\n\t" \
  "global_store_dword v254, v18, s[80:81] offset:256 " STPOL "\n\t" \
  "global_store_dword v254, v19, s[82:83] offset:256 " STPOL "\n\t" \
  "global_store_dword v255, v20, s[76:77] offset:256 " STPOL "\n\t" \
  "s_waitcnt lgkmcnt(8)\n\t" \
  "v_mfma_f32_16x16x128_f8f6f4 v[76:79], v[128:135], v[208:215], v[76:79] blgp:1\n\t" \
  "ds_read_b64_tr_b8 v[240:241], v250 offset:2304\n\t" \
  "ds_read_b64_tr_b8 v[242:243], v250 offset:3328\n\t" \
  "global_store_dword v255, v21, s[78:79] offset:256 " STPOL "\n\t" \
  "global_store_dword v255, v22, s[80:81] offset:256 " STPOL "\n\t" \
  "global_store_dword v255, v23, s[82:83] offset:256 " STPOL "\n\t" \
  "v_mul_f32 v216, s49, v48\n\t" \
  "v_mul_f32 v217, s49, v49\n\t" \
  "v_mul_f32 v218, s49, v50\n\t" \
  "v_mul_f32 v219, s49, v51\n\t" \
  "v_mul_f32 v220, s49, v52\n\t" \
  "v_mul_f32 v221, s49, v53\n\t" \
  "v_mfma_f32_16x16x128_f8f6f4 v[108:111], v[136:143], v[208:215], v[108:111] blgp:1\n\t" \
  "ds_read_b64_tr_b8 v[244:245], v250 offset:10496\n\t" \
  "ds_read_b64_tr_b8 v[246:247], v250 offset:11520\n\t" \
  "v_mul_f32 v222, s49, v54\n\t" \
  "v_mul_f32 v223, s49, v55\n\t" \
  "v_min_f32 v216, s92, v216\n\t" \
  "v_min_f32 v217, s92, v217\n\t" \
  "v_min_f32 v218, s92, v218\n\t" \
  "v_min_f32 v219, s92, v219\n\t" \
  "v_min_f32 v220, s92, v220\n\t" \
  "v_min_f32 v221, s92, v221\n\t" \
  "v_min_f32 v222, s92, v222\n\t" \
  "s_waitcnt lgkmcnt(8)\n\t" \
  "v_mfma_f32_16x16x128_f8f6f4 v[80:83], v[128:135], v[224:231], v[80:83] blgp:1\n\t" \
  "ds_read_b64_tr_b8 v[208:209], v251 offset:2304\n\t" \
  "ds_read_b64_tr_b8 v[210:211], v251 offset:3328\n\t" \
  "v_min_f32 v223, s92, v223\n\t" \
  "v_exp_f32 v216, v216\n\t" \
  "v_exp_f32 v217, v217\n\t" \
  "v_exp_f32 v218, v218\n\t" \
  "v_exp_f32 v219, v219\n\t" \
  "v_mfma_f32_16x16x128_f8f6f4 v[112:115], v[136:143], v[224:231], v[112:115] blgp:1\n\t" \
  "ds_read_b64_tr_b8 v[212:213], v251 offset:10496\n\t" \
  "ds_read_b64_tr_b8 v[214:215], v251 offset:11520\n\t" \
  "v_exp_f32 v220, v220\n\t" \
  "v_exp_f32 v221, v221\n\t" \
  "v_exp_f32 v222, v222\n\t" \
  "v_exp_f32 v223, v223\n\t" \
  "v_mul_f32 v48, s48, v48\n\t" \
  "s_waitcnt lgkmcnt(8)\n\t" \
  "v_mfma_f32_16x16x128_f8f6f4 v[84:87], v[128:135], v[232:239], v[84:87] blgp:1\n\t" \
  "ds_read_b64_tr_b8 v[224:225], v248 offset:18432\n\t" \
  "ds_read_b64_tr_b8 v[226:227], v248 offset:19456\n\t" \
  "v_mul_f32 v49, s48, v49\n\t" \
  "v_mul_f32 v50, s48, v50\n\t" \
  "v_mul_f32 v51, s48, v51\n\t" \
  "v_mul_f32 v52, s48, v52\n\t" \
  "v_mul_f32 v53, s48, v53\n\t" \
  "v_mul_f32 v54, s48, v54\n\t" \
  "v_mul_f32 v55, s48, v55\n\t" \
  "v_add_f32 v216, 1.0, v216\n\t" \
  "v_add_f32 v217, 1.0, v217\n\t" \
  "v_mfma_f32_16x16x128_f8f6f4 v[116:119], v[136:143], v[232:239], v[116:119] blgp:1\n\t" \
  "ds_read_b64_tr_b8 v[228:229], v248 offset:26624\n\t" \
  "ds_read_b64_tr_b8 v[230:231], v248 offset:27648\n\t" \
  "v_add_f32 v218, 1.0, v218\n\t" \
  "v_add_f32 v219, 1.0, v219\n\t" \
  "v_add_f32 v220, 1.0, v220\n\t" \
  "v_add_f32 v221, 1.0, v221\n\t" \
  "v_add_f32 v222, 1.0, v222\n\t" \
  "v_add_f32 v223, 1.0, v223\n\t" \
  "v_rcp_f32 v216, v216\n\t" \
  "v_rcp_f32 v217, v217\n\t" \
  "s_waitcnt lgkmcnt(8)\n\t" \
  "v_mfma_f32_16x16x128_f8f6f4 v[88:91], v[128:135], v[240:247], v[88:91] blgp:1\n\t" \
  "ds_read_b64_tr_b8 v[232:233], v249 offset:18432\n\t" \
  "ds_read_b64_tr_b8 v[234:235], v249 offset:19456\n\t" \
  "v_rcp_f32 v218, v218\n\t" \
  "v_rcp_f32 v219, v219\n\t" \
  "v_rcp_f32 v220, v220\n\t" \
  "v_rcp_f32 v221, v221\n\t" \
  "v_rcp_f32 v222, v222\n\t" \
  "v_mfma_f32_16x16x128_f8f6f4 v[120:123], v[136:143], v[240:247], v[120:123] blgp:1\n\t" \
  "ds_read_b64_tr_b8 v[236:237], v249 offset:26624\n\t" \
  "ds_read_b64_tr_b8 v[238:239], v249 offset:27648\n\t" \
  "v_rcp_f32 v223, v223\n\t" \
  "s_nop 0\n\t" \
  "v_mul_f32 v48, v48, v216\n\t" \
  "v_mul_f32 v49, v49, v217\n\t" \
  "v_mul_f32 v50, v50, v218\n\t" \
  "v_mul_f32 v51, v51, v219\n\t" \
  "v_mul_f32 v52, v52, v220\n\t" \
  "v_mul_f32 v53, v53, v221\n\t" \
  "s_waitcnt lgkmcnt(8)\n\t" \
  "v_mfma_f32_16x16x128_f8f6f4 v[92:95], v[128:135], v[208:215], v[92:95] blgp:1\n\t" \
  "ds_read_b64_tr_b8 v[240:241], v250 offset:18432\n\t" \
  "ds_read_b64_tr_b8 v[242:243], v250 offset:19456\n\t" \
  "v_mul_f32 v54, v54, v222\n\t" \
  "v_mul_f32 v55, v55, v223\n\t" \
  "s_nop 1\n\t" \
  "v_permlane16_swap_b32 v48, v52\n\t" \
  "v_permlane16_swap_b32 v49, v53\n\t" \
  "v_permlane16_swap_b32 v50, v54\n\t" \
  "v_permlane16_swap_b32 v51, v55\n\t" \
  "global_store_dword v254, v48, s[84:85] offset:256 " STPOL "\n\t" \
  "global_store_dword v254, v49, s[86:87] offset:256 " STPOL "\n\t" \
  "v_mfma_f32_16x16x128_f8f6f4 v[124:127], v[136:143], v[208:215], v[124:127] blgp:1\n\t" \
  "ds_read_b64_tr_b8 v[244:245], v250 offset:26624\n\t" \
  "ds_read_b64_tr_b8 v[246:247], v250 offset:27648\n\t" \
  "global_store_dword v254, v50, s[88:89] offset:256 " STPOL "\n\t" \
  "global_store_dword v254, v51, s[90:91] offset:256 " STPOL "\n\t" \
  "global_store_dword v255, v52, s[84:85] offset:256 " STPOL "\n\t" \
  "global_store_dword v255, v53, s[86:87] offset:256 " STPOL "\n\t" \
  "global_store_dword v255, v54, s[88:89] offset:256 " STPOL "\n\t" \
  "global_store_dword v255, v55, s[90:91] offset:256 " STPOL "\n\t" \
  "v_mul_f32 v216, s49, v24\n\t" \
  "v_mul_f32 v217, s49, v25\n\t" \
  "v_mul_f32 v218, s49, v26\n\t" \
  "s_waitcnt vmcnt(48)\n\t" \
  "s_waitcnt lgkmcnt(8)\n\t" \
  "v_mfma_f32_16x16x128_f8f6f4 v[64:67], v[160:167], v[224:231], v[64:67] blgp:1\n\t" \
  "ds_read_b64_tr_b8 v[208:209], v251 offset:18432\n\t" \
  "ds_read_b64_tr_b8 v[210:211], v251 offset:19456\n\t" \
  "v_mul_f32 v219, s49, v27\n\t" \
  "v_mul_f32 v220, s49, v28\n\t" \
  "v_mul_f32 v221, s49, v29\n\t" \
  "v_mul_f32 v222, s49, v30\n\t" \
  "v_mul_f32 v223, s49, v31\n\t" \
  "v_min_f32 v216, s92, v216\n\t" \
  "v_min_f32 v217, s92, v217\n\t" \
  "v_min_f32 v218, s92, v218\n\t" \
  "v_min_f32 v219, s92, v219\n\t" \
  "v_mfma_f32_16x16x128_f8f6f4 v[96:99], v[168:175], v[224:231], v[96:99] blgp:1\n\t" \
  "ds_read_b64_tr_b8 v[212:213], v251 offset:26624\n\t" \
  "ds_read_b64_tr_b8 v[214:215], v251 offset:27648\n\t" \
  "v_min_f32 v220, s92, v220\n\t" \
  "v_min_f32 v221, s92, v221\n\t" \
  "v_min_f32 v222, s92, v222\n\t" \
  "v_min_f32 v223, s92, v223\n\t" \
  "v_exp_f32 v216, v216\n\t" \
  "v_exp_f32 v217, v217\n\t" \
  "v_exp_f32 v218, v218\n\t" \
  "s_waitcnt lgkmcnt(8)\n\t" \
  "v_mfma_f32_16x16x128_f8f6f4 v[68:71], v[160:167], v[232:239], v[68:71] blgp:1\n\t" \
  "ds_read_b64_tr_b8 v[224:225], v248 offset:18688\n\t" \
  "ds_read_b64_tr_b8 v[226:227], v248 offset:19712\n\t" \
  "v_exp_f32 v219, v219\n\t" \
  "v_exp_f32 v220, v220\n\t" \
  "v_exp_f32 v221, v221\n\t" \
  "v_exp_f32 v222, v222\n\t" \
  "v_exp_f32 v223, v223\n\t" \
  "v_mfma_f32_16x16x128_f8f6f4 v[100:103], v[168:175], v[232:239], v[100:103] blgp:1\n\t" \
  "ds_read_b64_tr_b8 v[228:229], v248 offset:26880\n\t" \
  "ds_read_b64_tr_b8 v[230:231], v248 offset:27904\n\t" \
  "v_mul_f32 v24, s48, v24\n\t" \
  "v_mul_f32 v25, s48, v25\n\t" \
  "v_mul_f32 v26, s48, v26\n\t" \
  "v_mul_f32 v27, s48, v27\n\t" \
  "v_mul_f32 v28, s48, v28\n\t" \
  "v_mul_f32 v29, s48, v29\n\t" \
  "v_mul_f32 v30, s48, v30\n\t" \
  "v_mul_f32 v31, s48, v31\n\t" \
  "v_add_f32 v216, 1.0, v216\n\t" \
  "s_waitcnt lgkmcnt(8)\n\t" \
  "v_mfma_f32_16x16x128_f8f6f4 v[72:75], v[160:167], v[240:247], v[72:75] blgp:1\n\t" \
  "ds_read_b64_tr_b8 v[232:233], v249 offset:18688\n\t" \
  "ds_read_b64_tr_b8 v[234:235], v249 offset:19712\n\t" \
  "v_add_f32 v217, 1.0, v217\n\t" \
  "v_add_f32 v218, 1.0, v218\n\t" \
  "v_add_f32 v219, 1.0, v219\n\t" \
  "v_add_f32 v220, 1.0, v220\n\t" \
  "v_add_f32 v221, 1.0, v221\n\t" \
  "v_add_f32 v222, 1.0, v222\n\t" \
  "v_add_f32 v223, 1.0, v223\n\t" \
  "v_rcp_f32 v216, v216\n\t" \
  "v_mfma_f32_16x16x128_f8f6f4 v[104:107], v[168:175], v[240:247], v[104:107] blgp:1\n\t" \
  "ds_read_b64_tr_b8 v[236:237], v249 offset:26880\n\t" \
  "ds_read_b64_tr_b8 v[238:239], v249 offset:27904\n\t" \
  "v_rcp_f32 v217, v217\n\t" \
  "v_rcp_f32 v218, v218\n\t" \
  "v_rcp_f32 v219, v219\n\t" \
  "v_rcp_f32 v220, v220\n\t" \
  "v_rcp_f32 v221, v221\n\t" \
  "s_waitcnt lgkmcnt(8)\n\t" \
  "v_mfma_f32_16x16x128_f8f6f4 v[76:79], v[160:167], v[208:215], v[76:79] blgp:1\n\t" \
  "ds_read_b64_tr_b8 v[240:241], v250 offset:18688\n\t" \
  "ds_read_b64_tr_b8 v[242:243], v250 offset:19712\n\t" \
  "v_rcp_f32 v222, v222\n\t" \
  "v_rcp_f32 v223, v223\n\t" \
  "s_nop 0\n\t" \
  "v_mul_f32 v24, v24, v216\n\t" \
  "v_mul_f32 v25, v25, v217\n\t" \
  "v_mul_f32 v26, v26, v218\n\t" \
  "v_mul_f32 v27, v27, v219\n\t" \
  "v_mfma_f32_16x16x128_f8f6f4 v[108:111], v[168:175], v[208:215], v[108:111] blgp:1\n\t" \
  "ds_read_b64_tr_b8 v[244:245], v250 offset:26880\n\t" \
  "ds_read_b64_tr_b8 v[246:247], v250 offset:27904\n\t" \
  "v_mul_f32 v28, v28, v220\n\t" \
  "v_mul_f32 v29, v29, v221\n\t" \
  "v_mul_f32 v30, v30, v222\n\t" \
  "v_mul_f32 v31, v31, v223\n\t" \
  "s_nop 1\n\t" \
  "v_permlane16_swap_b32 v24, v28\n\t" \
  "v_permlane16_swap_b32 v25, v29\n\t" \
  "v_permlane16_swap_b32 v26, v30\n\t" \
  "v_permlane16_swap_b32 v27, v31\n\t" \
  "s_waitcnt lgkmcnt(8)\n\t" \
  "v_mfma_f32_16x16x128_f8f6f4 v[80:83], v[160:167], v[224:231], v[80:83] blgp:1\n\t" \
  "ds_read_b64_tr_b8 v[208:209], v251 offset:18688\n\t" \
  "ds_read_b64_tr_b8 v[210:211], v251 offset:19712\n\t" \
  "global_store_dword v254, v24, s[76:77] offset:384 " STPOL "\n\t" \
  "global_store_dword v254, v25, s[78:79] offset:384 " STPOL "\n\t" \
  "global_store_dword v254, v26, s[80:81] offset:384 " STPOL "\n\t" \
  "global_store_dword v254, v27, s[82:83] offset:384 " STPOL "\n\t" \
  "global_store_dword v255, v28, s[76:77] offset:384 " STPOL "\n\t" \
  "global_store_dword v255, v29, s[78:79] offset:384 " STPOL "\n\t" \
  "global_store_dword v255, v30, s[80:81] offset:384 " STPOL "\n\t" \
  "global_store_dword v255, v31, s[82:83] offset:384 " STPOL "\n\t" \
  "v_mul_f32 v216, s49, v56\n\t" \
  "v_mfma_f32_16x16x128_f8f6f4 v[112:115], v[168:175], v[224:231], v[112:115] blgp:1\n\t" \
  "ds_read_b64_tr_b8 v[212:213], v251 offset:26880\n\t" \
  "ds_read_b64_tr_b8 v[214:215], v251 offset:27904\n\t" \
  "v_mul_f32 v217, s49, v57\n\t" \
  "v_mul_f32 v218, s49, v58\n\t" \
  "v_mul_f32 v219, s49, v59\n\t" \
  "v_mul_f32 v220, s49, v60\n\t" \
  "v_mul_f32 v221, s49, v61\n\t" \
  "v_mul_f32 v222, s49, v62\n\t" \
  "v_mul_f32 v223, s49, v63\n\t" \
  "v_min_f32 v216, s92, v216\n\t" \
  "v_min_f32 v217, s92, v217\n\t" \
  "s_waitcnt lgkmcnt(8)\n\t" \
  "v_mfma_f32_16x16x128_f8f6f4 v[84:87], v[160:167], v[232:239], v[84:87] blgp:1\n\t" \
  "v_min_f32 v218, s92, v218\n\t" \
  "v_min_f32 v219, s92, v219\n\t" \
  "v_min_f32 v220, s92, v220\n\t" \
  "v_min_f32 v221, s92, v221\n\t" \
  "v_min_f32 v222, s92, v222\n\t" \
  "v_min_f32 v223, s92, v223\n\t" \
  "v_exp_f32 v216, v216\n\t" \
  "v_exp_f32 v217, v217\n\t" \
  "v_mfma_f32_16x16x128_f8f6f4 v[116:119], v[168:175], v[232:239], v[116:119] blgp:1\n\t" \
  "v_exp_f32 v218, v218\n\t" \
  "v_exp_f32 v219, v219\n\t" \
  "v_exp_f32 v220, v220\n\t" \
  "v_exp_f32 v221, v221\n\t" \
  "v_exp_f32 v222, v222\n\t" \
  "s_waitcnt lgkmcnt(4)\n\t" \
  "v_mfma_f32_16x16x128_f8f6f4 v[88:91], v[160:167], v[240:247], v[88:91] blgp:1\n\t" \
  "v_exp_f32 v223, v223\n\t" \
  "v_mul_f32 v56, s48, v56\n\t" \
  "v_mul_f32 v57, s48, v57\n\t" \
  "v_mul_f32 v58, s48, v58\n\t" \
  "v_mul_f32 v59, s48, v59\n\t" \
  "v_mul_f32 v60, s48, v60\n\t" \
  "v_mul_f32 v61, s48, v61\n\t" \
  "v_mul_f32 v62, s48, v62\n\t" \
  "v_mfma_f32_16x16x128_f8f6f4 v[120:123], v[168:175], v[240:247], v[120:123] blgp:1\n\t" \
  "v_mul_f32 v63, s48, v63\n\t" \
  "v_add_f32 v216, 1.0, v216\n\t" \
  "v_add_f32 v217, 1.0, v217\n\t" \
  "v_add_f32 v218, 1.0, v218\n\t" \
  "v_add_f32 v219, 1.0, v219\n\t" \
  "v_add_f32 v220, 1.0, v220\n\t" \
  "v_add_f32 v221, 1.0, v221\n\t" \
  "v_add_f32 v222, 1.0, v222\n\t" \
  "v_add_f32 v223, 1.0, v223\n\t" \
  "s_waitcnt lgkmcnt(0)\n\t" \
  "v_mfma_f32_16x16x128_f8f6f4 v[92:95], v[160:167], v[208:215], v[92:95] blgp:1\n\t" \
  "v_rcp_f32 v216, v216\n\t" \
  "v_rcp_f32 v217, v217\n\t" \
  "v_rcp_f32 v218, v218\n\t" \
  "v_rcp_f32 v219, v219\n\t" \
  "v_rcp_f32 v220, v220\n\t" \
  "v_mfma_f32_16x16x128_f8f6f4 v[124:127], v[168:175], v[208:215], v[124:127] blgp:1\n\t" \
  "v_rcp_f32 v221, v221\n\t" \
  "v_rcp_f32 v222, v222\n\t" \
  "v_rcp_f32 v223, v223\n\t" \
  "s_nop 0\n\t" \
  "v_mul_f32 v56, v56, v216\n\t" \
  "v_mul_f32 v57, v57, v217\n\t" \
  "" PREEPI "\n\t" \
  "v_mul_f32 v58, v58, v218\n\t" \
  "v_mul_f32 v59, v59, v219\n\t" \
  "v_mul_f32 v60, v60, v220\n\t" \
  "v_mul_f32 v61, v61, v221\n\t" \
  "v_mul_f32 v62, v62, v222\n\t" \
  "v_mul_f32 v63, v63, v223\n\t" \
  "s_nop 1\n\t" \
  "v_permlane16_swap_b32 v56, v60\n\t" \
  "v_permlane16_swap_b32 v57, v61\n\t" \
  "v_permlane16_swap_b32 v58, v62\n\t" \
  "v_permlane16_swap_b32 v59, v63\n\t" \
  "global_store_dword v254, v56, s[84:85] offset:384 " STPOL "\n\t" \
  "global_store_dword v254, v57, s[86:87] offset:384 " STPOL "\n\t" \
  "global_store_dword v254, v58, s[88:89] offset:384 " STPOL "\n\t" \
  "global_store_dword v254, v59, s[90:91] offset:384 " STPOL "\n\t" \
  "global_store_dword v255, v60, s[84:85] offset:384 " STPOL "\n\t" \
  "global_store_dword v255, v61, s[86:87] offset:384 " STPOL "\n\t" \
  "global_store_dword v255, v62, s[88:89] offset:384 " STPOL "\n\t" \
  "global_store_dword v255, v63, s[90:91] offset:384 " STPOL "\n\t" \
  "s_add_u32 s76, s76, 0x200000\n\t" \
  "s_addc_u32 s77, s77, 0\n\t" \
  "s_add_u32 s78, s78, 0x200000\n\t" \
  "s_addc_u32 s79, s79, 0\n\t" \
  "s_add_u32 s80, s80, 0x200000\n\t" \
  "s_addc_u32 s81, s81, 0\n\t" \
  "s_add_u32 s82, s82, 0x200000\n\t" \
  "s_addc_u32 s83, s83, 0\n\t" \
  "s_add_u32 s84, s84, 0x200000\n\t" \
  "s_addc_u32 s85, s85, 0\n\t" \
  "s_add_u32 s86, s86, 0x200000\n\t" \
  "s_addc_u32 s87, s87, 0\n\t" \
  "s_add_u32 s88, s88, 0x200000\n\t" \
  "s_addc_u32 s89, s89, 0\n\t" \
  "s_add_u32 s90, s90, 0x200000\n\t" \
  "s_addc_u32 s91, s91, 0\n\t" \
  "v_pk_mul_f32 v[128:129], v[64:65], s[96:97]\n\t" \
  "v_pk_mul_f32 v[130:131], v[66:67], s[96:97]\n\t" \
  "v_pk_mul_f32 v[132:133], v[68:69], s[96:97]\n\t" \
  "v_pk_mul_f32 v[134:135], v[70:71], s[96:97]\n\t" \
  "v_min_f32 v128, s92, v128\n\t" \
  "v_min_f32 v129, s92, v129\n\t" \
  "v_min_f32 v130, s92, v130\n\t" \
  "v_min_f32 v131, s92, v131\n\t" \
  "v_min_f32 v132, s92, v132\n\t" \
  "v_min_f32 v133, s92, v133\n\t" \
  "v_min_f32 v134, s92, v134\n\t" \
  "v_min_f32 v135, s92, v135\n\t" \
  "v_exp_f32 v128, v128\n\t" \
  "v_exp_f32 v129, v129\n\t" \
  "v_exp_f32 v130, v130\n\t" \
  "v_exp_f32 v131, v131\n\t" \
  "v_exp_f32 v132, v132\n\t" \
  "v_exp_f32 v133, v133\n\t" \
  "v_exp_f32 v134, v134\n\t" \
  "v_exp_f32 v135, v135\n\t" \
  "v_pk_mul_f32 v[64:65], v[64:65], s[94:95]\n\t" \
  "v_pk_mul_f32 v[66:67], v[66:67], s[94:95]\n\t" \
  "v_pk_mul_f32 v[68:69], v[68:69], s[94:95]\n\t" \
  "v_pk_mul_f32 v[70:71], v[70:71], s[94:95]\n\t" \
  "v_pk_add_f32 v[128:129], v[128:129], 1.0 op_sel_hi:[1,0]\n\t" \
  "v_pk_add_f32 v[130:131], v[130:131], 1.0 op_sel_hi:[1,0]\n\t" \
  "v_pk_add_f32 v[132:133], v[132:133], 1.0 op_sel_hi:[1,0]\n\t" \
  "v_pk_add_f32 v[134:135], v[134:135], 1.0 op_sel_hi:[1,0]\n\t" \
  "v_rcp_f32 v128, v128\n\t" \
  "v_rcp_f32 v129, v129\n\t" \
  "v_rcp_f32 v130, v130\n\t" \
  "v_rcp_f32 v131, v131\n\t" \
  "v_rcp_f32 v132, v132\n\t" \
  "v_rcp_f32 v133, v133\n\t" \
  "v_rcp_f32 v134, v134\n\t" \
  "v_rcp_f32 v135, v135\n\t" \
  "s_nop 0\n\t" \
  "v_pk_mul_f32 v[64:65], v[64:65], v[128:129]\n\t" \
  "v_pk_mul_f32 v[66:67], v[66:67], v[130:131]\n\t" \
  "v_pk_mul_f32 v[68:69], v[68:69], v[132:133]\n\t" \
  "v_pk_mul_f32 v[70:71], v[70:71], v[134:135]\n\t" \
  "s_nop 1\n\t" \
  "v_permlane16_swap_b32 v64, v68\n\t" \
  "v_permlane16_swap_b32 v65, v69\n\t" \
  "v_permlane16_swap_b32 v66, v70\n\t" \
  "v_permlane16_swap_b32 v67, v71\n\t" \
  "global_store_dword v254, v64, s[76:77] offset:0 " STPOL "\n\t" \
  "global_store_dword v254, v65, s[78:79] offset:0 " STPOL "\n\t" \
  "global_store_dword v254, v66, s[80:81] offset:0 " STPOL "\n\t" \
  "global_store_dword v254, v67, s[82:83] offset:0 " STPOL "\n\t" \
  "global_store_dword v255, v68, s[76:77] offset:0 " STPOL "\n\t" \
  "global_store_dword v255, v69, s[78:79] offset:0 " STPOL "\n\t" \
  "global_store_dword v255, v70, s[80:81] offset:0 " STPOL "\n\t" \
  "global_store_dword v255, v71, s[82:83] offset:0 " STPOL "\n\t" \
  "v_pk_mul_f32 v[136:137], v[96:97], s[96:97]\n\t" \
  "v_pk_mul_f32 v[138:139], v[98:99], s[96:97]\n\t" \
  "v_pk_mul_f32 v[140:141], v[100:101], s[96:97]\n\t" \
  "v_pk_mul_f32 v[142:143], v[102:103], s[96:97]\n\t" \
  "v_min_f32 v136, s92, v136\n\t" \
  "v_min_f32 v137, s92, v137\n\t" \
  "v_min_f32 v138, s92, v138\n\t" \
  "v_min_f32 v139, s92, v139\n\t" \
  "v_min_f32 v140, s92, v140\n\t" \
  "v_min_f32 v141, s92, v141\n\t" \
  "v_min_f32 v142, s92, v142\n\t" \
  "v_min_f32 v143, s92, v143\n\t" \
  "v_exp_f32 v136, v136\n\t" \
  "v_exp_f32 v137, v137\n\t" \
  "v_exp_f32 v138, v138\n\t" \
  "v_exp_f32 v139, v139\n\t" \
  "v_exp_f32 v140, v140\n\t" \
  "v_exp_f32 v141, v141\n\t" \
  "v_exp_f32 v142, v142\n\t" \
  "v_exp_f32 v143, v143\n\t" \
  "v_pk_mul_f32 v[96:97], v[96:97], s[94:95]\n\t" \
  "v_pk_mul_f32 v[98:99], v[98:99], s[94:95]\n\t" \
  "v_pk_mul_f32 v[100:101], v[100:101], s[94:95]\n\t" \
  "v_pk_mul_f32 v[102:103], v[102:103], s[94:95]\n\t" \
  "v_pk_add_f32 v[136:137], v[136:137], 1.0 op_sel_hi:[1,0]\n\t" \
  "v_pk_add_f32 v[138:139], v[138:139], 1.0 op_sel_hi:[1,0]\n\t" \
  "v_pk_add_f32 v[140:141], v[140:141], 1.0 op_sel_hi:[1,0]\n\t" \
  "v_pk_add_f32 v[142:143], v[142:143], 1.0 op_sel_hi:[1,0]\n\t" \
  "v_rcp_f32 v136, v136\n\t" \
  "v_rcp_f32 v137, v137\n\t" \
  "v_rcp_f32 v138, v138\n\t" \
  "v_rcp_f32 v139, v139\n\t" \
  "v_rcp_f32 v140, v140\n\t" \
  "v_rcp_f32 v141, v141\n\t" \
  "v_rcp_f32 v142, v142\n\t" \
  "v_rcp_f32 v143, v143\n\t" \
  "s_nop 0\n\t" \
  "v_pk_mul_f32 v[96:97], v[96:97], v[136:137]\n\t" \
  "v_pk_mul_f32 v[98:99], v[98:99], v[138:139]\n\t" \
  "v_pk_mul_f32 v[100:101], v[100:101], v[140:141]\n\t" \
  "v_pk_mul_f32 v[102:103], v[102:103], v[142:143]\n\t" \
  "s_nop 1\n\t" \
  "v_permlane16_swap_b32 v96, v100\n\t" \
  "v_permlane16_swap_b32 v97, v101\n\t" \
  "v_permlane16_swap_b32 v98, v102\n\t" \
  "v_permlane16_swap_b32 v99, v103\n\t" \
  "global_store_dword v254, v96, s[84:85] offset:0 " STPOL "\n\t" \
  "global_store_dword v254, v97, s[86:87] offset:0 " STPOL "\n\t" \
  "global_store_dword v254, v98, s[88:89] offset:0 " STPOL "\n\t" \
  "global_store_dword v254, v99, s[90:91] offset:0 " STPOL "\n\t" \
  "global_store_dword v255, v100, s[84:85] offset:0 " STPOL "\n\t" \
  "global_store_dword v255, v101, s[86:87] offset:0 " STPOL "\n\t" \
  "global_store_dword v255, v102, s[88:89] offset:0 " STPOL "\n\t" \
  "global_store_dword v255, v103, s[90:91] offset:0 " STPOL "\n\t" \
  "v_pk_mul_f32 v[128:129], v[72:73], s[96:97]\n\t" \
  "v_pk_mul_f32 v[130:131], v[74:75], s[96:97]\n\t" \
  "v_pk_mul_f32 v[132:133], v[76:77], s[96:97]\n\t" \
  "v_pk_mul_f32 v[134:135], v[78:79], s[96:97]\n\t" \
  "v_min_f32 v128, s92, v128\n\t" \
  "v_min_f32 v129, s92, v129\n\t" \
  "v_min_f32 v130, s92, v130\n\t" \
  "v_min_f32 v131, s92, v131\n\t" \
  "v_min_f32 v132, s92, v132\n\t" \
  "v_min_f32 v133, s92, v133\n\t" \
  "v_min_f32 v134, s92, v134\n\t" \
  "v_min_f32 v135, s92, v135\n\t" \
  "v_exp_f32 v128, v128\n\t" \
  "v_exp_f32 v129, v129\n\t" \
  "v_exp_f32 v130, v130\n\t" \
  "v_exp_f32 v131, v131\n\t" \
  "v_exp_f32 v132, v132\n\t" \
  "v_exp_f32 v133, v133\n\t" \
  "v_exp_f32 v134, v134\n\t" \
  "v_exp_f32 v135, v135\n\t" \
  "v_pk_mul_f32 v[72:73], v[72:73], s[94:95]\n\t" \
  "v_pk_mul_f32 v[74:75], v[74:75], s[94:95]\n\t" \
  "v_pk_mul_f32 v[76:77], v[76:77], s[94:95]\n\t" \
  "v_pk_mul_f32 v[78:79], v[78:79], s[94:95]\n\t" \
  "v_pk_add_f32 v[128:129], v[128:129], 1.0 op_sel_hi:[1,0]\n\t" \
  "v_pk_add_f32 v[130:131], v[130:131], 1.0 op_sel_hi:[1,0]\n\t" \
  "v_pk_add_f32 v[132:133], v[132:133], 1.0 op_sel_hi:[1,0]\n\t" \
  "v_pk_add_f32 v[134:135], v[134:135], 1.0 op_sel_hi:[1,0]\n\t" \
  "v_rcp_f32 v128, v128\n\t" \
  "v_rcp_f32 v129, v129\n\t" \
  "v_rcp_f32 v130, v130\n\t" \
  "v_rcp_f32 v131, v131\n\t" \
  "v_rcp_f32 v132, v132\n\t" \
  "v_rcp_f32 v133, v133\n\t" \
  "v_rcp_f32 v134, v134\n\t" \
  "v_rcp_f32 v135, v135\n\t" \
  "s_nop 0\n\t" \
  "v_pk_mul_f32 v[72:73], v[72:73], v[128:129]\n\t" \
  "v_pk_mul_f32 v[74:75], v[74:75], v[130:131]\n\t" \
  "v_pk_mul_f32 v[76:77], v[76:77], v[132:133]\n\t" \
  "v_pk_mul_f32 v[78:79], v[78:79], v[134:135]\n\t" \
  "s_nop 1\n\t" \
  "v_permlane16_swap_b32 v72, v76\n\t" \
  "v_permlane16_swap_b32 v73, v77\n\t" \
  "v_permlane16_swap_b32 v74, v78\n\t" \
  "v_permlane16_swap_b32 v75, v79\n\t" \
  "global_store_dword v254, v72, s[76:77] offset:128 " STPOL "\n\t" \
  "global_store_dword v254, v73, s[78:79] offset:128 " STPOL "\n\t" \
  "global_store_dword v254, v74, s[80:81] offset:128 " STPOL "\n\t" \
  "global_store_dword v254, v75, s[82:83] offset:128 " STPOL "\n\t" \
  "global_store_dword v255, v76, s[76:77] offset:128 " STPOL "\n\t" \
  "global_store_dword v255, v77, s[78:79] offset:128 " STPOL "\n\t" \
  "global_store_dword v255, v78, s[80:81] offset:128 " STPOL "\n\t" \
  "global_store_dword v255, v79, s[82:83] offset:128 " STPOL "\n\t" \
  "v_pk_mul_f32 v[136:137], v[104:105], s[96:97]\n\t" \
  "v_pk_mul_f32 v[138:139], v[106:107], s[96:97]\n\t" \
  "v_pk_mul_f32 v[140:141], v[108:109], s[96:97]\n\t" \
  "v_pk_mul_f32 v[142:143], v[110:111], s[96:97]\n\t" \
  "v_min_f32 v136, s92, v136\n\t" \
  "v_min_f32 v137, s92, v137\n\t" \
  "v_min_f32 v138, s92, v138\n\t" \
  "v_min_f32 v139, s92, v139\n\t" \
  "v_min_f32 v140, s92, v140\n\t" \
  "v_min_f32 v141, s92, v141\n\t" \
  "v_min_f32 v142, s92, v142\n\t" \
  "v_min_f32 v143, s92, v143\n\t" \
  "v_exp_f32 v136, v136\n\t" \
  "v_exp_f32 v137, v137\n\t" \
  "v_exp_f32 v138, v138\n\t" \
  "v_exp_f32 v139, v139\n\t" \
  "v_exp_f32 v140, v140\n\t" \
  "v_exp_f32 v141, v141\n\t" \
  "v_exp_f32 v142, v142\n\t" \
  "v_exp_f32 v143, v143\n\t" \
  "v_pk_mul_f32 v[104:105], v[104:105], s[94:95]\n\t" \
  "v_pk_mul_f32 v[106:107], v[106:107], s[94:95]\n\t" \
  "v_pk_mul_f32 v[108:109], v[108:109], s[94:95]\n\t" \
  "v_pk_mul_f32 v[110:111], v[110:111], s[94:95]\n\t" \
  "v_pk_add_f32 v[136:137], v[136:137], 1.0 op_sel_hi:[1,0]\n\t" \
  "v_pk_add_f32 v[138:139], v[138:139], 1.0 op_sel_hi:[1,0]\n\t" \
  "v_pk_add_f32 v[140:141], v[140:141], 1.0 op_sel_hi:[1,0]\n\t" \
  "v_pk_add_f32 v[142:143], v[142:143], 1.0 op_sel_hi:[1,0]\n\t" \
  "v_rcp_f32 v136, v136\n\t" \
  "v_rcp_f32 v137, v137\n\t" \
  "v_rcp_f32 v138, v138\n\t" \
  "v_rcp_f32 v139, v139\n\t" \
  "v_rcp_f32 v140, v140\n\t" \
  "v_rcp_f32 v141, v141\n\t" \
  "v_rcp_f32 v142, v142\n\t" \
  "v_rcp_f32 v143, v143\n\t" \
  "s_nop 0\n\t" \
  "v_pk_mul_f32 v[104:105], v[104:105], v[136:137]\n\t" \
  "v_pk_mul_f32 v[106:107], v[106:107], v[138:139]\n\t" \
  "v_pk_mul_f32 v[108:109], v[108:109], v[140:141]\n\t" \
  "v_pk_mul_f32 v[110:111], v[110:111], v[142:143]\n\t" \
  "s_nop 1\n\t" \
  "v_permlane16_swap_b32 v104, v108\n\t" \
  "v_permlane16_swap_b32 v105, v109\n\t" \
  "v_permlane16_swap_b32 v106, v110\n\t" \
  "v_permlane16_swap_b32 v107, v111\n\t" \
  "global_store_dword v254, v104, s[84:85] offset:128 " STPOL "\n\t" \
  "global_store_dword v254, v105, s[86:87] offset:128 " STPOL "\n\t" \
  "global_store_dword v254, v106, s[88:89] offset:128 " STPOL "\n\t" \
  "global_store_dword v254, v107, s[90:91] offset:128 " STPOL "\n\t" \
  "global_store_dword v255, v108, s[84:85] offset:128 " STPOL "\n\t" \
  "global_store_dword v255, v109, s[86:87] offset:128 " STPOL "\n\t" \
  "global_store_dword v255, v110, s[88:89] offset:128 " STPOL "\n\t" \
  "global_store_dword v255, v111, s[90:91] offset:128 " STPOL "\n\t" \
  "v_pk_mul_f32 v[128:129], v[80:81], s[96:97]\n\t" \
  "v_pk_mul_f32 v[130:131], v[82:83], s[96:97]\n\t" \
  "v_pk_mul_f32 v[132:133], v[84:85], s[96:97]\n\t" \
  "v_pk_mul_f32 v[134:135], v[86:87], s[96:97]\n\t" \
  "v_min_f32 v128, s92, v128\n\t" \
  "v_min_f32 v129, s92, v129\n\t" \
  "v_min_f32 v130, s92, v130\n\t" \
  "v_min_f32 v131, s92, v131\n\t" \
  "v_min_f32 v132, s92, v132\n\t" \
  "v_min_f32 v133, s92, v133\n\t" \
  "v_min_f32 v134, s92, v134\n\t" \
  "v_min_f32 v135, s92, v135\n\t" \
  "v_exp_f32 v128, v128\n\t" \
  "v_exp_f32 v129, v129\n\t" \
  "v_exp_f32 v130, v130\n\t" \
  "v_exp_f32 v131, v131\n\t" \
  "v_exp_f32 v132, v132\n\t" \
  "v_exp_f32 v133, v133\n\t" \
  "v_exp_f32 v134, v134\n\t" \
  "v_exp_f32 v135, v135\n\t" \
  "v_pk_mul_f32 v[80:81], v[80:81], s[94:95]\n\t" \
  "v_pk_mul_f32 v[82:83], v[82:83], s[94:95]\n\t" \
  "v_pk_mul_f32 v[84:85], v[84:85], s[94:95]\n\t" \
  "v_pk_mul_f32 v[86:87], v[86:87], s[94:95]\n\t" \
  "v_pk_add_f32 v[128:129], v[128:129], 1.0 op_sel_hi:[1,0]\n\t" \
  "v_pk_add_f32 v[130:131], v[130:131], 1.0 op_sel_hi:[1,0]\n\t" \
  "v_pk_add_f32 v[132:133], v[132:133], 1.0 op_sel_hi:[1,0]\n\t" \
  "v_pk_add_f32 v[134:135], v[134:135], 1.0 op_sel_hi:[1,0]\n\t" \
  "v_rcp_f32 v128, v128\n\t" \
  "v_rcp_f32 v129, v129\n\t" \
  "v_rcp_f32 v130, v130\n\t" \
  "v_rcp_f32 v131, v131\n\t" \
  "v_rcp_f32 v132, v132\n\t" \
  "v_rcp_f32 v133, v133\n\t" \
  "v_rcp_f32 v134, v134\n\t" \
  "v_rcp_f32 v135, v135\n\t" \
  "s_nop 0\n\t" \
  "v_pk_mul_f32 v[80:81], v[80:81], v[128:129]\n\t" \
  "v_pk_mul_f32 v[82:83], v[82:83], v[130:131]\n\t" \
  "v_pk_mul_f32 v[84:85], v[84:85], v[132:133]\n\t" \
  "v_pk_mul_f32 v[86:87], v[86:87], v[134:135]\n\t" \
  "s_nop 1\n\t" \
  "v_permlane16_swap_b32 v80, v84\n\t" \
  "v_permlane16_swap_b32 v81, v85\n\t" \
  "v_permlane16_swap_b32 v82, v86\n\t" \
  "v_permlane16_swap_b32 v83, v87\n\t" \
  "global_store_dword v254, v80, s[76:77] offset:256 " STPOL "\n\t" \
  "global_store_dword v254, v81, s[78:79] offset:256 " STPOL "\n\t" \
  "global_store_dword v254, v82, s[80:81] offset:256 " STPOL "\n\t" \
  "global_store_dword v254, v83, s[82:83] offset:256 " STPOL "\n\t" \
  "global_store_dword v255, v84, s[76:77] offset:256 " STPOL "\n\t" \
  "global_store_dword v255, v85, s[78:79] offset:256 " STPOL "\n\t" \
  "global_store_dword v255, v86, s[80:81] offset:256 " STPOL "\n\t" \
  "global_store_dword v255, v87, s[82:83] offset:256 " STPOL "\n\t" \
  "v_pk_mul_f32 v[136:137], v[112:113], s[96:97]\n\t" \
  "v_pk_mul_f32 v[138:139], v[114:115], s[96:97]\n\t" \
  "v_pk_mul_f32 v[140:141], v[116:117], s[96:97]\n\t" \
  "v_pk_mul_f32 v[142:143], v[118:119], s[96:97]\n\t" \
  "v_min_f32 v136, s92, v136\n\t" \
  "v_min_f32 v137, s92, v137\n\t" \
  "v_min_f32 v138, s92, v138\n\t" \
  "v_min_f32 v139, s92, v139\n\t" \
  "v_min_f32 v140, s92, v140\n\t" \
  "v_min_f32 v141, s92, v141\n\t" \
  "v_min_f32 v142, s92, v142\n\t" \
  "v_min_f32 v143, s92, v143\n\t" \
  "v_exp_f32 v136, v136\n\t" \
  "v_exp_f32 v137, v137\n\t" \
  "v_exp_f32 v138, v138\n\t" \
  "v_exp_f32 v139, v139\n\t" \
  "v_exp_f32 v140, v140\n\t" \
  "v_exp_f32 v141, v141\n\t" \
  "v_exp_f32 v142, v142\n\t" \
  "v_exp_f32 v143, v143\n\t" \
  "v_pk_mul_f32 v[112:113], v[112:113], s[94:95]\n\t" \
  "v_pk_mul_f32 v[114:115], v[114:115], s[94:95]\n\t" \
  "v_pk_mul_f32 v[116:117], v[116:117], s[94:95]\n\t" \
  "v_pk_mul_f32 v[118:119], v[118:119], s[94:95]\n\t" \
  "v_pk_add_f32 v[136:137], v[136:137], 1.0 op_sel_hi:[1,0]\n\t" \
  "v_pk_add_f32 v[138:139], v[138:139], 1.0 op_sel_hi:[1,0]\n\t" \
  "v_pk_add_f32 v[140:141], v[140:141], 1.0 op_sel_hi:[1,0]\n\t" \
  "v_pk_add_f32 v[142:143], v[142:143], 1.0 op_sel_hi:[1,0]\n\t" \
  "v_rcp_f32 v136, v136\n\t" \
  "v_rcp_f32 v137, v137\n\t" \
  "v_rcp_f32 v138, v138\n\t" \
  "v_rcp_f32 v139, v139\n\t" \
  "v_rcp_f32 v140, v140\n\t" \
  "v_rcp_f32 v141, v141\n\t" \
  "v_rcp_f32 v142, v142\n\t" \
  "v_rcp_f32 v143, v143\n\t" \
  "s_nop 0\n\t" \
  "v_pk_mul_f32 v[112:113], v[112:113], v[136:137]\n\t" \
  "v_pk_mul_f32 v[114:115], v[114:115], v[138:139]\n\t" \
  "v_pk_mul_f32 v[116:117], v[116:117], v[140:141]\n\t" \
  "v_pk_mul_f32 v[118:119], v[118:119], v[142:143]\n\t" \
  "s_nop 1\n\t" \
  "v_permlane16_swap_b32 v112, v116\n\t" \
  "v_permlane16_swap_b32 v113, v117\n\t" \
  "v_permlane16_swap_b32 v114, v118\n\t" \
  "v_permlane16_swap_b32 v115, v119\n\t" \
  "global_store_dword v254, v112, s[84:85] offset:256 " STPOL "\n\t" \
  "global_store_dword v254, v113, s[86:87] offset:256 " STPOL "\n\t" \
  "global_store_dword v254, v114, s[88:89] offset:256 " STPOL "\n\t" \
  "global_store_dword v254, v115, s[90:91] offset:256 " STPOL "\n\t" \
  "global_store_dword v255, v116, s[84:85] offset:256 " STPOL "\n\t" \
  "global_store_dword v255, v117, s[86:87] offset:256 " STPOL "\n\t" \
  "global_store_dword v255, v118, s[88:89] offset:256 " STPOL "\n\t" \
  "global_store_dword v255, v119, s[90:91] offset:256 " STPOL "\n\t" \
  "v_pk_mul_f32 v[128:129], v[88:89], s[96:97]\n\t" \
  "v_pk_mul_f32 v[130:131], v[90:91], s[96:97]\n\t" \
  "v_pk_mul_f32 v[132:133], v[92:93], s[96:97]\n\t" \
  "v_pk_mul_f32 v[134:135], v[94:95], s[96:97]\n\t" \
  "v_min_f32 v128, s92, v128\n\t" \
  "v_min_f32 v129, s92, v129\n\t" \
  "v_min_f32 v130, s92, v130\n\t" \
  "v_min_f32 v131, s92, v131\n\t" \
  "v_min_f32 v132, s92, v132\n\t" \
  "v_min_f32 v133, s92, v133\n\t" \
  "v_min_f32 v134, s92, v134\n\t" \
  "v_min_f32 v135, s92, v135\n\t" \
  "v_exp_f32 v128, v128\n\t" \
  "v_exp_f32 v129, v129\n\t" \
  "v_exp_f32 v130, v130\n\t" \
  "v_exp_f32 v131, v131\n\t" \
  "v_exp_f32 v132, v132\n\t" \
  "v_exp_f32 v133, v133\n\t" \
  "v_exp_f32 v134, v134\n\t" \
  "v_exp_f32 v135, v135\n\t" \
  "v_pk_mul_f32 v[88:89], v[88:89], s[94:95]\n\t" \
  "v_pk_mul_f32 v[90:91], v[90:91], s[94:95]\n\t" \
  "v_pk_mul_f32 v[92:93], v[92:93], s[94:95]\n\t" \
  "v_pk_mul_f32 v[94:95], v[94:95], s[94:95]\n\t" \
  "v_pk_add_f32 v[128:129], v[128:129], 1.0 op_sel_hi:[1,0]\n\t" \
  "v_pk_add_f32 v[130:131], v[130:131], 1.0 op_sel_hi:[1,0]\n\t" \
  "v_pk_add_f32 v[132:133], v[132:133], 1.0 op_sel_hi:[1,0]\n\t" \
  "v_pk_add_f32 v[134:135], v[134:135], 1.0 op_sel_hi:[1,0]\n\t" \
  "v_rcp_f32 v128, v128\n\t" \
  "v_rcp_f32 v129, v129\n\t" \
  "v_rcp_f32 v130, v130\n\t" \
  "v_rcp_f32 v131, v131\n\t" \
  "v_rcp_f32 v132, v132\n\t" \
  "v_rcp_f32 v133, v133\n\t" \
  "v_rcp_f32 v134, v134\n\t" \
  "v_rcp_f32 v135, v135\n\t" \
  "s_nop 0\n\t" \
  "v_pk_mul_f32 v[88:89], v[88:89], v[128:129]\n\t" \
  "v_pk_mul_f32 v[90:91], v[90:91], v[130:131]\n\t" \
  "v_pk_mul_f32 v[92:93], v[92:93], v[132:133]\n\t" \
  "v_pk_mul_f32 v[94:95], v[94:95], v[134:135]\n\t" \
  "s_nop 1\n\t" \
  "v_permlane16_swap_b32 v88, v92\n\t" \
  "v_permlane16_swap_b32 v89, v93\n\t" \
  "v_permlane16_swap_b32 v90, v94\n\t" \
  "v_permlane16_swap_b32 v91, v95\n\t" \
  "global_store_dword v254, v88, s[76:77] offset:384 " STPOL "\n\t" \
  "global_store_dword v254, v89, s[78:79] offset:384 " STPOL "\n\t" \
  "global_store_dword v254, v90, s[80:81] offset:384 " STPOL "\n\t" \
  "global_store_dword v254, v91, s[82:83] offset:384 " STPOL "\n\t" \
  "global_store_dword v255, v92, s[76:77] offset:384 " STPOL "\n\t" \
  "global_store_dword v255, v93, s[78:79] offset:384 " STPOL "\n\t" \
  "global_store_dword v255, v94, s[80:81] offset:384 " STPOL "\n\t" \
  "global_store_dword v255, v95, s[82:83] offset:384 " STPOL "\n\t" \
  "v_pk_mul_f32 v[136:137], v[120:121], s[96:97]\n\t" \
  "v_pk_mul_f32 v[138:139], v[122:123], s[96:97]\n\t" \
  "v_pk_mul_f32 v[140:141], v[124:125], s[96:97]\n\t" \
  "v_pk_mul_f32 v[142:143], v[126:127], s[96:97]\n\t" \
  "v_min_f32 v136, s92, v136\n\t" \
  "v_min_f32 v137, s92, v137\n\t" \
  "v_min_f32 v138, s92, v138\n\t" \
  "v_min_f32 v139, s92, v139\n\t" \
  "v_min_f32 v140, s92, v140\n\t" \
  "v_min_f32 v141, s92, v141\n\t" \
  "v_min_f32 v142, s92, v142\n\t" \
  "v_min_f32 v143, s92, v143\n\t" \
  "v_exp_f32 v136, v136\n\t" \
  "v_exp_f32 v137, v137\n\t" \
  "v_exp_f32 v138, v138\n\t" \
  "v_exp_f32 v139, v139\n\t" \
  "v_exp_f32 v140, v140\n\t" \
  "v_exp_f32 v141, v141\n\t" \
  "v_exp_f32 v142, v142\n\t" \
  "v_exp_f32 v143, v143\n\t" \
  "v_pk_mul_f32 v[120:121], v[120:121], s[94:95]\n\t" \
  "v_pk_mul_f32 v[122:123], v[122:123], s[94:95]\n\t" \
  "v_pk_mul_f32 v[124:125], v[124:125], s[94:95]\n\t" \
  "v_pk_mul_f32 v[126:127], v[126:127], s[94:95]\n\t" \
  "v_pk_add_f32 v[136:137], v[136:137], 1.0 op_sel_hi:[1,0]\n\t" \
  "v_pk_add_f32 v[138:139], v[138:139], 1.0 op_sel_hi:[1,0]\n\t" \
  "v_pk_add_f32 v[140:141], v[140:141], 1.0 op_sel_hi:[1,0]\n\t" \
  "v_pk_add_f32 v[142:143], v[142:143], 1.0 op_sel_hi:[1,0]\n\t" \
  "v_rcp_f32 v136, v136\n\t" \
  "v_rcp_f32 v137, v137\n\t" \
  "v_rcp_f32 v138, v138\n\t" \
  "v_rcp_f32 v139, v139\n\t" \
  "v_rcp_f32 v140, v140\n\t" \
  "v_rcp_f32 v141, v141\n\t" \
  "v_rcp_f32 v142, v142\n\t" \
  "v_rcp_f32 v143, v143\n\t" \
  "s_nop 0\n\t" \
  "v_pk_mul_f32 v[120:121], v[120:121], v[136:137]\n\t" \
  "v_pk_mul_f32 v[122:123], v[122:123], v[138:139]\n\t" \
  "v_pk_mul_f32 v[124:125], v[124:125], v[140:141]\n\t" \
  "v_pk_mul_f32 v[126:127], v[126:127], v[142:143]\n\t" \
  "s_nop 1\n\t" \
  "v_permlane16_swap_b32 v120, v124\n\t" \
  "v_permlane16_swap_b32 v121, v125\n\t" \
  "v_permlane16_swap_b32 v122, v126\n\t" \
  "v_permlane16_swap_b32 v123, v127\n\t" \
  "global_store_dword v254, v120, s[84:85] offset:384 " STPOL "\n\t" \
  "global_store_dword v254, v121, s[86:87] offset:384 " STPOL "\n\t" \
  "global_store_dword v254, v122, s[88:89] offset:384 " STPOL "\n\t" \
  "global_store_dword v254, v123, s[90:91] offset:384 " STPOL "\n\t" \
  "global_store_dword v255, v124, s[84:85] offset:384 " STPOL "\n\t" \
  "global_store_dword v255, v125, s[86:87] offset:384 " STPOL "\n\t" \
  "global_store_dword v255, v126, s[88:89] offset:384 " STPOL "\n\t" \
  "global_store_dword v255, v127, s[90:91] offset:384 " STPOL "\n\t" \
  "s_endpgm\n\t" \
  "Lc1_spin%=:\n\t" \
  "s_sub_u32 s56, s56, 1\n\t" \
  "s_cbranch_scc1 Lc1_quit%=\n\t" \
  "s_sleep 1\n\t" \
  "s_mov_b32 m0, 0\n\t" \
  "s_nop 0\n\t" \
  "ds_read_addtid_b32 v253 offset:0\n\t" \
  "s_waitcnt lgkmcnt(0)\n\t" \
  "s_branch Lc1_retry%=\n\t" \
  "Lc1_quit%=:\n\t" \
  "s_mov_b32 s56, 0\n\t" \
  "s_mov_b32 s48, 0x7fc00000\n\t" \
  "s_branch Lc1_ok%=\n\t" \
  "Lc2_spin%=:\n\t" \
  "s_sub_u32 s56, s56, 1\n\t" \
  "s_cbranch_scc1 Lc2_quit%=\n\t" \
  "s_sleep 1\n\t" \
  "s_mov_b32 m0, 0\n\t" \
  "s_nop 0\n\t" \
  "ds_read_addtid_b32 v253 offset:0\n\t" \
  "s_waitcnt lgkmcnt(0)\n\t" \
  "s_branch Lc2_retry%=\n\t" \
  "Lc2_quit%=:\n\t" \
  "s_mov_b32 s56, 0\n\t" \
  "s_mov_b32 s48, 0x7fc00000\n\t" \
  "s_branch Lc2_ok%=\n\t" \
  "Lc3_spin%=:\n\t" \
  "s_sub_u32 s56, s56, 1\n\t" \
  "s_cbranch_scc1 Lc3_quit%=\n\t" \
  "s_sleep 1\n\t" \
  "s_mov_b32 m0, 0\n\t" \
  "s_nop 0\n\t" \
  "ds_read_addtid_b32 v253 offset:0\n\t" \
  "s_waitcnt lgkmcnt(0)\n\t" \
  "s_branch Lc3_retry%=\n\t" \
  "Lc3_quit%=:\n\t" \
  "s_mov_b32 s56, 0\n\t" \
  "s_mov_b32 s48, 0x7fc00000\n\t" \
  "s_branch Lc3_ok%=\n\t" \
  "Lc4_spin%=:\n\t" \
  "s_sub_u32 s56, s56, 1\n\t" \
  "s_cbranch_scc1 Lc4_quit%=\n\t" \
  "s_sleep 1\n\t" \
  "s_mov_b32 m0, 0\n\t" \
  "s_nop 0\n\t" \
  "ds_read_addtid_b32 v253 offset:0\n\t" \
  "s_waitcnt lgkmcnt(0)\n\t" \
  "s_branch Lc4_retry%=\n\t" \
  "Lc4_quit%=:\n\t" \
  "s_mov_b32 s56, 0\n\t" \
  "s_mov_b32 s48, 0x7fc00000\n\t" \
  "s_branch Lc4_ok%=\n\t" \
  "Lc5_spin%=:\n\t" \
  "s_sub_u32 s56, s56, 1\n\t" \
  "s_cbranch_scc1 Lc5_quit%=\n\t" \
  "s_sleep 1\n\t" \
  "s_mov_b32 m0, 0\n\t" \
  "s_nop 0\n\t" \
  "ds_read_addtid_b32 v253 offset:0\n\t" \
  "s_waitcnt lgkmcnt(0)\n\t" \
  "s_branch Lc5_retry%=\n\t" \
  "Lc5_quit%=:\n\t" \
  "s_mov_b32 s56, 0\n\t" \
  "s_mov_b32 s48, 0x7fc00000\n\t" \
  "s_branch Lc5_ok%=\n\t" \
  "Lc6_spin%=:\n\t" \
  "s_sub_u32 s56, s56, 1\n\t" \
  "s_cbranch_scc1 Lc6_quit%=\n\t" \
  "s_sleep 1\n\t" \
  "s_mov_b32 m0, 0\n\t" \
  "s_nop 0\n\t" \
  "ds_read_addtid_b32 v253 offset:0\n\t" \
  "s_waitcnt lgkmcnt(0)\n\t" \
  "s_branch Lc6_retry%=\n\t" \
  "Lc6_quit%=:\n\t" \
  "s_mov_b32 s56, 0\n\t" \
  "s_mov_b32 s48, 0x7fc00000\n\t" \
  "s_branch Lc6_ok%=\n\t" \
  "Lc7_spin%=:\n\t" \
  "s_sub_u32 s56, s56, 1\n\t" \
  "s_cbranch_scc1 Lc7_quit%=\n\t" \
  "s_sleep 1\n\t" \
  "s_mov_b32 m0, 0\n\t" \
  "s_nop 0\n\t" \
  "ds_read_addtid_b32 v253 offset:0\n\t" \
  "s_waitcnt lgkmcnt(0)\n\t" \
  "s_branch Lc7_retry%=\n\t" \
  "Lc7_quit%=:\n\t" \
  "s_mov_b32 s56, 0\n\t" \
  "s_mov_b32 s48, 0x7fc00000\n\t" \
  "s_branch Lc7_ok%=\n\t" \
  "Lc8_spin%=:\n\t" \
  "s_sub_u32 s56, s56, 1\n\t" \
  "s_cbranch_scc1 Lc8_quit%=\n\t" \
  "s_sleep 1\n\t" \
  "s_mov_b32 m0, 0\n\t" \
  "s_nop 0\n\t" \
  "ds_read_addtid_b32 v253 offset:0\n\t" \
  "s_waitcnt lgkmcnt(0)\n\t" \
  "s_branch Lc8_retry%=\n\t" \
  "Lc8_quit%=:\n\t" \
  "s_mov_b32 s56, 0\n\t" \
  "s_mov_b32 s48, 0x7fc00000\n\t" \
  "s_branch Lc8_ok%=\n\t" \
  "Lc9_spin%=:\n\t" \
  "s_sub_u32 s56, s56, 1\n\t" \
  "s_cbranch_scc1 Lc9_quit%=\n\t" \
  "s_sleep 1\n\t" \
  "s_mov_b32 m0, 0\n\t" \
  "s_nop 0\n\t" \
  "ds_read_addtid_b32 v253 offset:0\n\t" \
  "s_waitcnt lgkmcnt(0)\n\t" \
  "s_branch Lc9_retry%=\n\t" \
  "Lc9_quit%=:\n\t" \
  "s_mov_b32 s56, 0\n\t" \
  "s_mov_b32 s48, 0x7fc00000\n\t" \
  "s_branch Lc9_ok%=\n\t" \
  "Lc10_spin%=:\n\t" \
  "s_sub_u32 s56, s56, 1\n\t" \
  "s_cbranch_scc1 Lc10_quit%=\n\t" \
  "s_sleep 1\n\t" \
  "s_mov_b32 m0, 0\n\t" \
  "s_nop 0\n\t" \
  "ds_read_addtid_b32 v253 offset:0\n\t" \
  "s_waitcnt lgkmcnt(0)\n\t" \
  "s_branch Lc10_retry%=\n\t" \
  "Lc10_quit%=:\n\t" \
  "s_mov_b32 s56, 0\n\t" \
  "s_mov_b32 s48, 0x7fc00000\n\t" \
  "s_branch Lc10_ok%=\n\t" \
  "Lc11_spin%=:\n\t" \
  "s_sub_u32 s56, s56, 1\n\t" \
  "s_cbranch_scc1 Lc11_quit%=\n\t" \
  "s_sleep 1\n\t" \
  "s_mov_b32 m0, 0\n\t" \
  "s_nop 0\n\t" \
  "ds_read_addtid_b32 v253 offset:0\n\t" \
  "s_waitcnt lgkmcnt(0)\n\t" \
  "s_branch Lc11_retry%=\n\t" \
  "Lc11_quit%=:\n\t" \
  "s_mov_b32 s56, 0\n\t" \
  "s_mov_b32 s48, 0x7fc00000\n\t" \
  "s_branch Lc11_ok%=\n\t" \
  "Lend%=:\n\t"
#define CLOB_V "v0","v1","v2","v3","v4","v5","v6","v7","v8","v9","v10","v11","v12","v13","v14","v15","v16","v17","v18","v19","v20","v21","v22","v23","v24","v25","v26","v27","v28","v29","v30","v31","v32","v33","v34","v35","v36","v37","v38","v39","v40","v41","v42","v43","v44","v45","v46","v47","v48","v49","v50","v51","v52","v53","v54","v55","v56","v57","v58","v59","v60","v61","v62","v63","v64","v65","v66","v67","v68","v69","v70","v71","v72","v73","v74","v75","v76","v77","v78","v79","v80","v81","v82","v83","v84","v85","v86","v87","v88","v89","v90","v91","v92","v93","v94","v95","v96","v97","v98","v99","v100","v101","v102","v103","v104","v105","v106","v107","v108","v109","v110","v111","v112","v113","v114","v115","v116","v117","v118","v119","v120","v121","v122","v123","v124","v125","v126","v127","v128","v129","v130","v131","v132","v133","v134","v135","v136","v137","v138","v139","v140","v141","v142","v143","v144","v145","v146","v147","v148","v149","v150","v151","v152","v153","v154","v155","v156","v157","v158","v159","v160","v161","v162","v163","v164","v165","v166","v167","v168","v169","v170","v171","v172","v173","v174","v175","v176","v177","v178","v179","v180","v181","v182","v183","v184","v185","v186","v187","v188","v189","v190","v191","v192","v193","v194","v195","v196","v197","v198","v199","v200","v201","v202","v203","v204","v205","v206","v207","v208","v209","v210","v211","v212","v213","v214","v215","v216","v217","v218","v219","v220","v221","v222","v223","v224","v225","v226","v227","v228","v229","v230","v231","v232","v233","v234","v235","v236","v237","v238","v239","v240","v241","v242","v243","v244","v245","v246","v247","v248","v249","v250","v251","v252","v253","v254","v255"
#define CLOB_S "s40","s41","s42","s43","s44","s45","s46","s47","s48","s49","s50","s51","s52","s53","s54","s55","s56","s57","s58","s59","s60","s61","s62","s63","s64","s65","s66","s67","s68","s69","s70","s71","s72","s73","s74","s75","s76","s77","s78","s79","s80","s81","s82","s83","s84","s85","s86","s87","s88","s89","s90","s91","s92","s93","s94","s95","s96","s97"
#define XLAST 1
constexpr unsigned L_TAB = 0, L_PROG = 256, L_RING = 2048, LDS_BYTES = 133120;

constexpr int W_PUSH = 0, W_LOAD = 1, W_POLL = 2, W_COMP = 3;
#define NFIRST 1
#define PAIR_SWAP 1
#define TOUCH 1
#define EARLYLD 0
#define PUSHPOL "sc0 sc1"
#define PUSH_FIRST 1
#define OTOUCH 0
#define ATOUCH 0
#define ATOUCH_AHEAD 6
#define TOUCH_START 2
#define TOUCH_AHEAD 5
#define WTHR0 12
#define WMAXOUT 63
#define WDEPTH 6
#define WBURST 1
#define NREP 8
#define PUSH_PACE 200
#define POLL_AHEAD 7
#define POLL_SLEEP 4
constexpr int NDIM = 16384;
constexpr int NRANK = 8;
constexpr size_t WS_CNT = 0;
constexpr size_t WS_TMO = 65536;
constexpr size_t WS_FLAG = 131072;
constexpr size_t WS_STAGE = 1u << 20;
constexpr size_t WS_NEED = WS_STAGE + (size_t)64 * 65536;
constexpr unsigned long long DEADLINE_TICKS = 50000000ull;

struct Params {
  const unsigned char* x;
  const unsigned char* w;
  const float* sx;
  const float* sw;
  float* out;
  unsigned char* myws;
  int rank;
  int pad0;
  unsigned long long* stamps;
  unsigned char* peer[NRANK];
};
static_assert(sizeof(Params) == 128, "kernarg layout");

DI unsigned rfl(unsigned x) { return __builtin_amdgcn_readfirstlane(x); }

#define WPOL ""
#define GL(i, off) "global_load_lds_dwordx4 %[o" #i "], s[40:41] offset:" #off " " WPOL "\n\t"
#define GL16 \
      "s_mov_b32 m0, %[lds]\n\t" \
      "s_add_u32 %[lds], %[lds], 0x4000\n\t" \
      GL(0, 0) GL(1, 1024) GL(2, 2048) GL(3, 3072) \
      "s_add_u32 m0, m0, 0x1000\n\t" \
      "s_add_u32 %[iss], %[iss], 2\n\t" \
      GL(4, 0) GL(5, 1024) GL(6, 2048) GL(7, 3072) \
      "s_add_u32 s40, s40, 0x100000\n\t" \
      "s_addc_u32 s41, s41, 0\n\t" \
      "s_add_u32 m0, m0, 0x1000\n\t" \
      "s_cmp_eq_u32 %[lds], %[ldsend]\n\t" \
      GL(0, 0) GL(1, 1024) GL(2, 2048) GL(3, 3072) \
      "s_cselect_b32 %[lds], %[wrap], %[lds]\n\t" \
      "s_add_u32 m0, m0, 0x1000\n\t" \
      "s_nop 0\n\t" \
      GL(4, 0) GL(5, 1024) GL(6, 2048) GL(7, 3072)
DI void loader_w(const unsigned char* wbase, const unsigned (&voff)[8], unsigned vrd, unsigned first_chunk) {
  unsigned s_iss, s_lds, s_t0, s_t1, s_t2, s_spin, s_wrap, vval, vfull = L_TAB + 128;
  asm volatile(
      "s_setprio 3\n\t"
      "s_mov_b32 %[iss], 0\n\t"
      "s_mov_b32 %[lds], %[ldsbeg]\n\t"
      "s_mov_b32 %[wrap], %[ldsbeg]\n\t"
      "s_mov_b32 %[spin], 0x1000000\n\t"
      "Lfirst%=:\n\t"
      "s_lshl1_add_u32 s42, %[fc], %[iss]\n\t"
      "s_lshl_b32 s42, s42, 20\n\t"
      "s_add_u32 s40, %[wlo], s42\n\t"
      "s_addc_u32 s41, %[whi], 0\n\t"
      GL16
      "s_cmp_lt_u32 %[iss], %[nfirst]\n\t"
      "s_cbranch_scc1 Lfirst%=\n\t"
      "Lpoll%=:\n\t"
      "s_getreg_b32 %[t0], hwreg(HW_REG_IB_STS, 0, 24)\n\t"
      "ds_read_b32 v60, %[vrd]\n\t"
      "s_bfe_u32 %[t1], %[t0], 0x20016\n\t"
      "s_and_b32 %[t0], %[t0], 15\n\t"
      "s_lshl4_add_u32 s43, %[t1], %[t0]\n\t"
      "s_add_u32 %[t0], s43, 7\n\t"
      "s_lshr_b32 %[t0], %[t0], 3\n\t"
      "s_sub_u32 %[t0], %[iss], %[t0]\n\t"
      "v_mov_b32 %[vval], %[t0]\n\t"
      "ds_write_b32 %[vfull], %[vval]\n\t"
      "s_lshr_b32 %[t2], %[iss], 1\n\t"
      "s_waitcnt lgkmcnt(1)\n\t"
      "v_readlane_b32 %[t0], v60, 32\n\t"
      "v_readlane_b32 %[t1], v60, 33\n\t"
      "v_readlane_b32 s40, v60, 34\n\t"
      "v_readlane_b32 s41, v60, 35\n\t"
      "v_readlane_b32 s42, v60, %[t2]\n\t"
      "s_min_u32 %[t0], %[t0], %[t1]\n\t"
      "s_min_u32 s40, s40, s41\n\t"
      "s_min_u32 %[t0], %[t0], s40\n\t"
      "s_cmp_eq_u32 %[t0], 0\n\t"
      "s_cselect_b32 %[t1], 4, %[depth]\n\t"
      "s_add_u32 %[t0], %[t0], %[t1]\n\t"
      "s_cmp_lt_u32 %[t2], 4\n\t"
      "s_cselect_b32 %[t1], %[thr0], %[thr1]\n\t"
      "s_cmp_le_u32 s43, %[t1]\n\t"
      "s_cselect_b32 s42, s42, 0\n\t"
      "s_min_u32 %[t2], %[t2], %[lastq]\n\t"
      "s_or_b32 %[t2], %[t2], %[bmask]\n\t"
      "s_cmp_lt_u32 %[t2], %[t0]\n\t"
      "s_cselect_b32 s42, s42, 0\n\t"
      "s_cmp_lg_u32 s42, 0\n\t"
      "s_cbranch_scc1 Lissue%=\n\t"
      "s_sub_u32 %[spin], %[spin], 1\n\t"
      "s_cbranch_scc1 Lquit%=\n\t"
      "s_sleep 1\n\t"
      "s_branch Lpoll%=\n\t"
      "Lissue%=:\n\t"
      "s_cmp_eq_u32 %[iss], 58\n\t"
      "s_cselect_b32 %[wrap], %[wraplast], %[wrap]\n\t"
      "s_sub_u32 s42, s42, 1\n\t"
      "s_and_b32 s42, s42, 31\n\t"
      "s_lshl_b32 s42, s42, 21\n\t"
      "s_add_u32 s40, %[wlo], s42\n\t"
      "s_addc_u32 s41, %[whi], 0\n\t"
      GL16
      "s_cmp_lt_u32 %[iss], 64\n\t"
      "s_cbranch_scc1 Lpoll%=\n\t"
      "Ldrain%=:\n\t"
      "s_getreg_b32 %[t0], hwreg(HW_REG_IB_STS, 0, 24)\n\t"
      "s_bfe_u32 %[t1], %[t0], 0x20016\n\t"
      "s_and_b32 %[t0], %[t0], 15\n\t"
      "s_lshl4_add_u32 %[t0], %[t1], %[t0]\n\t"
      "s_add_u32 %[t1], %[t0], 7\n\t"
      "s_lshr_b32 %[t1], %[t1], 3\n\t"
      "s_sub_u32 %[t1], 64, %[t1]\n\t"
      "s_cmp_eq_u32 %[t0], 0\n\t"
      "v_mov_b32 %[vval], %[t1]\n\t"
      "ds_write_b32 %[vfull], %[vval]\n\t"
      "s_sleep 1\n\t"
      "s_cbranch_scc0 Ldrain%=\n\t"
      "Lquit%=:\n\t"
      "s_waitcnt vmcnt(0) lgkmcnt(0)\n\t"
      : [iss] "=&s"(s_iss), [lds] "=&s"(s_lds), [t0] "=&s"(s_t0), [t1] "=&s"(s_t1), [t2] "=&s"(s_t2), [spin] "=&s"(s_spin), [wrap] "=&s"(s_wrap), [vval] "=&v"(vval)
      : [wlo] "s"((unsigned)(size_t)wbase), [whi] "s"((unsigned)((size_t)wbase >> 32)),
        [o0] "v"(voff[0]), [o1] "v"(voff[1]), [o2] "v"(voff[2]), [o3] "v"(voff[3]),
        [o4] "v"(voff[4]), [o5] "v"(voff[5]), [o6] "v"(voff[6]), [o7] "v"(voff[7]),
        [vfull] "v"(vfull), [vrd] "v"(vrd), [fc] "s"(first_chunk),
        [ldsbeg] "i"(L_RING), [ldsend] "i"(L_RING + 6 * 16384), [nfirst] "i"(2 * NFIRST), [depth] "i"(WDEPTH), [bmask] "i"(WBURST - 1), [thr0] "i"(WTHR0), [lastq] "i"(XLAST ? 29 : 31), [wraplast] "i"(XLAST ? L_RING + 6 * 16384 : L_RING), [thr1] "i"(WMAXOUT)
      : "memory", "scc", "s40", "s41", "s42", "s43", "v60");
}

DI void st16_sys(unsigned char* p, v4u v) {
  asm volatile("global_store_dwordx4 %0, %1, off " PUSHPOL "\n\ts_nop 1" ::"v"(p), "v"(v) : "memory");
}
DI int lane_id() { int l; asm volatile("v_mbcnt_lo_u32_b32 %0, -1, 0\n\tv_mbcnt_hi_u32_b32 %0, -1, %0" : "=v"(l)); return l; }
DI void wait_vm0() { asm volatile("s_waitcnt vmcnt(0)" ::: "memory"); }

__global__ __launch_bounds__(448, 2) void a2a_gemm_kernel(Params p) {
  extern __shared__ __attribute__((aligned(16))) unsigned char smem[];
  const int wid = rfl(threadIdx.x >> 6);
  const int bid = blockIdx.x;
  {
    const void* kp = (const void*)__builtin_amdgcn_kernarg_segment_ptr();
    const unsigned poff = 64u + (((unsigned)bid >> 3) & 7u) * 8u;
    asm volatile("s_load_dwordx16 s[80:95], %0, 0x0\n\ts_load_dwordx2 s[96:97], %0, %1" ::"s"(kp), "s"(poff)
                 : "memory", "s80", "s81", "s82", "s83", "s84", "s85", "s86", "s87", "s88", "s89", "s90", "s91", "s92", "s93", "s94", "s95", "s96", "s97");
    if (wid < 5) asm volatile("s_lshl_b32 m0, %0, 8\n\tv_mov_b32 v1, 0\n\tds_write_addtid_b32 v1" ::"s"(wid) : "memory", "v1", "scc");
  }
#define ARGS(BAR) \
  const unsigned char *px, *pw, *psx, *psw; \
  unsigned char *pout, *myws, *dws; \
  int rank; \
  asm volatile("s_waitcnt lgkmcnt(0)\n\t" BAR \
               "s_mov_b64 %0, s[80:81]\n\ts_mov_b64 %1, s[82:83]\n\ts_mov_b64 %2, s[84:85]\n\ts_mov_b64 %3, s[86:87]\n\t" \
               "s_mov_b64 %4, s[88:89]\n\ts_mov_b64 %5, s[90:91]\n\ts_mov_b64 %6, s[96:97]\n\ts_mov_b32 %7, s92" \
               : "=s"(px), "=s"(pw), "=s"(psx), "=s"(psw), "=s"(pout), "=s"(myws), "=s"(dws), "=s"(rank) \
               : \
               : "memory", "s80", "s81", "s82", "s83", "s84", "s85", "s86", "s87", "s88", "s89", "s90", "s91", "s92", "s93", "s94", "s95", "s96", "s97");
  const int xcd = bid & 7, idx = bid >> 3;
  const int tn = xcd * 16 + (idx >> 1), tm = idx & 1;
  const int m0 = tm * 256, n0 = tn * 128;

  if (wid == W_PUSH) {
    const int c = bid >> 6;
    const int di = (bid >> 3) & 7, u = bid & 7;
    const int lane = lane_id();
    const unsigned voff = (lane & 15) * 512 + (lane >> 4) * 16;
    if (di == 7) { asm volatile("s_waitcnt lgkmcnt(0)" ::: "memory"); __builtin_amdgcn_endpgm(); }
    ARGS("");
    const int dst = (rank + 1 + di) & (NRANK - 1);
    const unsigned char* src = px + (size_t)(dst * 512 + u * 64) * 512 + c * 128;
    unsigned* cnt = (unsigned*)(myws + WS_CNT + (size_t)bid * 128);
    unsigned prev;
    __builtin_amdgcn_s_setprio(3);
    const unsigned long long t0 = __builtin_amdgcn_s_memrealtime();
    if (c != 0 || !PUSH_FIRST) asm volatile("s_barrier" ::: "memory");
    if (c != 0) {
      while (__builtin_amdgcn_s_memrealtime() - t0 < (unsigned long long)(EARLYLD ? EARLYLD : c * PUSH_PACE)) __builtin_amdgcn_s_sleep(8);
    }
    v4u d[8];
#pragma unroll
    for (int q = 0; q < 4; ++q) {
      asm volatile("global_load_dwordx4 %0, %1, %2" : "=&v"(d[2 * q]) : "v"(voff), "s"(src + q * 8192) : "memory");
      asm volatile("global_load_dwordx4 %0, %1, %2 offset:64" : "=&v"(d[2 * q + 1]) : "v"(voff), "s"(src + q * 8192) : "memory");
    }
    asm volatile("global_load_dword %0, %1, off sc1" : "=&v"(prev) : "v"(cnt) : "memory");
    if (c == 0 && PUSH_FIRST) asm volatile("s_barrier" ::: "memory");
    unsigned char* dstage = dws + WS_STAGE + (size_t)(rank * 4 + c) * 65536 + u * 8192 + lane * 16;
    unsigned char* dflag = dws + WS_FLAG + (size_t)((lane & (NREP - 1)) * 2 + (u >> 2)) * 128 + (rank * 4 + c) * 4 + (u & 3);
    asm volatile("s_waitcnt vmcnt(0)" : "+v"(prev), "+v"(d[0]), "+v"(d[1]), "+v"(d[2]), "+v"(d[3]), "+v"(d[4]), "+v"(d[5]), "+v"(d[6]), "+v"(d[7])::"memory");
    if (EARLYLD && c != 0) while (__builtin_amdgcn_s_memrealtime() - t0 < (unsigned long long)(100 + c * PUSH_PACE)) __builtin_amdgcn_s_sleep(8);
    const unsigned call = rfl(prev) + 1u;
    dstage += (size_t)(call & 1u) * (32 * 65536);
    dflag += (call & 1u) * (NREP * 256);
#pragma unroll
    for (int q = 0; q < 8; ++q) st16_sys(dstage + q * 1024, d[q]);
    wait_vm0();
    if (lane < NREP) asm volatile("global_store_byte %0, %1, off sc0 sc1" ::"v"(dflag), "v"(call) : "memory");
    if (lane == 0) __hip_atomic_store(cnt, call, __ATOMIC_RELAXED, __HIP_MEMORY_SCOPE_AGENT);
    __builtin_amdgcn_endpgm();
  }

  if (wid == W_LOAD) {
    const int lane = lane_id();
    unsigned voff[8];
#pragma unroll
    for (int i = 0; i < 8; ++i) {
      const int k = 8 * i + 4 * (lane >> 5) + ((lane >> 2) & 3);
      const int a = (lane >> 4) & 1, b = (lane & 3) ^ ((lane >> 5) | (i & 2));
      voff[i] = (unsigned)(k * NDIM + (4 * a + b) * 16) - (unsigned)((i & 3) * 1024);
    }
    const unsigned vrd = lane < 32 ? L_TAB + lane * 4 : L_PROG + (lane & 3) * 256;
    asm volatile("" ::"v"(voff[0]), "v"(voff[1]), "v"(voff[2]), "v"(voff[3]), "v"(voff[4]), "v"(voff[5]), "v"(voff[6]), "v"(voff[7]), "v"(vrd));
    ARGS("s_barrier\n\t");
    loader_w(pw + n0, voff, vrd, (unsigned)rank * 4u);
    __builtin_amdgcn_endpgm();
  }

  if (wid == W_POLL) {
    const int lane = lane_id();
    ARGS("s_barrier\n\t");
    unsigned* cnt = (unsigned*)(myws + WS_CNT + (size_t)bid * 128 + 64);
    unsigned prev;
    asm volatile("global_load_dword %0, %1, off sc1" : "=&v"(prev) : "v"(cnt) : "memory");
    if (lane < 4) ((volatile LAS unsigned*)(size_t)L_TAB)[lane] = (unsigned)(rank * 4 + lane + 1);
    const int q = (lane ^ (tm ? PAIR_SWAP : 0)) & 31;
    const unsigned id = (((unsigned)rank + 1u + (unsigned)(q % 7)) & 7u) * 4u + (unsigned)((q / 7) & 3);
    asm volatile("s_waitcnt vmcnt(0)" : "+v"(prev)::"memory");
    const unsigned call = rfl(prev) + 1u;
    const unsigned par = call & 1u;
    const unsigned want = (call & 255u) * 0x01010101u;
    unsigned* tmo = (unsigned*)(myws + WS_TMO);
    const unsigned char* fa = myws + WS_FLAG + (size_t)((par * NREP + (bid & (NREP - 1))) * 2 + tm) * 128 + id * 4;
    unsigned done = 0xF0000000u;
    unsigned ntab = 4;
    const unsigned long long t0 = __builtin_amdgcn_s_memrealtime();
    if (lane == 0) __hip_atomic_store(cnt, call, __ATOMIC_RELAXED, __HIP_MEMORY_SCOPE_AGENT);
    unsigned polls = 0;
    unsigned toff;
    {
      const unsigned l0 = (unsigned)((size_t)pw >> 7) + xcd * 16;
      const unsigned b0 = (3u - l0) & 7u, b2 = (63u - l0) & 127u;
      const unsigned t = (lane >> 2) & 3;
      const unsigned tile = t == 1 ? b0 + 8 : (t == 2 && b2 < 16u) ? b2 : b0;
      toff = (unsigned)(idx * 4 + (lane & 3)) * NDIM + (xcd * 16 + tile) * 128;
    }
    unsigned ntouch = TOUCH ? 0u : 7u;
    unsigned natouch = ATOUCH ? 1u : 8u;
    const unsigned aoff = (unsigned)(tm * 32768 + (idx >> 1) * 2048 + (lane & 15) * 128);
    while (ntab < 32 || ntouch < 7 || natouch < 8) {
      const unsigned prog = rfl(((volatile LAS unsigned*)(size_t)L_PROG)[0]);
      if (ntouch < 7 && ntouch * 4 <= prog + TOUCH_AHEAD && rfl(((volatile LAS unsigned*)(size_t)L_TAB)[32]) >= TOUCH_START) {
        const int qq = (int)ntouch * 4 + (lane >> 4);
        const unsigned cid = (((unsigned)rank + 1u + (unsigned)(qq % 7)) & 7u) * 4u + (unsigned)(qq / 7);
        asm volatile("global_load_dword v200, %0, %1" ::"v"(toff + cid * (128u * NDIM)), "s"(pw) : "memory", "v200");
        ++ntouch;
        continue;
      }
      if (natouch < 8 && natouch * 4 + 3 < ntab && natouch * 4 <= prog + ATOUCH_AHEAD) {
        const unsigned ent = ((volatile LAS unsigned*)(size_t)L_TAB)[natouch * 4 + (lane >> 4)];
        asm volatile("global_load_dword v201, %0, %1" ::"v"(aoff + (ent - 1u) * 65536u), "s"(myws + WS_STAGE) : "memory", "v201");
        ++natouch;
        continue;
      }
      if (ntab >= 32 || ntab > prog + POLL_AHEAD) { __builtin_amdgcn_s_sleep(8); continue; }
      unsigned f;
      asm volatile("global_load_dword %0, %1, off sc0 sc1\n\ts_waitcnt vmcnt(0)" : "=&v"(f) : "v"(fa) : "memory");
      unsigned fresh = (unsigned)__ballot(f == want) & ~done;
      ++polls;
      if (fresh == 0) {
        if (((polls & 63u) == 0u && rfl(__hip_atomic_load(tmo, __ATOMIC_RELAXED, __HIP_MEMORY_SCOPE_AGENT)) == call) ||
            __builtin_amdgcn_s_memrealtime() - t0 > DEADLINE_TICKS) {
          if (lane == 0) __hip_atomic_store(tmo, call, __ATOMIC_RELAXED, __HIP_MEMORY_SCOPE_AGENT);
          if (lane == 0) ((volatile LAS unsigned*)(size_t)L_TAB)[33] = 1u;
          asm volatile("s_waitcnt lgkmcnt(0)" ::: "memory");
          fresh = ~done;
        } else {
          __builtin_amdgcn_s_sleep(POLL_SLEEP);
        }
      }
      if (fresh) {
        const bool take = lane < 32 && ((fresh >> lane) & 1u);
        const unsigned pos = ntab + __builtin_amdgcn_mbcnt_lo(fresh, 0);
        if (take) ((volatile LAS unsigned*)(size_t)L_TAB)[pos] = par * 32u + id + 1u;
        ntab += __builtin_popcount(fresh);
        done |= fresh;
      }
    }
    if (OTOUCH) {
      const unsigned char* po = pout + ((size_t)m0 * NDIM + n0) * 4;
#pragma unroll
      for (int r = 0; r < OTOUCH; ++r) asm volatile("global_load_dword v202, %0, %1 nt" ::"v"((unsigned)(lane * 4 + r) * (NDIM * 4)), "s"(po) : "memory", "v202");
    }
    __builtin_amdgcn_endpgm();
  }

  const int cw = wid - W_COMP;
  ARGS("s_barrier\n\t");
  const unsigned char* xloc = px + (size_t)(rank * 512 + m0 + cw * 64) * 512;
  const unsigned char* wsa = myws + WS_STAGE + (size_t)(tm * 16 + cw * 4) * 2048 + 4096 - 65536;
  unsigned char* outp = pout + ((size_t)(m0 + cw * 64) * NDIM + n0) * 4;
  asm volatile(COMPUTE_ASM
      :
      : [xloc] "s"(xloc), [wsa] "s"(wsa), [out] "s"(outp), [sx] "s"(psx), [sw] "s"(psw), [wid] "s"(cw)
      : "memory", "vcc", "scc", CLOB_V, CLOB_S);
}

static unsigned long long* g_stamps = nullptr;

extern "C" void kernel_launch(int rank, int nranks, ncclComm_t comm,
                              hipStream_t stream,
                              void* const* d_in, const int* in_sizes, int n_in,
                              void* d_out, int out_size,
                              const int* in_dtypes, int out_dtype,
                              void* const* d_out_all,
                              void* const* d_ws_all, size_t ws_bytes) {
  if (nranks != NRANK || (unsigned)rank >= (unsigned)NRANK || n_in != 4 || ws_bytes < WS_NEED ||
      in_sizes[0] != 4096 * 512 || in_sizes[1] != 4096 * NDIM || out_size != 512 * NDIM) {
    fprintf(stderr, "kernel_launch: unexpected problem shape\n");
    return;
  }
  HIP_CHECK(hipFuncSetAttribute((const void*)a2a_gemm_kernel, hipFuncAttributeMaxDynamicSharedMemorySize, LDS_BYTES));
  Params p;
  p.x = (const unsigned char*)d_in[0];
  p.w = (const unsigned char*)d_in[1];
  p.sx = (const float*)d_in[2];
  p.sw = (const float*)d_in[3];
  p.out = (float*)d_out;
  p.myws = (unsigned char*)d_ws_all[rank];
  for (int i = 0; i < NRANK; ++i) p.peer[i] = (unsigned char*)d_ws_all[(rank + 1 + i) % NRANK];
  p.rank = rank;
  p.pad0 = 0;
  p.stamps = g_stamps;
  hipLaunchKernelGGL(a2a_gemm_kernel, dim3(256), dim3(448), LDS_BYTES, stream, p);
  HIP_CHECK(hipGetLastError());
}
```
